```python
import numpy as np

kernel_name = "gpu_ag_gemm_m4096_k4096_n16384_e4m3e4m3_dqsilu"


ABSMAX_RTOL = 0.03

COMPUTE_DTYPE = "float32"
EPILOGUE = "dequant_silu"
IN_QUANT = None
OUT_QUANT = None
IN_FP8 = "e4m3e4m3"
IN_DTYPES = {"x": "e4m3", "w_mat": "e4m3",
             "scale_x": "float32", "scale_w": "float32"}
WORLD_SIZE = 8

def _snap_fp8(x, mant_bits, max_abs, min_normal):
    a = np.abs(x.astype(np.float64))
    _m, _e = np.frexp(a)
    q = np.where(a >= min_normal, np.ldexp(1.0, _e - 1 - mant_bits),
                 min_normal * 2.0 ** -mant_bits)
    snapped = np.minimum(np.round(a / q) * q, max_abs)
    return (np.sign(x) * snapped).astype(np.float32)

def setup_inputs():

    x = _snap_fp8(np.random.randn(4096, 4096).astype(np.float32), 3, 448.0, 2.0 ** -6)
    w_mat = _snap_fp8(np.random.randn(4096, 16384).astype(np.float32), 3, 448.0, 2.0 ** -6)
    scale_x = (0.02 * (1.0 + np.random.rand(1))).astype(np.float32)
    scale_w = (0.02 * (1.0 + np.random.rand(1))).astype(np.float32)
    return {"x": x, "w_mat": w_mat, "scale_x": scale_x, "scale_w": scale_w}

def reference(x, w_mat, scale_x, scale_w):
    acc = (x @ w_mat).astype(np.float32)
    y = (acc * (scale_x * scale_w)).astype(np.float32)
    return (y / (1.0 + np.exp(-np.clip(y, -60.0, 60.0)))).astype(np.float32)
```

```cpp
#include <hip/hip_runtime.h>
#include <rccl/rccl.h>
#include <cstdio>

#define HIP_CHECK(expr) do { hipError_t _e = (expr); if (_e != hipSuccess) { \
    fprintf(stderr, "HIP error %s at %s:%d\n", hipGetErrorString(_e), __FILE__, __LINE__); \
    return; } } while (0)

namespace {

typedef int v8i __attribute__((ext_vector_type(8)));
typedef unsigned u32x4 __attribute__((ext_vector_type(4)));
typedef float v4f __attribute__((ext_vector_type(4)));
#define LAS __attribute__((address_space(3)))
#define GAS __attribute__((address_space(1)))

constexpr int NR = 8;
constexpr int KD = 4096;
constexpr int NL = 2048;
constexpr int SH = 512;
constexpr int IROWS = 16;
constexpr int NI = SH / IROWS;
constexpr int IBYTES = IROWS * KD;
constexpr int SHBYTES = SH * KD;
constexpr int NP = 32;
constexpr int NWG = 256;
#define RGX 8
#define A_AUX 17
#define OUT_AUX 16
#define NO_STORE_T 2
#define PU 4
#define PWV 2
constexpr int CGX = 32 / RGX;
constexpr int FW = (IBYTES / 1024) / PU;
constexpr int NU = NI * FW;
constexpr int NPW = NP * PWV;
constexpr int TICKETS = NU;

constexpr size_t WS_CNT = 0;
constexpr size_t WS_TMO = 32768;
constexpr size_t WS_TKT = 32768 + 256;
constexpr size_t WS_FLAG = 65536;
constexpr size_t WS_STAGE = 1u << 20;
constexpr size_t WS_NEED = WS_STAGE + 2ull * NR * SHBYTES;
constexpr size_t WS_DBG = 64u << 20;
constexpr unsigned long long DEADLINE_TICKS = 40000000ull;
static_assert(WS_FLAG + (size_t)NR * NU * 4 <= WS_STAGE, "flags");
static_assert(NU <= (NWG - NP) * 4, "one local unit per consumer wave");

struct Params {
    char* ws[NR];
    const char* x;
    const char* w;
    const float* sx;
    const float* sw;
    float* out;
    int rank;
    int pad;
};

#define STAMP(i) do {} while (0)

__device__ __forceinline__ __amdgpu_buffer_rsrc_t mk_rsrc(const void* base, unsigned bytes) {
    return __builtin_amdgcn_make_buffer_rsrc((void*)base, 0, bytes, 0x00020000);
}

__device__ __forceinline__ float silu_dq(float acc, float s) {
    const float y = acc * s;
    const float c = __builtin_amdgcn_fmed3f(y, -60.0f, 60.0f);
    const float e = __builtin_amdgcn_exp2f(c * -1.44269504088896340736f);
    return y * __builtin_amdgcn_rcpf(1.0f + e);
}

constexpr int RED_BYTES = 4 * 3 * 1024;

__global__ __launch_bounds__(256, 1) void ag_gemm_kernel(const Params p) {
    __shared__ __attribute__((aligned(16))) char lds[2 * RED_BYTES];
    const int tid = threadIdx.x;
    const int lane = tid & 63;
    const int wid = __builtin_amdgcn_readfirstlane(tid >> 6);
    const int b = blockIdx.x;
    const unsigned rank = (unsigned)p.rank;
    char* const myws = p.ws[rank & (NR - 1)];

    GAS unsigned* cnt = (GAS unsigned*)(myws + WS_CNT + (size_t)b * 128 + wid * 4);

    const int rg = ((b & 7) / RGX) * RGX + (b >> 3) / CGX;
    const int cg = ((b & 7) % RGX) * CGX + (b >> 3) % CGX;
    STAMP(0);
    if (rg == 0) {
        if (wid >= PWV) {
            const unsigned cv = __hip_atomic_load(cnt, __ATOMIC_RELAXED, __HIP_MEMORY_SCOPE_AGENT);
            if (lane == 0) __hip_atomic_store(cnt, cv + 1u, __ATOMIC_RELAXED, __HIP_MEMORY_SCOPE_AGENT);
            return;
        }
        const int n = lane & 15, q = lane >> 4;
        const int v = wid * NP + cg;
        GAS unsigned* const tkt = (GAS unsigned*)(myws + WS_TKT);
        u32x4 d0[PU], d1[PU];
        unsigned tk = 0;
#define PUSH_SRC(U) (p.x + (size_t)(((U) / FW) * IROWS + ((n & 12) | ((n & 3) ^ ((((U) % FW) * PU) >> 4)))) * KD + ((U) % FW) * (PU * 64) + q * 16)
#define PUSH_LOAD(D, U) do { const char* s_ = PUSH_SRC(U); _Pragma("unroll") for (int i_ = 0; i_ < PU; ++i_) \
        asm volatile("global_load_dwordx4 %0, %1, off offset:%2" : "=&v"(D[i_]) : "v"(s_), "i"(i_ * 64) : "memory"); } while (0)
#define PUSH_PIN(D) do { _Pragma("unroll") for (int i_ = 0; i_ < PU; ++i_) asm volatile("" : "+v"(D[i_])); } while (0)
        PUSH_LOAD(d0, v);
        const unsigned cv = __hip_atomic_load(cnt, __ATOMIC_RELAXED, __HIP_MEMORY_SCOPE_AGENT);
        asm volatile("s_waitcnt vmcnt(0)" ::: "memory");
        PUSH_PIN(d0);
        const unsigned call = (unsigned)__builtin_amdgcn_readfirstlane(cv) + 1u;
        if (lane == 0) __hip_atomic_store(cnt, call, __ATOMIC_RELAXED, __HIP_MEMORY_SCOPE_AGENT);
        const unsigned tbase = (call - 1u) * (unsigned)TICKETS - 2u * NPW;
        const size_t soff = WS_STAGE + (size_t)(call & 1u) * (NR * SHBYTES) + (size_t)rank * SHBYTES;
        const unsigned fpeer = (rank + 1u + (unsigned)lane) & (NR - 1);
        char* fws = p.ws[0];
#pragma unroll
        for (int i = 1; i < NR; ++i) fws = fpeer == (unsigned)i ? p.ws[i] : fws;
        GAS unsigned* const flag = (GAS unsigned*)(fws + WS_FLAG) + rank * NU;
        const unsigned one = 1u;
        STAMP(1);
#define PUSH_ROUND(DC, DN, U, UN) do { \
        if (lane == 0) asm volatile("global_atomic_add %0, %1, %2, off sc0" : "=&v"(tk) : "v"(tkt), "v"(one) : "memory"); \
        if ((UN) < NU) PUSH_LOAD(DN, UN); \
        _Pragma("unroll") for (int pr_ = 0; pr_ < NR - 1; ++pr_) { \
            char* t_ = p.ws[(rank + 1u + (unsigned)((pr_ + v) % (NR - 1))) & (NR - 1)] + soff + (size_t)(U) * (PU * 1024) + lane * 16; \
            _Pragma("unroll") for (int i_ = 0; i_ < PU; ++i_) \
                asm volatile("global_store_dwordx4 %0, %1, off offset:%2 sc0 sc1\n\ts_nop 1" :: "v"(t_), "v"(DC[i_]), "i"(i_ * 1024) : "memory"); } \
        asm volatile("s_waitcnt vmcnt(0)" : "+v"(tk) :: "memory"); \
        PUSH_PIN(DN); \
        if (lane < NR - 1) asm volatile("global_store_dword %0, %1, off sc0 sc1" :: "v"(flag + (U)), "v"(call) : "memory"); \
        } while (0)
        int u = v, un = v + NPW;
        int rounds = 0;
        for (;;) {
            PUSH_ROUND(d0, d1, u, un);
            STAMP(2 + (rounds < 40 ? rounds : 40)); ++rounds;
            if (un >= NU) break;
            u = un; un = (int)((unsigned)__builtin_amdgcn_readfirstlane(tk) - tbase);
            PUSH_ROUND(d1, d0, u, un);
            STAMP(2 + (rounds < 40 ? rounds : 40)); ++rounds;
            if (un >= NU) break;
            u = un; un = (int)((unsigned)__builtin_amdgcn_readfirstlane(tk) - tbase);
        }
        STAMP(63);
        return;
    }

    const unsigned cv = __hip_atomic_load(cnt, __ATOMIC_RELAXED, __HIP_MEMORY_SCOPE_AGENT);
    const int pi = rg - 1;
    const unsigned srank = (rank + 1u + (unsigned)pi) & (NR - 1);
    const int nloc = pi < (NI - 4 * (NR - 1)) ? 5 : 4;
    const int q = lane >> 4, n = lane & 15;
#define SB0 __builtin_amdgcn_sched_barrier(0)

    const int lu = (pi * 32 + cg) * 4 + wid;
    u32x4 lx[PU];
    {
        const __amdgpu_buffer_rsrc_t rx = mk_rsrc(p.x, lu < NU ? (unsigned)SHBYTES : 0u);
        const int vo = ((n & 12) | ((n & 3) ^ (((lu % FW) * PU) >> 4))) * KD + q * 16;
#pragma unroll
        for (int i = 0; i < PU; ++i) lx[i] = __builtin_amdgcn_raw_buffer_load_b128(rx, vo, (lu / FW) * IBYTES + ((lu % FW) * PU + i) * 64, 0);
    }
    v8i Wf[8][4];
    unsigned call;
    const char* stage;
    {
        const __amdgpu_buffer_rsrc_t rw = mk_rsrc(p.w, (unsigned)KD * NL);
        const int voff = (q * 16) * NL + n * 4;
        const int sbase = (wid * 1024) * NL + cg * 64;
        unsigned d[2][8][4];
#define W_LOAD(KS) do { _Pragma("unroll") for (int r = 0; r < 8; ++r) _Pragma("unroll") for (int c = 0; c < 4; ++c) \
            d[(KS) & 1][r][c] = __builtin_amdgcn_raw_buffer_load_b32(rw, voff, sbase + ((KS) * 128 + (r >> 2) * 64 + (r & 3) * 4 + c) * NL, 0); } while (0)
        W_LOAD(0);
        SB0;
        call = (unsigned)__builtin_amdgcn_readfirstlane(cv) + 1u;
        if (lane == 0) __hip_atomic_store(cnt, call, __ATOMIC_RELAXED, __HIP_MEMORY_SCOPE_AGENT);
        stage = myws + WS_STAGE + (size_t)(call & 1u) * (NR * SHBYTES);
        {
            const __amdgpu_buffer_rsrc_t rl = mk_rsrc(stage + (size_t)rank * SHBYTES, lu < NU ? (unsigned)SHBYTES : 0u);
#pragma unroll
            for (int i = 0; i < PU; ++i) __builtin_amdgcn_raw_buffer_store_b128(lx[i], rl, lane * 16, (lu * PU + i) * 1024, 17);
        }
        SB0;
#pragma unroll
        for (int ks = 0; ks < 8; ++ks) {
            if (ks + 1 < 8) W_LOAD(ks + 1);
            SB0;
#pragma unroll
            for (int r = 0; r < 8; ++r) {
                const unsigned* e = d[ks & 1][r];
                const unsigned t0 = __builtin_amdgcn_perm(e[1], e[0], 0x05010400u), t1 = __builtin_amdgcn_perm(e[1], e[0], 0x07030602u);
                const unsigned t2 = __builtin_amdgcn_perm(e[3], e[2], 0x05010400u), t3 = __builtin_amdgcn_perm(e[3], e[2], 0x07030602u);
                Wf[ks][0][r] = (int)__builtin_amdgcn_perm(t2, t0, 0x05040100u);
                Wf[ks][1][r] = (int)__builtin_amdgcn_perm(t2, t0, 0x07060302u);
                Wf[ks][2][r] = (int)__builtin_amdgcn_perm(t3, t1, 0x05040100u);
                Wf[ks][3][r] = (int)__builtin_amdgcn_perm(t3, t1, 0x07060302u);
            }
#pragma unroll
            for (int j = 0; j < 4; ++j) asm volatile("" : "+a"(Wf[ks][j]));
            SB0;
            if (ks == 1) {
                asm volatile("s_waitcnt vmcnt(32)" ::: "memory");
                if (lane == 0 && lu < NU) __hip_atomic_store((GAS unsigned*)(myws + WS_FLAG) + rank * NU + lu, call, __ATOMIC_RELAXED, __HIP_MEMORY_SCOPE_SYSTEM);
                SB0;
            }
        }
    }
    const float scale = p.sx[0] * p.sw[0];
    STAMP(1);

    GAS unsigned* const tmo = (GAS unsigned*)(myws + WS_TMO);
    const __amdgpu_buffer_rsrc_t rs = mk_rsrc(stage, (unsigned)NR * SHBYTES);
    const __amdgpu_buffer_rsrc_t rf = mk_rsrc(myws + WS_FLAG + (size_t)srank * (NU * 4), (unsigned)NU * 4u);
    const int voff_a = lane * 16;
    const int voff_out = ((q * 4 + wid) * NL + n * 4) * 4;
    v4f cinit = {0.f, 0.f, 0.f, 0.f};
#define SPIN(READY_EXPR, LOADV) do { \
        const unsigned long long t0_ = __builtin_amdgcn_s_memrealtime(); \
        for (;;) { \
            const unsigned v_ = LOADV; \
            if (READY_EXPR) break; \
            if ((unsigned)__builtin_amdgcn_readfirstlane(__hip_atomic_load(tmo, __ATOMIC_RELAXED, __HIP_MEMORY_SCOPE_AGENT)) == call || \
                __builtin_amdgcn_s_memrealtime() - t0_ > DEADLINE_TICKS) { \
                if (lane == 0) __hip_atomic_store(tmo, call, __ATOMIC_RELAXED, __HIP_MEMORY_SCOPE_AGENT); \
                cinit = (v4f){__builtin_nanf(""), __builtin_nanf(""), __builtin_nanf(""), __builtin_nanf("")}; \
                rdy = NI; \
                break; \
            } \
            __builtin_amdgcn_s_sleep(1); \
        } } while (0)
#define LEADING(V) ({ const unsigned long long m_ = __ballot((int)((V) - call) >= 0); (m_ == ~0ull ? 64 : __builtin_ctzll(~m_)) / FW; })
    int rdy = 0;
    {
        const __amdgpu_buffer_rsrc_t rfl = mk_rsrc(myws + WS_FLAG + (size_t)rank * (NU * 4), (unsigned)NU * 4u);
        for (int i = 0; i < nloc; i += 64 / FW) {
            const int it = i + lane / FW;
            const int vo = ((pi + 7 * it) * FW + lane % FW) * 4;
            SPIN(__all(it >= nloc || (int)(v_ - call) >= 0), __builtin_amdgcn_raw_buffer_load_b32(rfl, vo, 0, 17));
        }
    }

    u32x4 Af[8][2];
#define LOAD_FRAG(KS, RS, SB) do { \
        Af[KS][0] = __builtin_amdgcn_raw_buffer_load_b128(RS, voff_a + (((KS) & 1) * 2048), (SB) + ((KS) >> 1) * 4096, A_AUX); \
        Af[KS][1] = __builtin_amdgcn_raw_buffer_load_b128(RS, voff_a + (((KS) & 1) * 2048 + 1024), (SB) + ((KS) >> 1) * 4096, A_AUX); } while (0)
    LAS char* const put1 = (LAS char*)lds + ((1 ^ wid) * 3 + 0) * 1024 + lane * 16;
    LAS char* const put2 = (LAS char*)lds + ((2 ^ wid) * 3 + 1) * 1024 + lane * 16;
    LAS char* const put3 = (LAS char*)lds + ((3 ^ wid) * 3 + 2) * 1024 + lane * 16;
    LAS char* const get = (LAS char*)lds + wid * 3072 + lane * 16;
#define MFMA0(J) asm volatile("v_mfma_f32_16x16x128_f8f6f4 %0, %1, %2, %3" : "=&v"(acc[J]) : "v"(a_), "a"(Wf[0][J]), "v"(cinit))
#define MFMA(KS, J) asm volatile("v_mfma_f32_16x16x128_f8f6f4 %0, %1, %2, %0" : "+v"(acc[J]) : "v"(a_), "a"(Wf[KS][J]))
#define AFRAG(KS) const v8i a_ = __builtin_bit_cast(v8i, __builtin_shufflevector(Af[KS][0], Af[KS][1], 0, 1, 2, 3, 4, 5, 6, 7))
#define TAKE(J) do { _Pragma("unroll") for (int r = 0; r < 4; ++r) { P[r][J] = acc[J][r]; } \
        asm volatile("" : "+v"(P[0]), "+v"(P[1]), "+v"(P[2]), "+v"(P[3])); } while (0)

#define PIN(X) asm volatile("" : "+v"(X))
#define OP4_s(OP, V) asm volatile(OP " %0, %0, %4\n\t" OP " %1, %1, %5\n\t" OP " %2, %2, %6\n\t" OP " %3, %3, %7" : "+v"(s0), "+v"(s1), "+v"(s2), "+v"(s3) : "v"(V[0]), "v"(V[1]), "v"(V[2]), "v"(V[3]))
#define OP4_g(OP, V) asm volatile(OP " %0, %0, %4\n\t" OP " %1, %1, %5\n\t" OP " %2, %2, %6\n\t" OP " %3, %3, %7" : "+v"(g0), "+v"(g1_), "+v"(g2_), "+v"(g3_) : "v"(V[0]), "v"(V[1]), "v"(V[2]), "v"(V[3]))
#define OP4(OP, D, V) OP4_##D(OP, V)
    float c60 = 60.0f;
    PIN(c60);
#define NEXT_SA(TN, SA) ((TN) == 0 ? (int)srank * SHBYTES + wid * 16384 : (SA) + ((TN) < 0 ? 7 * IBYTES : IBYTES))
    int sa = (int)rank * SHBYTES + pi * IBYTES + wid * 16384;
    int san = sa + 7 * IBYTES;
    int so = (((int)rank * SH + pi * IROWS) * NL + cg * 64) * 4;
    int so1 = 0, so2 = 0;
    bool have = false;
    unsigned nrec = (unsigned)NR * SHBYTES;
    unsigned orec = 0;
    const char* outp = (const char*)p.out;
    asm volatile("" : "+s"(outp));
    PIN(cinit);
    v4f acc[4], P[4];
#pragma unroll
    for (int j = 0; j < 4; ++j) { acc[j] = (v4f){0.f, 0.f, 0.f, 0.f}; P[j] = acc[j]; }
    int bo = 0;
    for (int tn = 1 - nloc; tn <= NI; ++tn) {
        v4f g1 = *(LAS v4f*)(get + bo), g2 = *(LAS v4f*)(get + bo + 1024), g3 = *(LAS v4f*)(get + bo + 2048);
        if (!have) {
            if (tn > rdy) {
                SPIN(((rdy += LEADING(v_)) >= tn), __builtin_amdgcn_raw_buffer_load_b32(rf, lane * 4, rdy * (FW * 4), 17));
                PIN(cinit);
                nrec = tn < rdy ? (unsigned)NR * SHBYTES : 0u;
            }
            SB0;
#pragma unroll
            for (int ks = 0; ks < 8; ++ks) { LOAD_FRAG(ks, rs, sa); SB0; }
        }
        unsigned fv = __builtin_amdgcn_raw_buffer_load_b32(rf, lane * 4, rdy * (FW * 4), 17);
        const __amdgpu_buffer_rsrc_t rn = mk_rsrc(stage, nrec);
        const __amdgpu_buffer_rsrc_t ro = mk_rsrc(outp, orec);
        float s0 = P[0][0], s1 = P[0][1], s2 = P[0][2], s3 = P[0][3];
        float g0 = g2[0], g1_ = g2[1], g2_ = g2[2], g3_ = g2[3];
        bo = RED_BYTES - bo;
        SB0;
        { AFRAG(0); TAKE(0); SB0; MFMA0(0); SB0; TAKE(1); SB0; MFMA0(1); SB0; TAKE(2); SB0; MFMA0(2); SB0; TAKE(3); SB0; MFMA0(3); SB0; }
        LOAD_FRAG(0, rn, san); SB0;
        { AFRAG(1); MFMA(1, 0); SB0; *(LAS v4f*)(put1 + bo) = P[1]; SB0; MFMA(1, 1); SB0; *(LAS v4f*)(put2 + bo) = P[2]; SB0;
          MFMA(1, 2); SB0; *(LAS v4f*)(put3 + bo) = P[3]; SB0; MFMA(1, 3); SB0; }
        LOAD_FRAG(1, rn, san); SB0;
        { AFRAG(2); MFMA(2, 0); OP4("v_add_f32", s, g1); MFMA(2, 1); OP4("v_add_f32", g, g3); MFMA(2, 2);
          asm volatile("v_add_f32 %0, %0, %4\n\tv_add_f32 %1, %1, %5\n\tv_add_f32 %2, %2, %6\n\tv_add_f32 %3, %3, %7"
                       : "+v"(s0), "+v"(s1), "+v"(s2), "+v"(s3) : "v"(g0), "v"(g1_), "v"(g2_), "v"(g3_));
          MFMA(2, 3); SB0; }
        LOAD_FRAG(2, rn, san); SB0;
        { AFRAG(3); MFMA(3, 0);
          asm volatile("v_mul_f32 %0, %0, %4\n\tv_mul_f32 %1, %1, %4\n\tv_mul_f32 %2, %2, %4\n\tv_mul_f32 %3, %3, %4" : "+v"(s0), "+v"(s1), "+v"(s2), "+v"(s3) : "v"(scale));
          MFMA(3, 1);
          asm volatile("v_med3_f32 %0, %4, %8, %9\n\tv_med3_f32 %1, %5, %8, %9\n\tv_med3_f32 %2, %6, %8, %9\n\tv_med3_f32 %3, %7, %8, %9"
                       : "=&v"(g0), "=&v"(g1_), "=&v"(g2_), "=&v"(g3_) : "v"(s0), "v"(s1), "v"(s2), "v"(s3), "s"(-60.0f), "v"(c60));
          MFMA(3, 2);
          asm volatile("v_mul_f32 %0, %4, %0\n\tv_mul_f32 %1, %4, %1\n\tv_mul_f32 %2, %4, %2\n\tv_mul_f32 %3, %4, %3" : "+v"(g0), "+v"(g1_), "+v"(g2_), "+v"(g3_) : "s"(-1.44269504088896340736f));
          MFMA(3, 3); SB0; }
        LOAD_FRAG(3, rn, san); SB0;
        { AFRAG(4); MFMA(4, 0); asm volatile("v_exp_f32 %0, %0\n\tv_exp_f32 %1, %1" : "+v"(g0), "+v"(g1_)); MFMA(4, 1);
          asm volatile("v_exp_f32 %0, %0\n\tv_exp_f32 %1, %1" : "+v"(g2_), "+v"(g3_)); MFMA(4, 2);
          asm volatile("v_add_f32 %0, 1.0, %0\n\tv_add_f32 %1, 1.0, %1\n\tv_add_f32 %2, 1.0, %2\n\tv_add_f32 %3, 1.0, %3" : "+v"(g0), "+v"(g1_), "+v"(g2_), "+v"(g3_));
          MFMA(4, 3); SB0; }
        LOAD_FRAG(4, rn, san); SB0;
        { AFRAG(5); MFMA(5, 0); asm volatile("v_rcp_f32 %0, %0\n\tv_rcp_f32 %1, %1" : "+v"(g0), "+v"(g1_)); MFMA(5, 1);
          asm volatile("v_rcp_f32 %0, %0\n\tv_rcp_f32 %1, %1" : "+v"(g2_), "+v"(g3_)); MFMA(5, 2);
          asm volatile("v_mul_f32 %0, %0, %4\n\tv_mul_f32 %1, %1, %5\n\tv_mul_f32 %2, %2, %6\n\tv_mul_f32 %3, %3, %7"
                       : "+v"(s0), "+v"(s1), "+v"(s2), "+v"(s3) : "v"(g0), "v"(g1_), "v"(g2_), "v"(g3_));
          MFMA(5, 3); SB0; }
        LOAD_FRAG(5, rn, san); SB0;
        { AFRAG(6); MFMA(6, 0); SB0; __builtin_amdgcn_raw_buffer_store_b128(__builtin_bit_cast(u32x4, ((v4f){s0, s1, s2, s3})), ro, voff_out, so2, OUT_AUX); SB0; MFMA(6, 1); SB0;
          so2 = so1; so1 = so; so = tn == 0 ? (((int)srank * SH) * NL + cg * 64) * 4 : so + (tn < 0 ? 7 : 1) * (IROWS * NL * 4);
          orec = tn + nloc >= NO_STORE_T ? (unsigned)NR * SH * NL * 4u : 0u;
          asm volatile("" : "+s"(so), "+s"(orec)); SB0; MFMA(6, 2); MFMA(6, 3); SB0; }
        LOAD_FRAG(6, rn, san); SB0;
        int sax;
        { AFRAG(7); MFMA(7, 0); SB0; PIN(fv); rdy += LEADING(fv); asm volatile("" : "+s"(rdy)); SB0; MFMA(7, 1); SB0;
          have = nrec != 0u; nrec = (unsigned)__builtin_amdgcn_readfirstlane(tn + 1 < rdy ? NR * SHBYTES : 0); sax = __builtin_amdgcn_readfirstlane(NEXT_SA(tn + 1, san));
          asm volatile("" : "+s"(nrec), "+s"(sax)); SB0; MFMA(7, 2); MFMA(7, 3); SB0; }
        LOAD_FRAG(7, rn, san); SB0;
        sa = san; san = sax;
        STAMP(2 + tn - 1 + nloc);
        asm volatile("s_waitcnt lgkmcnt(0)\n\ts_barrier" ::: "memory");
    }
    {
        const __amdgpu_buffer_rsrc_t ro = mk_rsrc(outp, (unsigned)NR * SH * NL * 4u);
        const v4f g1 = *(LAS v4f*)(get + bo), g2 = *(LAS v4f*)(get + bo + 1024), g3 = *(LAS v4f*)(get + bo + 2048);
        const v4f own = P[0];
        asm volatile("s_nop 11" : "+v"(acc[0]), "+v"(acc[1]), "+v"(acc[2]), "+v"(acc[3]));
        TAKE(0); TAKE(1); TAKE(2); TAKE(3);
        *(LAS v4f*)(put1 + (RED_BYTES - bo)) = P[1]; *(LAS v4f*)(put2 + (RED_BYTES - bo)) = P[2]; *(LAS v4f*)(put3 + (RED_BYTES - bo)) = P[3];
        asm volatile("s_waitcnt lgkmcnt(0)\n\ts_barrier" ::: "memory");
        const v4f h1 = *(LAS v4f*)(get + (RED_BYTES - bo)), h2 = *(LAS v4f*)(get + (RED_BYTES - bo) + 1024), h3 = *(LAS v4f*)(get + (RED_BYTES - bo) + 2048);
        const v4f sa_ = (own + g1) + (g2 + g3), sb_ = (P[0] + h1) + (h2 + h3);
        v4f oa, ob;
#pragma unroll
        for (int j = 0; j < 4; ++j) { oa[j] = silu_dq(sa_[j], scale); ob[j] = silu_dq(sb_[j], scale); }
        asm volatile("" : "+v"(oa), "+v"(ob));
        __builtin_amdgcn_raw_buffer_store_b128(__builtin_bit_cast(u32x4, ob), ro, voff_out, so1, OUT_AUX);
        __builtin_amdgcn_raw_buffer_store_b128(__builtin_bit_cast(u32x4, oa), ro, voff_out, so2, OUT_AUX);
    }
    STAMP(63);
}

}

extern "C" void kernel_launch(int rank, int nranks, ncclComm_t comm,
                              hipStream_t stream,
                              void* const* d_in, const int* in_sizes, int n_in,
                              void* d_out, int out_size,
                              const int* in_dtypes, int out_dtype,
                              void* const* d_out_all,
                              void* const* d_ws_all, size_t ws_bytes) {
    (void)comm; (void)in_dtypes; (void)out_dtype; (void)d_out_all;
    if (nranks != NR || rank < 0 || rank >= NR || n_in < 4 || in_sizes[0] != SH * KD || in_sizes[1] != KD * NL ||
        out_size != NR * SH * NL || ws_bytes < WS_NEED) {
        fprintf(stderr, "ag_gemm: unexpected problem shape\n");
        return;
    }
    Params p;
    for (int i = 0; i < NR; ++i) p.ws[i] = (char*)d_ws_all[i];
    p.x = (const char*)d_in[0];
    p.w = (const char*)d_in[1];
    p.sx = (const float*)d_in[2];
    p.sw = (const float*)d_in[3];
    p.out = (float*)d_out;
    p.rank = rank;
    p.pad = 0;
    ag_gemm_kernel<<<dim3(NWG), dim3(256), 0, stream>>>(p);
    HIP_CHECK(hipGetLastError());
}
```

```cpp
#include <hip/hip_runtime.h>
#include <rccl/rccl.h>
#include <cstdio>

#define HIP_CHECK(expr) do { hipError_t _e = (expr); if (_e != hipSuccess) { \
    fprintf(stderr, "HIP error %s at %s:%d\n", hipGetErrorString(_e), __FILE__, __LINE__); \
    return; } } while (0)

namespace {

typedef int v8i __attribute__((ext_vector_type(8)));
typedef unsigned u32x4 __attribute__((ext_vector_type(4)));
typedef float v4f __attribute__((ext_vector_type(4)));
typedef short short2v __attribute__((ext_vector_type(2)));
#define LAS __attribute__((address_space(3)))
#define GAS __attribute__((address_space(1)))

constexpr int NR = 8;
constexpr int KD = 4096;
constexpr int NL = 2048;
constexpr int SH = 512;
constexpr int IROWS = 16;
constexpr int NI = SH / IROWS;
constexpr int IBYTES = IROWS * KD;
constexpr int SHBYTES = SH * KD;
constexpr int NP = 32;
constexpr int NWG = 256;
#define BC 8
#define A_AUX 17
#define OUT_BITS "sc1"
#define NO_STORE_T 2
#define ABL 0
#define X_VALU(x) do { if (!(ABL & 1)) { x; } } while (0)
#define X_LDS(x) do { if (!(ABL & 2)) { x; } } while (0)
#define X_STORE(x) do { if (!(ABL & 4)) { x; } } while (0)
#define X_SAMPLE(x) do { if (!(ABL & 8)) { x; } } while (0)
#define X_LOAD(x) do { if (!(ABL & 16)) { x; } } while (0)
#define X_BAR "s_barrier"
#define PACE 575
#define LEAD 575
#define PU 2
#define PWV 1
#define WP 2
constexpr int FW = (IBYTES / 1024) / PU;
constexpr int NU = NI * FW;
constexpr int NPW = NP * PWV;
constexpr int TICKETS = NU;

constexpr size_t WS_CNT = 0;
constexpr size_t WS_TMO = 32768;
constexpr size_t WS_TKT = 32768 + 256;
constexpr size_t WS_PROG = 40960;
constexpr size_t WS_FLAG = 65536;
constexpr size_t WS_DUMMY = 49152;
constexpr size_t WS_STAGE = 2u << 20;
constexpr size_t WS_NEED = WS_STAGE + 2ull * NR * SHBYTES;
constexpr size_t WS_DBG = 64u << 20;
constexpr unsigned long long DEADLINE_TICKS = 40000000ull;
static_assert(WS_FLAG + (size_t)NR * NU * 2 <= WS_STAGE && WS_PROG + 56 * 128 <= WS_DUMMY, "flags");
constexpr int LU = 4;
constexpr int NLU = SHBYTES / 1024 / LU;
static_assert(NLU <= (NWG - NP) * 4, "one local unit per consumer wave");
constexpr int PC = (WP + PU - 1) / PU;
constexpr int push_slot_ops(int j) { return (((j + 4 * PU - WP) % PU == PU - 1) ? 1 : 0) + ((j % PU == 0) ? 1 : 0) + 1 + (NR - 1); }
constexpr int push_allowed(int k) { int a = 0; for (int j = k - WP + 1; j < k; ++j) a += push_slot_ops(j + 4 * PU); return a; }

struct Params {
    char* ws[NR];
    const char* x;
    const char* w;
    const float* sx;
    const float* sw;
    float* out;
    int rank;
    int pad;
};

#define STAMP(i) do {} while (0)
#define STAMPF(i) do {} while (0)

__device__ __forceinline__ __amdgpu_buffer_rsrc_t mk_rsrc(const void* base, unsigned bytes) {
    return __builtin_amdgcn_make_buffer_rsrc((void*)base, 0, bytes, 0x00020000);
}

__device__ __forceinline__ float silu_dq(float acc, float s) {
    const float y = acc * s;
    const float c = __builtin_amdgcn_fmed3f(y, -60.0f, 60.0f);
    const float e = __builtin_amdgcn_exp2f(c * -1.44269504088896340736f);
    return y * __builtin_amdgcn_rcpf(1.0f + e);
}

constexpr int RED_BYTES = 4 * 3 * 1024;

__global__ __launch_bounds__(256, 1) void ag_gemm_kernel(const Params p) {
    __shared__ __attribute__((aligned(16))) char lds[2 * RED_BYTES];
    const int tid = threadIdx.x;
    const int lane = tid & 63;
    const int wid = __builtin_amdgcn_readfirstlane(tid >> 6);
    const int b = blockIdx.x;
    const unsigned rank = (unsigned)p.rank;
    char* const myws = p.ws[0];

    GAS unsigned* cnt = (GAS unsigned*)(myws + WS_CNT + (size_t)b * 128 + wid * 4);

    int rg = 0, cg = (b & 7) * 4 + (b >> 3);
    if (b >= NP) {
        const int i = (b & 7) * 28 + (b >> 3) - 4;
        rg = 1 + (i / BC) % 7;
        cg = (i / (7 * BC)) * BC + i % BC;
    }
    STAMP(0);
    if (rg == 0) {
        if (wid >= PWV) {
            const unsigned call = (unsigned)__builtin_amdgcn_readfirstlane(__hip_atomic_load(cnt, __ATOMIC_RELAXED, __HIP_MEMORY_SCOPE_AGENT)) + 1u;
            if (lane == 0) __hip_atomic_store(cnt, call, __ATOMIC_RELAXED, __HIP_MEMORY_SCOPE_AGENT);
            if (wid == 3 && (cg & 3) == 0 && cg < 4 * (NR - 1)) {
                const int pw = cg >> 2;
                const __amdgpu_buffer_rsrc_t rf = mk_rsrc(myws + WS_FLAG + (size_t)((rank + 1u + (unsigned)pw) & (NR - 1)) * (NU * 2), (unsigned)NU * 2u);
                GAS unsigned* const prog = (GAS unsigned*)(myws + WS_PROG + (size_t)pw * 1024 + (lane & 7) * 128);
                GAS unsigned* const tmo = (GAS unsigned*)(myws + WS_TMO);
                const unsigned long long t0 = __builtin_amdgcn_s_memrealtime();
                for (int rdy = 0, spins = 0; rdy < NI; ) {
                    const unsigned f = __builtin_amdgcn_raw_buffer_load_b32(rf, lane * 4, rdy * (FW * 2), 17);
                    const unsigned long long m = __ballot((short)(f - call) >= 0 && (short)((f >> 16) - call) >= 0);
                    const int k = (m == ~0ull ? 64 : __builtin_ctzll(~m)) / (FW / 2);
                    if (k > 0) {
                        rdy = min(rdy + k, NI);
                        if (lane < 8) __hip_atomic_store(prog, call * 64u + (unsigned)rdy, __ATOMIC_RELAXED, __HIP_MEMORY_SCOPE_AGENT);
                        for (int z = rdy - k + 1; z <= rdy; ++z) STAMPF(2 + z);
                    } else if ((++spins & 15) == 0 &&
                               ((unsigned)__builtin_amdgcn_readfirstlane(__hip_atomic_load(tmo, __ATOMIC_RELAXED, __HIP_MEMORY_SCOPE_AGENT)) == call ||
                                __builtin_amdgcn_s_memrealtime() - t0 > DEADLINE_TICKS)) {
                        if (lane == 0) __hip_atomic_store(tmo, call, __ATOMIC_RELAXED, __HIP_MEMORY_SCOPE_AGENT);
                        break;
                    }
                }
                STAMP(61);
            }
            return;
        }
        const int n = lane & 15, q = lane >> 4;
        const int v = wid * NP + cg;
        GAS unsigned* const tkt = (GAS unsigned*)(myws + WS_TKT);
        u32x4 d[4] = {};
        unsigned tk[4] = {0u, 0u, 0u, 0u};
        int id[2 * PC + 1];
#pragma unroll
        for (int i = 0; i <= 2 * PC; ++i) id[i] = i < PC + 1 ? -1 : v + (i - PC - 1) * NPW;
#define PUSH_LOAD(D, U, PP) do { const int pc_ = ((U) < NU ? (U) : NU - 1) * PU + (PP); \
        const char* s_ = p.x + (size_t)((pc_ >> 6) * IROWS + ((n & 12) | ((n & 3) ^ ((pc_ >> 4) & 3)))) * KD + (pc_ & 63) * 64 + q * 16; \
        asm volatile("global_load_dwordx4 %0, %1, off" : "=&v"(D) : "v"(s_) : "memory"); } while (0)
#pragma unroll
        for (int k = 0; k < WP; ++k) PUSH_LOAD(d[k], v + (k / PU) * NPW, k % PU);
        const unsigned cv = __hip_atomic_load(cnt, __ATOMIC_RELAXED, __HIP_MEMORY_SCOPE_AGENT);
        asm volatile("s_waitcnt vmcnt(0)" : "+v"(d[0]), "+v"(d[1]), "+v"(d[2]) :: "memory");
        const unsigned call = (unsigned)__builtin_amdgcn_readfirstlane(cv) + 1u;
        const unsigned tbase = (call - 1u) * (unsigned)TICKETS - 2u * PC * NPW;
        const size_t soff = WS_STAGE + (size_t)(call & 1u) * (NR * SHBYTES) + (size_t)rank * SHBYTES + lane * 16;
        char* fws = p.ws[1];
#pragma unroll
        for (int i = 2; i < NR; ++i) fws = lane == i - 1 ? p.ws[i] : fws;
        GAS unsigned short* const flag = (GAS unsigned short*)(fws + WS_FLAG) + rank * NU;
        GAS unsigned short* const dummy = (GAS unsigned short*)(myws + WS_DUMMY + cg * 128) + wid * 8 + (lane & 7);
        const unsigned one = 1u;
        const unsigned tp0 = (unsigned)__builtin_amdgcn_s_memrealtime();
        STAMP(1);
#define PUSH_FLAG(U) do { GAS unsigned short* f_ = (U) >= 0 ? flag + (U) : dummy; \
        if (lane < NR - 1) asm volatile("global_store_short %0, %1, off sc0 sc1" :: "v"(f_), "v"(call) : "memory"); } while (0)
        for (int nu = 0;; ) {
#pragma unroll
            for (int k = 0; k < 4; ++k) {
                asm volatile("s_waitcnt vmcnt(%5)" : "+v"(d[k]), "+v"(tk[0]), "+v"(tk[1]), "+v"(tk[2]), "+v"(tk[3]) : "i"(push_allowed(k)) : "memory");
                if (k % PU == 0) {
#pragma unroll
                    for (int i = 0; i < 2 * PC; ++i) id[i] = id[i + 1];
                    const int tv = (int)((unsigned)__builtin_amdgcn_readfirstlane(tk[((k / PU) + 4 - PC) % (4 / PU)]) - tbase);
                    id[2 * PC] = nu < PC ? v + (nu + PC) * NPW : tv;
                    if (id[PC] >= NU) goto push_done;
                    ++nu;
                }
                if ((k + 4 * PU - WP) % PU == PU - 1) PUSH_FLAG(id[PC - (WP - (k % PU) + PU - 1) / PU]);
                if (k % PU == 0) { if (lane == 0) asm volatile("global_atomic_add %0, %1, %2, off sc0" : "=&v"(tk[(k / PU) % (4 / PU)]) : "v"(tkt), "v"(one) : "memory"); }
                PUSH_LOAD(d[(k + WP) & 3], id[PC + ((k % PU) + WP) / PU], ((k % PU) + WP) % PU);
                if (PACE > 0) {
                    const int due = (((id[PC] / NPW) * PU + (k % PU)) * (PACE * PWV) - LEAD) / 10;
                    while ((int)((unsigned)__builtin_amdgcn_s_memrealtime() - tp0) < due) __builtin_amdgcn_s_sleep(1);
                }
                const size_t po = soff + (size_t)(id[PC] * PU + (k % PU)) * 1024;
#pragma unroll
                for (int pr = 0; pr < NR - 1; ++pr)
                    asm volatile("global_store_dwordx4 %0, %1, off sc0 sc1\n\ts_nop 1" :: "v"(p.ws[1 + pr] + po), "v"(d[k]) : "memory");
            }
        }
push_done:
        asm volatile("s_waitcnt vmcnt(0)" : "+v"(d[0]), "+v"(d[1]), "+v"(d[2]), "+v"(d[3]), "+v"(tk[0]), "+v"(tk[1]), "+v"(tk[2]), "+v"(tk[3]) :: "memory");
#pragma unroll
        for (int i = 0; i < PC; ++i) PUSH_FLAG(id[i]);
        if (lane == 0) __hip_atomic_store(cnt, call, __ATOMIC_RELAXED, __HIP_MEMORY_SCOPE_AGENT);
        STAMP(63);
        return;
    }

    const unsigned cv = __hip_atomic_load(cnt, __ATOMIC_RELAXED, __HIP_MEMORY_SCOPE_AGENT);
    const int pi = rg - 1;
    const unsigned srank = (rank + 1u + (unsigned)pi) & (NR - 1);
    const int nloc = pi < 5 ? 4 : 6;
    const int l0 = pi < 5 ? 4 * pi : 6 * pi - 10;
    const int q = lane >> 4, n = lane & 15;
#define SB0 __builtin_amdgcn_sched_barrier(0)

    const int lu = (pi * 32 + cg) * 4 + wid;
    u32x4 lx[LU];
    {
        const __amdgpu_buffer_rsrc_t rx = mk_rsrc(p.x, lu < NLU ? (unsigned)SHBYTES : 0u);
        const int pc = lu * LU;
        const int vo = ((n & 12) | ((n & 3) ^ ((pc >> 4) & 3))) * KD + q * 16;
#pragma unroll
        for (int i = 0; i < LU; ++i) lx[i] = __builtin_amdgcn_raw_buffer_load_b128(rx, vo, (pc >> 6) * IBYTES + ((pc & 63) + i) * 64, 0);
    }
    v8i Wf[8][4];
    unsigned call;
    const char* stage;
    {
        const __amdgpu_buffer_rsrc_t rw = mk_rsrc(p.w, (unsigned)KD * NL);
        const int voff = (q * 16) * NL + n * 4;
        const int sbase = (wid * 1024) * NL + cg * 64;
        unsigned d[2][8][4];
#define W_LOAD(KS) do { _Pragma("unroll") for (int r = 0; r < 8; ++r) _Pragma("unroll") for (int c = 0; c < 4; ++c) \
            d[(KS) & 1][r][c] = __builtin_amdgcn_raw_buffer_load_b32(rw, voff, sbase + ((KS) * 128 + (r >> 2) * 64 + (r & 3) * 4 + c) * NL, 0); } while (0)
        W_LOAD(0);
        SB0;
        call = (unsigned)__builtin_amdgcn_readfirstlane(cv) + 1u;
        if (lane == 0) __hip_atomic_store(cnt, call, __ATOMIC_RELAXED, __HIP_MEMORY_SCOPE_AGENT);
        stage = myws + WS_STAGE + (size_t)(call & 1u) * (NR * SHBYTES);
        {
            const __amdgpu_buffer_rsrc_t rl = mk_rsrc(stage + (size_t)rank * SHBYTES, lu < NLU ? (unsigned)SHBYTES : 0u);
#pragma unroll
            for (int i = 0; i < LU; ++i) __builtin_amdgcn_raw_buffer_store_b128(lx[i], rl, lane * 16, (lu * LU + i) * 1024, 17);
        }
        SB0;
#pragma unroll
        for (int ks = 0; ks < 8; ++ks) {
            if (ks + 1 < 8) W_LOAD(ks + 1);
            SB0;
#pragma unroll
            for (int r = 0; r < 8; ++r) {
                const unsigned* e = d[ks & 1][r];
                const unsigned t0 = __builtin_amdgcn_perm(e[1], e[0], 0x05010400u), t1 = __builtin_amdgcn_perm(e[1], e[0], 0x07030602u);
                const unsigned t2 = __builtin_amdgcn_perm(e[3], e[2], 0x05010400u), t3 = __builtin_amdgcn_perm(e[3], e[2], 0x07030602u);
                Wf[ks][0][r] = (int)__builtin_amdgcn_perm(t2, t0, 0x05040100u);
                Wf[ks][1][r] = (int)__builtin_amdgcn_perm(t2, t0, 0x07060302u);
                Wf[ks][2][r] = (int)__builtin_amdgcn_perm(t3, t1, 0x05040100u);
                Wf[ks][3][r] = (int)__builtin_amdgcn_perm(t3, t1, 0x07060302u);
            }
#pragma unroll
            for (int j = 0; j < 4; ++j) asm volatile("" : "+a"(Wf[ks][j]));
            SB0;
            if (ks == 1) {
                asm volatile("s_waitcnt vmcnt(32)" ::: "memory");
                if (lane < LU / PU && lu < NLU) __hip_atomic_store((GAS unsigned short*)(myws + WS_FLAG) + rank * NU + lu * (LU / PU) + lane, (unsigned short)call, __ATOMIC_RELAXED, __HIP_MEMORY_SCOPE_SYSTEM);
                SB0;
            }
        }
    }
    const float scale = p.sx[0] * p.sw[0];
    STAMP(1);

    GAS unsigned* const tmo = (GAS unsigned*)(myws + WS_TMO);
    const __amdgpu_buffer_rsrc_t rs = mk_rsrc(stage, (unsigned)NR * SHBYTES);
    const __amdgpu_buffer_rsrc_t rp = mk_rsrc(myws + WS_PROG + (size_t)pi * 1024 + (b & 7) * 128, 4u);
    const int voff_a = lane * 16;
    const int voff_out = ((q * 4 + wid) * NL + n * 4) * 4;
    bool dead = false;
#define SPIN(READY_EXPR, LOADV) do { \
        const unsigned long long t0_ = __builtin_amdgcn_s_memrealtime(); \
        for (int spins_ = 1;; ++spins_) { \
            const unsigned v_ = LOADV; \
            if (READY_EXPR) break; \
            if ((spins_ & 15) == 0 && ((unsigned)__builtin_amdgcn_readfirstlane(__hip_atomic_load(tmo, __ATOMIC_RELAXED, __HIP_MEMORY_SCOPE_AGENT)) == call || \
                __builtin_amdgcn_s_memrealtime() - t0_ > DEADLINE_TICKS)) { \
                if (lane == 0) __hip_atomic_store(tmo, call, __ATOMIC_RELAXED, __HIP_MEMORY_SCOPE_AGENT); \
                dead = true; \
                rdy = NI; \
                break; \
            } \
            __builtin_amdgcn_s_sleep(1); \
        } } while (0)
#define SAMPLE() __builtin_amdgcn_raw_buffer_load_b32(rp, 0, 0, 16)
#define ADVANCE(V) do { rdy = max(rdy, (int)((unsigned)__builtin_amdgcn_readfirstlane(V) - call * 64u)); } while (0)
    int rdy = 0;
    {
        const __amdgpu_buffer_rsrc_t rfl = mk_rsrc(myws + WS_FLAG + (size_t)rank * (NU * 2), (unsigned)NU * 2u);
        for (int i = 0; i < nloc * FW; i += 128)
            SPIN(__all(i + 2 * lane >= nloc * FW || ((short)(v_ - call) >= 0 && (short)((v_ >> 16) - call) >= 0)),
                 __builtin_amdgcn_raw_buffer_load_b32(rfl, lane * 4, (l0 * FW + i) * 2, 17));
    }

#define AREG(B, KS) (128 + 64 * (B) + 8 * (KS))
#define OWN(B) (90 + 4 * (B))
#define LOAD_FRAG(B, KS, RS, SB) asm volatile( \
        "buffer_load_dwordx4 v[%c3:%c4], %0, %1, %2 offen offset:%c7 sc0 sc1\n\tbuffer_load_dwordx4 v[%c5:%c6], %0, %1, %2 offen offset:%c8 sc0 sc1" \
        :: "v"(voff_a), "s"(RS), "s"((SB) + ((KS) >> 1) * 4096), "i"(AREG(B, KS)), "i"(AREG(B, KS) + 3), "i"(AREG(B, KS) + 4), "i"(AREG(B, KS) + 7), \
           "i"(((KS) & 1) * 2048), "i"(((KS) & 1) * 2048 + 1024) : "memory")
#define BURST(B, RS, SB) do { _Pragma("unroll") for (int ks_ = 0; ks_ < 8; ++ks_) LOAD_FRAG(B, ks_, RS, SB); } while (0)
#define WAITV(N) asm volatile("s_waitcnt vmcnt(%c0)" :: "i"(N) : "memory")
#define SAMPLE_ISSUE(B) asm volatile("buffer_load_dword v%c1, off, %0, 0 sc1" :: "s"(rp), "i"(126 + (B)) : "memory")
#define SAMPLE_TAKE(B) do { unsigned sv_; asm volatile("v_readfirstlane_b32 %0, v%c1" : "=s"(sv_) : "i"(126 + (B)) : "memory"); \
        rdy = max(rdy, (int)(sv_ - call * 64u)); } while (0)
#define LOOP_REGS "v74", "v75", "v76", "v77", "v78", "v79", "v80", "v81", "v82", "v83", "v84", "v85", "v86", "v87", "v88", "v89", "v90", "v91", "v92", "v93", "v94", "v95", "v96", "v97", "v98", "v99", "v100", "v101", "v102", "v103", "v104", "v105", "v106", "v107", "v108", "v109", "v110", "v111", "v112", "v113", "v114", "v115", "v116", "v117", "v118", "v119", "v120", "v121", "v122", "v123", "v124", "v125", "v126", "v127", "v128", "v129", "v130", "v131", "v132", "v133", "v134", "v135", "v136", "v137", "v138", "v139", "v140", "v141", "v142", "v143", "v144", "v145", "v146", "v147", "v148", "v149", "v150", "v151", "v152", "v153", "v154", "v155", "v156", "v157", "v158", "v159", "v160", "v161", "v162", "v163", "v164", "v165", "v166", "v167", "v168", "v169", "v170", "v171", "v172", "v173", "v174", "v175", "v176", "v177", "v178", "v179", "v180", "v181", "v182", "v183", "v184", "v185", "v186", "v187", "v188", "v189", "v190", "v191", "v192", "v193", "v194", "v195", "v196", "v197", "v198", "v199", "v200", "v201", "v202", "v203", "v204", "v205", "v206", "v207", "v208", "v209", "v210", "v211", "v212", "v213", "v214", "v215", "v216", "v217", "v218", "v219", "v220", "v221", "v222", "v223", "v224", "v225", "v226", "v227", "v228", "v229", "v230", "v231", "v232", "v233", "v234", "v235", "v236", "v237", "v238", "v239", "v240", "v241", "v242", "v243", "v244", "v245", "v246", "v247", "v248", "v249", "v250", "v251", "v252", "v253", "v254", "v255"
#define MFMA0(B, J) asm volatile("v_mfma_f32_16x16x128_f8f6f4 v[%c1:%c2], v[%c3:%c4], %0, v[74:77]" \
        :: "a"(Wf[0][J]), "i"(110 + 4 * (J)), "i"(113 + 4 * (J)), "i"(AREG(B, 0)), "i"(AREG(B, 0) + 7))
#define SET_CINIT() asm volatile("v_mov_b32 v74, %0\n\tv_mov_b32 v75, %0\n\tv_mov_b32 v76, %0\n\tv_mov_b32 v77, %0" :: "s"(dead ? 0x7fc00000u : 0u))
#define MFMA(B, KS, J) asm volatile("v_mfma_f32_16x16x128_f8f6f4 v[%c1:%c2], v[%c3:%c4], %0, v[%c1:%c2]" \
        :: "a"(Wf[KS][J]), "i"(110 + 4 * (J)), "i"(113 + 4 * (J)), "i"(AREG(B, KS)), "i"(AREG(B, KS) + 7))
#define TAKE(B, J) asm volatile("v_mov_b32 v%c0, v%c4\n\tv_mov_b32 v%c1, v%c5\n\tv_mov_b32 v%c2, v%c6\n\tv_mov_b32 v%c3, v%c7" \
        :: "i"(OWN(B) + (J)), "i"(98 + (J)), "i"(102 + (J)), "i"(106 + (J)), "i"(110 + 4 * (J)), "i"(111 + 4 * (J)), "i"(112 + 4 * (J)), "i"(113 + 4 * (J)))
#define PUT(B, R) asm volatile("ds_write_b128 %0, v[%c1:%c2] offset:%c3" :: "v"(put##R), "i"(94 + 4 * (R)), "i"(97 + 4 * (R)), "i"((B) * RED_BYTES) : "memory")
#define GET3(B) asm volatile("ds_read_b128 v[78:81], %0 offset:%c1\n\tds_read_b128 v[82:85], %0 offset:%c2\n\tds_read_b128 v[86:89], %0 offset:%c3" \
        :: "v"(get), "i"((B) * RED_BYTES), "i"((B) * RED_BYTES + 1024), "i"((B) * RED_BYTES + 2048) : "memory")
#define VOP4(OP, D, A, C) asm volatile(OP " v%c0, v%c4, v%c8\n\t" OP " v%c1, v%c5, v%c9\n\t" OP " v%c2, v%c6, v%c10\n\t" OP " v%c3, v%c7, v%c11" \
        :: "i"(D), "i"((D) + 1), "i"((D) + 2), "i"((D) + 3), "i"(A), "i"((A) + 1), "i"((A) + 2), "i"((A) + 3), "i"(C), "i"((C) + 1), "i"((C) + 2), "i"((C) + 3))
#define VOPK4(OP, D, K) asm volatile(OP " v%c1, %0, v%c1\n\t" OP " v%c2, %0, v%c2\n\t" OP " v%c3, %0, v%c3\n\t" OP " v%c4, %0, v%c4" \
        :: "s"(K), "i"(D), "i"((D) + 1), "i"((D) + 2), "i"((D) + 3))
#define VUN2(OP, D) asm volatile(OP " v%c0, v%c0\n\t" OP " v%c1, v%c1" :: "i"(D), "i"((D) + 1))
#define CLAMP4(D, A) asm volatile("v_med3_f32 v%c2, v%c6, %0, %1\n\tv_med3_f32 v%c3, v%c7, %0, %1\n\tv_med3_f32 v%c4, v%c8, %0, %1\n\tv_med3_f32 v%c5, v%c9, %0, %1" \
        :: "s"(-60.0f), "v"(c60), "i"(D), "i"((D) + 1), "i"((D) + 2), "i"((D) + 3), "i"(A), "i"((A) + 1), "i"((A) + 2), "i"((A) + 3))
#define STORE4(A, RS, SO) asm volatile("buffer_store_dwordx4 v[%c3:%c4], %0, %1, %2 offen " OUT_BITS :: "v"(voff_out), "s"(RS), "s"(SO), "i"(A), "i"((A) + 3) : "memory")
#define PIN(X) asm volatile("" : "+v"(X))
#define NEXT_SA(TN, SA) ((TN) == 0 ? (int)srank * SHBYTES + wid * 16384 : (SA) + IBYTES)
    const unsigned put1 = (unsigned)(size_t)((LAS char*)lds + ((1 ^ wid) * 3 + 0) * 1024 + lane * 16);
    const unsigned put2 = (unsigned)(size_t)((LAS char*)lds + ((2 ^ wid) * 3 + 1) * 1024 + lane * 16);
    const unsigned put3 = (unsigned)(size_t)((LAS char*)lds + ((3 ^ wid) * 3 + 2) * 1024 + lane * 16);
    const unsigned get = (unsigned)(size_t)((LAS char*)lds + wid * 3072 + lane * 16);
    const float scale_s = __builtin_bit_cast(float, __builtin_amdgcn_readfirstlane(__builtin_bit_cast(int, scale)));
    float c60 = 60.0f;
    PIN(c60);
    constexpr float NLOG2E = -1.44269504088896340736f;
    int sa = (int)rank * SHBYTES + l0 * IBYTES + wid * 16384;
    int sa1 = sa + IBYTES;
    int so = (((int)rank * SH + l0 * IROWS) * NL + cg * 64) * 4;
    int so1 = 0, so2 = 0;
    bool have0 = false;
    int pf = 0;
    unsigned orec = 0;
    const char* outp = (const char*)p.out;
    asm volatile("" : "+s"(outp));
#define DBG_TOP do {} while (0)
#define DBG_END do {} while (0)
#define DBG_TOP1 do {} while (0)
#define DBG_SEEN do {} while (0)
#define EDGE(K) do { if (PF) X_LOAD(LOAD_FRAG(0, K, rs, sa1)); if ((K) < 7) WAITV((PF ? 14 : 12 - 2 * (K)) + ((K) >= 5) + ((K) >= 6)); } while (0)
#define ITERATION(B) do { \
        if (__builtin_expect(pf == 0, 0)) { \
            DBG_TOP1; \
              \
            if (have0) { WAITV(7); SAMPLE_TAKE(0); } \
            else { \
                WAITV(1); SAMPLE_TAKE(0); \
                DBG_TOP; \
                if (tn > rdy) { SPIN(({ ADVANCE(v_); rdy >= tn; }), SAMPLE()); SET_CINIT(); __builtin_amdgcn_s_waitcnt(0xC07F); } \
                DBG_SEEN; \
                BURST(0, rs, sa); \
            } \
            pf = tn < rdy ? 77 : 0; \
        } \
        if (pf) BODY(B, true); else BODY(B, false); \
    } while (0)
#define BODY(B, PF_) do { \
        constexpr bool PF = PF_; \
        constexpr int S = OWN(1 - (B)); \
        const __amdgpu_buffer_rsrc_t ro = mk_rsrc(outp, orec); \
        X_LDS(GET3(B)); X_VALU(TAKE(B, 0)); WAITV(14); \
        MFMA0(0, 0); X_VALU(TAKE(B, 1)); MFMA0(0, 1); X_VALU(TAKE(B, 2)); MFMA0(0, 2); X_VALU(TAKE(B, 3)); MFMA0(0, 3); \
        EDGE(0); \
        MFMA(0, 1, 0); X_LDS(PUT(1 - (B), 1)); MFMA(0, 1, 1); X_LDS(PUT(1 - (B), 2)); MFMA(0, 1, 2); X_LDS(PUT(1 - (B), 3)); MFMA(0, 1, 3); \
        EDGE(1); \
        MFMA(0, 2, 0); asm volatile("s_waitcnt lgkmcnt(3)" ::: "memory"); X_VALU(VOP4("v_add_f32", S, S, 78)); MFMA(0, 2, 1); \
        X_VALU(VOP4("v_add_f32", 82, 82, 86)); MFMA(0, 2, 2); X_VALU(VOP4("v_add_f32", S, S, 82)); MFMA(0, 2, 3); \
        EDGE(2); \
        MFMA(0, 3, 0); X_VALU(VOPK4("v_mul_f32", S, scale_s)); MFMA(0, 3, 1); X_VALU(CLAMP4(78, S)); MFMA(0, 3, 2); X_VALU(VOPK4("v_mul_f32", 78, NLOG2E)); MFMA(0, 3, 3); \
        EDGE(3); \
        MFMA(0, 4, 0); X_VALU(VUN2("v_exp_f32", 78)); MFMA(0, 4, 1); X_VALU(VUN2("v_exp_f32", 80)); MFMA(0, 4, 2); X_VALU(VOPK4("v_add_f32", 78, 1.0f)); MFMA(0, 4, 3); \
        EDGE(4); \
          \
        MFMA(0, 5, 0); SAMPLE_TAKE(0); X_SAMPLE(SAMPLE_ISSUE(0)); asm volatile("" : "+s"(rdy)); SB0; MFMA(0, 5, 1); \
        X_VALU(VUN2("v_rcp_f32", 78)); MFMA(0, 5, 2); X_VALU(VUN2("v_rcp_f32", 80)); MFMA(0, 5, 3); \
        EDGE(5); \
        MFMA(0, 6, 0); X_VALU(VOP4("v_mul_f32", S, S, 78)); MFMA(0, 6, 1); X_STORE(STORE4(S, ro, so2)); SB0; MFMA(0, 6, 2); SB0; \
        so2 = so1; so1 = so; so = tn == 0 ? (((int)srank * SH) * NL + cg * 64) * 4 : so + IROWS * NL * 4; \
        orec = tn + nloc >= NO_STORE_T ? (unsigned)NR * SH * NL * 4u : 0u; \
        asm volatile("" : "+s"(so), "+s"(orec)); SB0; MFMA(0, 6, 3); \
        EDGE(6); \
        MFMA(0, 7, 0); SB0; \
        have0 = PF; \
        int tq = tn + 1; asm volatile("" : "+s"(tq)); \
        int pn = have0 && tq < rdy ? 77 : 0; int sax = NEXT_SA(tq, sa1); \
        asm volatile("" : "+s"(pn), "+s"(sax)); SB0; MFMA(0, 7, 1); MFMA(0, 7, 2); MFMA(0, 7, 3); \
        EDGE(7); SB0; \
        DBG_END; \
        sa = sa1; sa1 = sax; pf = pn; \
        if (tn >= 21 && wid == 0) STAMPF(2 + tn); \
        asm volatile("s_waitcnt lgkmcnt(0)\n\t" X_BAR ::: "memory", LOOP_REGS); \
    } while (0)
    asm volatile("v_mov_b32 v126, 0\n\tv_mov_b32 v127, 0" ::: "memory", LOOP_REGS);
    SET_CINIT();
    for (int tn = 1 - nloc; tn <= NI; ++tn) {
        ITERATION(0);
        ++tn;
        ITERATION(1);
    }
    {
        const __amdgpu_buffer_rsrc_t ro = mk_rsrc(outp, (unsigned)NR * SH * NL * 4u);
        GET3(0);
        asm volatile("s_nop 7" ::: "memory");
        TAKE(0, 0); TAKE(0, 1); TAKE(0, 2); TAKE(0, 3);
        PUT(1, 1); PUT(1, 2); PUT(1, 3);
        asm volatile("s_waitcnt lgkmcnt(3)" ::: "memory");
        VOP4("v_add_f32", OWN(1), OWN(1), 78); VOP4("v_add_f32", 82, 82, 86); VOP4("v_add_f32", OWN(1), OWN(1), 82);
        asm volatile("s_waitcnt lgkmcnt(0)\n\ts_barrier" ::: "memory", LOOP_REGS);
        GET3(1);
        VOPK4("v_mul_f32", OWN(1), scale_s); CLAMP4(98, OWN(1)); VOPK4("v_mul_f32", 98, NLOG2E);
        VUN2("v_exp_f32", 98); VUN2("v_exp_f32", 100);
        asm volatile("s_waitcnt lgkmcnt(0)" ::: "memory");
        VOP4("v_add_f32", OWN(0), OWN(0), 78); VOP4("v_add_f32", 82, 82, 86); VOP4("v_add_f32", OWN(0), OWN(0), 82);
        VOPK4("v_mul_f32", OWN(0), scale_s); CLAMP4(78, OWN(0)); VOPK4("v_mul_f32", 78, NLOG2E);
        VUN2("v_exp_f32", 78); VUN2("v_exp_f32", 80); VOPK4("v_add_f32", 98, 1.0f); VOPK4("v_add_f32", 78, 1.0f);
        VUN2("v_rcp_f32", 78); VUN2("v_rcp_f32", 80); VUN2("v_rcp_f32", 98); VUN2("v_rcp_f32", 100);
        VOP4("v_mul_f32", OWN(0), OWN(0), 78); VOP4("v_mul_f32", OWN(1), OWN(1), 98);
        STORE4(OWN(0), ro, so1); STORE4(OWN(1), ro, so2);
    }
    STAMP(63);
}

}

extern "C" void kernel_launch(int rank, int nranks, ncclComm_t comm,
                              hipStream_t stream,
                              void* const* d_in, const int* in_sizes, int n_in,
                              void* d_out, int out_size,
                              const int* in_dtypes, int out_dtype,
                              void* const* d_out_all,
                              void* const* d_ws_all, size_t ws_bytes) {
    (void)comm; (void)in_dtypes; (void)out_dtype; (void)d_out_all;
    if (nranks != NR || rank < 0 || rank >= NR || n_in < 4 || in_sizes[0] != SH * KD || in_sizes[1] != KD * NL ||
        out_size != NR * SH * NL || ws_bytes < WS_NEED) {
        fprintf(stderr, "ag_gemm: unexpected problem shape\n");
        return;
    }
    Params p;
    for (int i = 0; i < NR; ++i) p.ws[i] = (char*)d_ws_all[(rank + i) % NR];
    p.x = (const char*)d_in[0];
    p.w = (const char*)d_in[1];
    p.sx = (const float*)d_in[2];
    p.sw = (const float*)d_in[3];
    p.out = (float*)d_out;
    p.rank = rank;
    p.pad = 0;
    ag_gemm_kernel<<<dim3(NWG), dim3(256), 0, stream>>>(p);
    HIP_CHECK(hipGetLastError());
}
```

```cpp
#include <hip/hip_runtime.h>
#include <rccl/rccl.h>
#include <cstdio>

#define HIP_CHECK(expr) do { hipError_t _e = (expr); if (_e != hipSuccess) { \
    fprintf(stderr, "HIP error %s at %s:%d\n", hipGetErrorString(_e), __FILE__, __LINE__); \
    return; } } while (0)

namespace {

typedef int v8i __attribute__((ext_vector_type(8)));
typedef unsigned u32x4 __attribute__((ext_vector_type(4)));
typedef float v4f __attribute__((ext_vector_type(4)));
typedef short short2v __attribute__((ext_vector_type(2)));
#define LAS __attribute__((address_space(3)))
#define GAS __attribute__((address_space(1)))

constexpr int NR = 8;
constexpr int KD = 4096;
constexpr int NL = 2048;
constexpr int SH = 512;
constexpr int IROWS = 16;
constexpr int NI = SH / IROWS;
constexpr int IBYTES = IROWS * KD;
constexpr int SHBYTES = SH * KD;
constexpr int NP = 32;
#define NWATCH 2
constexpr int NWG = 256;
#define BC 4
#define A_AUX 17
#define PUSH_BITS "sc0 sc1"
#define OUT_BITS "sc1"
#define NO_STORE_T 2
#define W_TOUCH 1
#define UPG 3
#define ABL 0
#define X_VALU(x) do { if (!(ABL & 1)) { x; } } while (0)
#define X_LDS(x) do { if (!(ABL & 2)) { x; } } while (0)
#define X_STORE(x) do { if (!(ABL & 4)) { x; } } while (0)
#define X_SAMPLE(x) do { if (!(ABL & 8)) { x; } } while (0)
#define X_LOAD(x) do { if (!(ABL & 16)) { x; } } while (0)
#define X_BAR "s_barrier"
#define PACE 565
#define LEAD 2800
#define PU 4
#define PWV 1
#define WP 2
constexpr int FW = (IBYTES / 1024) / PU;
constexpr int NU = NI * FW;
constexpr int NPU = NP - NWATCH;
constexpr int NPW = NPU * PWV;
constexpr int TICKETS = NU;

constexpr size_t WS_CNT = 0;
constexpr size_t WS_TMO = 32768;
constexpr size_t WS_TKT = 32768 + 256;
constexpr size_t WS_PROG = 131072;
constexpr size_t WS_FLAG = 65536;
constexpr size_t WS_LFLAG = 262144;
constexpr size_t WS_DUMMY = 49152;
constexpr size_t WS_STAGE = 2u << 20;
constexpr size_t WS_NEED = WS_STAGE + 2ull * NR * SHBYTES;
constexpr size_t WS_DBG = 64u << 20;
constexpr unsigned long long DEADLINE_TICKS = 40000000ull;
static_assert(WS_FLAG + (size_t)NR * NU * 2 <= WS_PROG && WS_PROG + 224 * 128 <= WS_LFLAG && WS_LFLAG + 512 * 128 <= WS_STAGE, "flags");
constexpr int LU = 4;
constexpr int NLU = SHBYTES / 1024 / LU;
static_assert(NLU <= (NWG - NP) * 4, "one local unit per consumer wave");
constexpr int PC = (WP + PU - 1) / PU;
constexpr int push_slot_ops(int j) { return (((j + 4 * PU - WP) % PU == PU - 1) ? 1 : 0) + ((j % PU == 0) ? 1 : 0) + 1 + (NR - 1); }
constexpr int push_allowed(int k) { int a = 0; for (int j = k - WP + 1; j < k; ++j) a += push_slot_ops(j + 4 * PU); return a; }

struct Params {
    char* ws[NR];
    const char* x;
    const char* w;
    const float* sx;
    const float* sw;
    float* out;
    int rank;
    int pad;
};

#define STAMP(i) do {} while (0)
#define STAMPF(i) do {} while (0)

__device__ __forceinline__ __amdgpu_buffer_rsrc_t mk_rsrc(const void* base, unsigned bytes) {
    return __builtin_amdgcn_make_buffer_rsrc((void*)base, 0, bytes, 0x00020000);
}

__device__ __forceinline__ float silu_dq(float acc, float s) {
    const float y = acc * s;
    const float c = __builtin_amdgcn_fmed3f(y, -60.0f, 60.0f);
    const float e = __builtin_amdgcn_exp2f(c * -1.44269504088896340736f);
    return y * __builtin_amdgcn_rcpf(1.0f + e);
}

constexpr int WRING_BYTES = 4 * 4 * 8192;
constexpr int RED_BYTES = 4 * 4096;

__global__ __launch_bounds__(256, 1) void ag_gemm_kernel(const Params p) {
    __shared__ __attribute__((aligned(16))) char lds[WRING_BYTES + 2 * RED_BYTES];
    const int tid = threadIdx.x;
    const int lane = tid & 63;
    const int wid = __builtin_amdgcn_readfirstlane(tid >> 6);
    const int b = blockIdx.x;
    const unsigned rank = (unsigned)p.rank;
    char* const myws = p.ws[0];

    GAS unsigned* cnt = (GAS unsigned*)(myws + WS_CNT + (size_t)b * 128 + wid * 4);

    int rg = 0, cg = b;
    if (b >= NP) {
        const int i = (b & 7) * 28 + (b >> 3) - 4;
        rg = 1 + (i / BC) % 7;
        cg = (i / (7 * BC)) * BC + i % BC;
    }
    STAMP(0);
    if (rg == 0) {
        if (wid >= PWV || cg >= NPU) {
            const unsigned call = (unsigned)__builtin_amdgcn_readfirstlane(__hip_atomic_load(cnt, __ATOMIC_RELAXED, __HIP_MEMORY_SCOPE_AGENT)) + 1u;
            if (lane == 0) __hip_atomic_store(cnt, call, __ATOMIC_RELAXED, __HIP_MEMORY_SCOPE_AGENT);
            const int pw = NWATCH ? (cg - NPU) * 4 + wid : (wid == 3 && (cg & 3) == 0 ? cg >> 2 : -1);
            if (pw >= 0 && pw < NR - 1) {
                const __amdgpu_buffer_rsrc_t rf = mk_rsrc(myws + WS_FLAG + (size_t)((rank + 1u + (unsigned)pw) & (NR - 1)) * (NU * 2), (unsigned)NU * 2u);
                GAS unsigned* const prog = (GAS unsigned*)(myws + WS_PROG + (size_t)pw * 4096 + (lane & 31) * 128);
                GAS unsigned* const tmo = (GAS unsigned*)(myws + WS_TMO);
                const unsigned long long t0 = __builtin_amdgcn_s_memrealtime();
                for (int rdy = 0, spins = 0; rdy < NI; ) {
                    const unsigned f = __builtin_amdgcn_raw_buffer_load_b32(rf, lane * 4, rdy * (FW * 2), 17);
                    const unsigned long long m = __ballot((short)(f - call) >= 0 && (short)((f >> 16) - call) >= 0);
                    const int k = (m == ~0ull ? 64 : __builtin_ctzll(~m)) / (FW / 2);
                    if (k > 0) {
                        rdy = min(rdy + k, NI);
                        if (lane < 32) __hip_atomic_store(prog, call * 64u + (unsigned)rdy, __ATOMIC_RELAXED, __HIP_MEMORY_SCOPE_AGENT);
                        for (int z = rdy - k + 1; z <= rdy; ++z) STAMPF(2 + z);
                    } else if ((++spins & 15) == 0 &&
                               ((unsigned)__builtin_amdgcn_readfirstlane(__hip_atomic_load(tmo, __ATOMIC_RELAXED, __HIP_MEMORY_SCOPE_AGENT)) == call ||
                                __builtin_amdgcn_s_memrealtime() - t0 > DEADLINE_TICKS)) {
                        if (lane == 0) __hip_atomic_store(tmo, call, __ATOMIC_RELAXED, __HIP_MEMORY_SCOPE_AGENT);
                        break;
                    }
                }
                STAMP(61);
            }
            return;
        }
        const int n = lane & 15, q = lane >> 4;
        const int v = wid * NPU + cg;
        GAS unsigned* const tkt = (GAS unsigned*)(myws + WS_TKT);
        u32x4 d[4] = {};
        unsigned tk[4] = {0u, 0u, 0u, 0u};
        int id[2 * PC + 1];
#pragma unroll
        for (int i = 0; i <= 2 * PC; ++i) id[i] = i < PC + 1 ? -1 : v + (i - PC - 1) * NPW;
#define PUSH_LOAD(D, U, PP) do { const int pc_ = ((U) < NU ? (U) : NU - 1) * PU + (PP); \
        const char* s_ = p.x + (size_t)((pc_ >> 6) * IROWS + n) * KD + (pc_ & 63) * 64 + q * 16; \
        asm volatile("global_load_dwordx4 %0, %1, off" : "=&v"(D) : "v"(s_) : "memory"); } while (0)
#pragma unroll
        for (int k = 0; k < WP; ++k) PUSH_LOAD(d[k], v + (k / PU) * NPW, k % PU);
        const unsigned cv = __hip_atomic_load(cnt, __ATOMIC_RELAXED, __HIP_MEMORY_SCOPE_AGENT);
        asm volatile("s_waitcnt vmcnt(0)" : "+v"(d[0]), "+v"(d[1]), "+v"(d[2]) :: "memory");
        const unsigned call = (unsigned)__builtin_amdgcn_readfirstlane(cv) + 1u;
        const unsigned tbase = (call - 1u) * (unsigned)TICKETS - 2u * PC * NPW;
        const size_t soff = WS_STAGE + (size_t)(call & 1u) * (NR * SHBYTES) + (size_t)rank * SHBYTES + lane * 16;
        char* fws = p.ws[1];
#pragma unroll
        for (int i = 2; i < NR; ++i) fws = lane == i - 1 ? p.ws[i] : fws;
        GAS unsigned short* const flag = (GAS unsigned short*)(fws + WS_FLAG) + rank * NU;
        GAS unsigned short* const dummy = (GAS unsigned short*)(myws + WS_DUMMY + cg * 128) + wid * 8 + (lane & 7);
        const unsigned one = 1u;
        const unsigned tp0 = (unsigned)__builtin_amdgcn_s_memrealtime();
        STAMP(1);
#define PUSH_FLAG(U) do { GAS unsigned short* f_ = (U) >= 0 ? flag + (U) : dummy; \
        if (lane < NR - 1) asm volatile("global_store_short %0, %1, off sc0 sc1" :: "v"(f_), "v"(call) : "memory"); } while (0)
        for (int nu = 0;; ) {
#pragma unroll
            for (int k = 0; k < 4; ++k) {
                asm volatile("s_waitcnt vmcnt(%5)" : "+v"(d[k]), "+v"(tk[0]), "+v"(tk[1]), "+v"(tk[2]), "+v"(tk[3]) : "i"(push_allowed(k)) : "memory");
                if (k % PU == 0) {
#pragma unroll
                    for (int i = 0; i < 2 * PC; ++i) id[i] = id[i + 1];
                    const int tv = (int)((unsigned)__builtin_amdgcn_readfirstlane(tk[((k / PU) + 4 - PC) % (4 / PU)]) - tbase);
                    id[2 * PC] = nu < PC ? v + (nu + PC) * NPW : tv;
                    if (id[PC] >= NU) goto push_done;
                    ++nu;
                }
                if ((k + 4 * PU - WP) % PU == PU - 1) PUSH_FLAG(id[PC - (WP - (k % PU) + PU - 1) / PU]);
                if (k % PU == 0) { if (lane == 0) asm volatile("global_atomic_add %0, %1, %2, off sc0" : "=&v"(tk[(k / PU) % (4 / PU)]) : "v"(tkt), "v"(one) : "memory"); }
                PUSH_LOAD(d[(k + WP) & 3], id[PC + ((k % PU) + WP) / PU], ((k % PU) + WP) % PU);
                if (PACE > 0) {
                    const int due = (id[PC] * (PU * PACE) / 32 + (k % PU) * (PACE * NPW / 32) - LEAD) / 10;
                    while ((int)((unsigned)__builtin_amdgcn_s_memrealtime() - tp0) < due) __builtin_amdgcn_s_sleep(1);
                }
                const size_t po = soff + (size_t)(id[PC] * PU + (k % PU)) * 1024;
#pragma unroll
                for (int pr = 0; pr < NR - 1; ++pr)
                    asm volatile("global_store_dwordx4 %0, %1, off " PUSH_BITS "\n\ts_nop 1" :: "v"(p.ws[1 + pr] + po), "v"(d[k]) : "memory");
            }
        }
push_done:
        asm volatile("s_waitcnt vmcnt(0)" : "+v"(d[0]), "+v"(d[1]), "+v"(d[2]), "+v"(d[3]), "+v"(tk[0]), "+v"(tk[1]), "+v"(tk[2]), "+v"(tk[3]) :: "memory");
#pragma unroll
        for (int i = 0; i < PC; ++i) PUSH_FLAG(id[i]);
        if (lane == 0) __hip_atomic_store(cnt, call, __ATOMIC_RELAXED, __HIP_MEMORY_SCOPE_AGENT);
        STAMP(63);
        return;
    }

    const int pi = rg - 1;
    const unsigned srank = (rank + 1u + (unsigned)pi) & (NR - 1);
    const int nloc = pi < 5 ? 4 : 6;
    const int l0 = pi < 5 ? 4 * pi : 6 * pi - 10;
    const int q = lane >> 4, n = lane & 15;
#define SB0 __builtin_amdgcn_sched_barrier(0)
#define WAITV(N) asm volatile("s_waitcnt vmcnt(%c0)" :: "i"(N) : "memory")
    typedef const __attribute__((address_space(4))) float* cfloat_p;
    const float scale_s = *(cfloat_p)p.sx * *(cfloat_p)p.sw;

    const int lu = (pi * 32 + cg) * 4 + wid;
    unsigned call;
    const char* stage;
    {
        const __amdgpu_buffer_rsrc_t rx = mk_rsrc(p.x, lu < NLU ? (unsigned)SHBYTES : 0u);
        const __amdgpu_buffer_rsrc_t rw = mk_rsrc(p.w, (unsigned)KD * NL);
        unsigned cv;
        u32x4 lx0, lx1, lx2, lx3;
        asm volatile("global_load_dword %0, %5, off sc1\n\t"
                     "buffer_load_dwordx4 %1, %6, %7, %8 offen\n\tbuffer_load_dwordx4 %2, %6, %7, %8 offen offset:64\n\t"
                     "buffer_load_dwordx4 %3, %6, %7, %8 offen offset:128\n\tbuffer_load_dwordx4 %4, %6, %7, %8 offen offset:192"
                     : "=&v"(cv), "=&v"(lx0), "=&v"(lx1), "=&v"(lx2), "=&v"(lx3)
                     : "v"(cnt), "v"(n * KD + q * 16), "s"(rx), "s"(((lu * LU) >> 6) * IBYTES + ((lu * LU) & 63) * 64) : "memory");
        const int vw0 = (lane >> 2) * NL + (((lane & 3) ^ (q & 1)) * 16), vw1 = (lane >> 2) * NL + (((lane & 3) ^ (q & 1) ^ 2) * 16);
        const int sw = (wid * 1024) * NL + cg * 64;
        const unsigned lw = (unsigned)(size_t)(LAS char*)lds + wid * 32768;
        unsigned tr[4];
#pragma unroll
        for (int j = 0; j < 4; ++j) tr[j] = lw + (q * 16 + (n >> 1)) * 64 + ((j ^ ((n >> 3) | ((q & 1) << 1))) * 16) + (n & 1) * 8;
#define W_DMA(KS) do { _Pragma("unroll") for (int i_ = 0; i_ < 8; ++i_) \
            asm volatile("s_mov_b32 m0, %3\n\ts_nop 0\n\tbuffer_load_dwordx4 %0, %1, %2 offen lds" \
                         :: "v"((i_ & 1) ? vw1 : vw0), "s"(rw), "s"(sw + ((KS) * 128 + i_ * 16) * NL), "s"(lw + ((KS) & 3) * 8192 + i_ * 1024) : "memory"); } while (0)
#define W_TR(KS) do { _Pragma("unroll") for (int j_ = 0; j_ < 4; ++j_) \
            asm volatile("ds_read_b64_tr_b8 a[%c1:%c1+1], %0 offset:%c2\n\tds_read_b64_tr_b8 a[%c1+2:%c1+3], %0 offset:%c2+512\n\t" \
                         "ds_read_b64_tr_b8 a[%c1+4:%c1+5], %0 offset:%c2+4096\n\tds_read_b64_tr_b8 a[%c1+6:%c1+7], %0 offset:%c2+4608" \
                         :: "v"(tr[j_]), "i"((KS) * 32 + j_ * 8), "i"(((KS) & 3) * 8192) : "memory"); \
            asm volatile("s_waitcnt lgkmcnt(0)" ::: "memory"); } while (0)
        asm volatile("" ::: "a0", "a1", "a2", "a3", "a4", "a5", "a6", "a7", "a8", "a9", "a10", "a11", "a12", "a13", "a14", "a15", "a16", "a17", "a18", "a19", "a20", "a21", "a22", "a23", "a24", "a25", "a26", "a27", "a28", "a29", "a30", "a31", "a32", "a33", "a34", "a35", "a36", "a37", "a38", "a39", "a40", "a41", "a42", "a43", "a44", "a45", "a46", "a47", "a48", "a49", "a50", "a51", "a52", "a53", "a54", "a55", "a56", "a57", "a58", "a59", "a60", "a61", "a62", "a63", "a64", "a65", "a66", "a67", "a68", "a69", "a70", "a71", "a72", "a73", "a74", "a75", "a76", "a77", "a78", "a79", "a80", "a81", "a82", "a83", "a84", "a85", "a86", "a87", "a88", "a89", "a90", "a91", "a92", "a93", "a94", "a95", "a96", "a97", "a98", "a99", "a100", "a101", "a102", "a103", "a104", "a105", "a106", "a107", "a108", "a109", "a110", "a111", "a112", "a113", "a114", "a115", "a116", "a117", "a118", "a119", "a120", "a121", "a122", "a123", "a124", "a125", "a126", "a127", "a128", "a129", "a130", "a131", "a132", "a133", "a134", "a135", "a136", "a137", "a138", "a139", "a140", "a141", "a142", "a143", "a144", "a145", "a146", "a147", "a148", "a149", "a150", "a151", "a152", "a153", "a154", "a155", "a156", "a157", "a158", "a159", "a160", "a161", "a162", "a163", "a164", "a165", "a166", "a167", "a168", "a169", "a170", "a171", "a172", "a173", "a174", "a175", "a176", "a177", "a178", "a179", "a180", "a181", "a182", "a183", "a184", "a185", "a186", "a187", "a188", "a189", "a190", "a191", "a192", "a193", "a194", "a195", "a196", "a197", "a198", "a199", "a200", "a201", "a202", "a203", "a204", "a205", "a206", "a207", "a208", "a209", "a210", "a211", "a212", "a213", "a214", "a215", "a216", "a217", "a218", "a219", "a220", "a221", "a222", "a223", "a224", "a225", "a226", "a227", "a228", "a229", "a230", "a231", "a232", "a233", "a234", "a235", "a236", "a237", "a238", "a239", "a240", "a241", "a242", "a243", "a244", "a245", "a246", "a247", "a248", "a249", "a250", "a251", "a252", "a253", "a254", "a255");
        unsigned t0_, t1_;
        {
            const int wv = ((b >> 3) - 4) * 4 + wid, r0 = wv * 37 + (lane >> 1);
            const int cb = (cg / BC) * (BC * 64) + (lane & 1) * 128;
            asm volatile("buffer_load_dword %0, %2, %4, 0 offen\n\tbuffer_load_dword %1, %3, %4, 0 offen"
                         : "=&v"(t0_), "=&v"(t1_) : "v"(r0 * NL + cb), "v"(lane < 10 ? (r0 + 32) * NL + cb : -1), "s"(rw) : "memory");
        }
#define NT 2
        W_DMA(0);
        asm volatile("s_waitcnt vmcnt(%c5)" : "+v"(cv), "+v"(lx0), "+v"(lx1), "+v"(lx2), "+v"(lx3) : "i"(8 + NT) : "memory");
        call = (unsigned)__builtin_amdgcn_readfirstlane(cv) + 1u;
        stage = myws + WS_STAGE + (size_t)(call & 1u) * (NR * SHBYTES);
        const __amdgpu_buffer_rsrc_t rl = mk_rsrc(stage + (size_t)rank * SHBYTES, lu < NLU ? (unsigned)SHBYTES : 0u);
        asm volatile("s_nop 4\n\tglobal_store_dword %0, %1, off sc1\n\t"
                     "buffer_store_dwordx4 %2, %6, %7, %8 offen sc0 sc1\n\tbuffer_store_dwordx4 %3, %6, %7, %8 offen offset:1024 sc0 sc1\n\t"
                     "buffer_store_dwordx4 %4, %6, %7, %8 offen offset:2048 sc0 sc1\n\tbuffer_store_dwordx4 %5, %6, %7, %8 offen offset:3072 sc0 sc1\n\ts_nop 1"
                     :: "v"(cnt), "v"(call), "v"(lx0), "v"(lx1), "v"(lx2), "v"(lx3), "v"(lane * 16), "s"(rl), "s"(lu * (LU * 1024)) : "memory");
        W_DMA(1); W_DMA(2); W_DMA(3);
        WAITV(24);
        asm volatile("" :: "v"(t0_), "v"(t1_));
        if (lane == 0 && lu < NLU) __hip_atomic_store((GAS unsigned*)(myws + WS_LFLAG + lu * 128), call, __ATOMIC_RELAXED, __HIP_MEMORY_SCOPE_SYSTEM);
        STAMP(40); W_TR(0); W_DMA(4);
        WAITV(24); W_TR(1); W_DMA(5);
        WAITV(24); W_TR(2); W_DMA(6);
        WAITV(24); STAMP(41); W_TR(3); W_DMA(7);
        WAITV(24); W_TR(4);
        WAITV(16); W_TR(5);
        WAITV(8); W_TR(6);
        WAITV(0); W_TR(7);
    }
    STAMP(1);

    GAS unsigned* const tmo = (GAS unsigned*)(myws + WS_TMO);
    const __amdgpu_buffer_rsrc_t rs = mk_rsrc(stage, (unsigned)NR * SHBYTES);
    const char* const prog = myws + WS_PROG + (size_t)pi * 4096 + cg * 128;
    const int voff_a = lane * 16;
    const int voff_out = ((wid * 4 + q) * NL + n * 4) * 4;
    bool dead = false;
#define SPIN(READY_EXPR, LOADV) do { \
        const unsigned long long t0_ = __builtin_amdgcn_s_memrealtime(); \
        for (int spins_ = 1;; ++spins_) { \
            const unsigned v_ = LOADV; \
            if (READY_EXPR) break; \
            if ((spins_ & 15) == 0 && ((unsigned)__builtin_amdgcn_readfirstlane(__hip_atomic_load(tmo, __ATOMIC_RELAXED, __HIP_MEMORY_SCOPE_AGENT)) == call || \
                __builtin_amdgcn_s_memrealtime() - t0_ > DEADLINE_TICKS)) { \
                if (lane == 0) __hip_atomic_store(tmo, call, __ATOMIC_RELAXED, __HIP_MEMORY_SCOPE_AGENT); \
                dead = true; \
                rdy = NI; \
                break; \
            } \
            __builtin_amdgcn_s_sleep(1); \
        } } while (0)
#define SAMPLE() ({ unsigned sv_; asm volatile("s_load_dword %0, %1, 0x0 glc\n\ts_waitcnt lgkmcnt(0)" : "=&s"(sv_) : "s"(prog) : "memory"); sv_; })
#define ADVANCE(V) do { rdy = max(rdy, (int)((V) - call * 64u)); } while (0)
    int rdy = 0;
    {
        const __amdgpu_buffer_rsrc_t rfl = mk_rsrc(myws + WS_LFLAG, (unsigned)NLU * 128u);
        constexpr int LPI = 64 / LU;
        SPIN(__all(lane >= nloc * 4 || (int)(v_ - call) >= 0), __builtin_amdgcn_raw_buffer_load_b32(rfl, (((lane >> 2) + l0) * LPI + wid * 4 + (lane & 3)) * 128, 0, 17));
    }
    STAMP(42);

#define AREG(B, KS) (128 + 64 * (B) + 8 * (KS))
#define LOAD_FRAG(B, KS, RS, SB) asm volatile( \
        "buffer_load_dwordx4 v[%c3:%c4], %0, %1, %2 offen offset:%c7 sc0 sc1\n\tbuffer_load_dwordx4 v[%c5:%c6], %0, %1, %2 offen offset:%c8 sc0 sc1" \
        :: "v"(voff_a), "s"(RS), "s"((SB) + ((KS) >> 1) * 4096), "i"(AREG(B, KS)), "i"(AREG(B, KS) + 3), "i"(AREG(B, KS) + 4), "i"(AREG(B, KS) + 7), \
           "i"(((KS) & 1) * 2048), "i"(((KS) & 1) * 2048 + 1024) : "memory")
#define SAMPLE_ISSUE() asm volatile("s_load_dword s96, %0, 0x0 glc" :: "s"(prog) : "memory", "s96")
#define SAMPLE_TAKE() do { unsigned sv_; asm volatile("s_mov_b32 %0, s96" : "=s"(sv_) :: "memory"); ADVANCE(sv_); } while (0)
#define LOOP_REGS "s96", "v74", "v75", "v76", "v77", "v78", "v79", "v80", "v81", "v82", "v83", "v84", "v85", "v86", "v87", "v88", "v89", "v90", "v91", "v92", "v93", "v94", "v95", "v96", "v97", "v98", "v99", "v100", "v101", "v102", "v103", "v104", "v105", "v106", "v107", "v108", "v109", "v110", "v111", "v112", "v113", "v114", "v115", "v116", "v117", "v118", "v119", "v120", "v121", "v122", "v123", "v124", "v125", "v126", "v127", "v128", "v129", "v130", "v131", "v132", "v133", "v134", "v135", "v136", "v137", "v138", "v139", "v140", "v141", "v142", "v143", "v144", "v145", "v146", "v147", "v148", "v149", "v150", "v151", "v152", "v153", "v154", "v155", "v156", "v157", "v158", "v159", "v160", "v161", "v162", "v163", "v164", "v165", "v166", "v167", "v168", "v169", "v170", "v171", "v172", "v173", "v174", "v175", "v176", "v177", "v178", "v179", "v180", "v181", "v182", "v183", "v184", "v185", "v186", "v187", "v188", "v189", "v190", "v191", "v192", "v193", "v194", "v195", "v196", "v197", "v198", "v199", "v200", "v201", "v202", "v203", "v204", "v205", "v206", "v207", "v208", "v209", "v210", "v211", "v212", "v213", "v214", "v215", "v216", "v217", "v218", "v219", "v220", "v221", "v222", "v223", "v224", "v225", "v226", "v227", "v228", "v229", "v230", "v231", "v232", "v233", "v234", "v235", "v236", "v237", "v238", "v239", "v240", "v241", "v242", "v243", "v244", "v245", "v246", "v247", "v248", "v249", "v250", "v251", "v252", "v253", "v254", "v255"
#define MFMA0(B, J) asm volatile("v_mfma_f32_16x16x128_f8f6f4 v[%c0:%c0+3], a[%c1:%c1+7], v[%c2:%c2+7], v[74:77]" :: "i"(110 + 4 * (J)), "i"(8 * (J)), "i"(AREG(B, 0)))
#define MFMA(B, KS, J) asm volatile("v_mfma_f32_16x16x128_f8f6f4 v[%c0:%c0+3], a[%c1:%c1+7], v[%c2:%c2+7], v[%c0:%c0+3]" \
        :: "i"(110 + 4 * (J)), "i"(32 * (KS) + 8 * (J)), "i"(AREG(B, KS)))
#define SET_CINIT() asm volatile("v_mov_b32 v74, %0\n\tv_mov_b32 v75, %0\n\tv_mov_b32 v76, %0\n\tv_mov_b32 v77, %0" :: "s"(dead ? 0x7fc00000u : 0u))
#define PUT(B, J) asm volatile("ds_write_b128 %0, v[%c1:%c1+3] offset:%c2" :: "v"(put[J]), "i"(110 + 4 * (J)), "i"((B) * RED_BYTES) : "memory")
#define GET4(B) asm volatile("ds_read_b128 v[78:81], %0 offset:%c1\n\tds_read_b128 v[82:85], %0 offset:%c1+4096\n\t" \
        "ds_read_b128 v[86:89], %0 offset:%c1+8192\n\tds_read_b128 v[90:93], %0 offset:%c1+12288" :: "v"(get), "i"((B) * RED_BYTES) : "memory")
#define VOP4(OP, D, A, C) asm volatile(OP " v%c0, v%c4, v%c8\n\t" OP " v%c1, v%c5, v%c9\n\t" OP " v%c2, v%c6, v%c10\n\t" OP " v%c3, v%c7, v%c11" \
        :: "i"(D), "i"((D) + 1), "i"((D) + 2), "i"((D) + 3), "i"(A), "i"((A) + 1), "i"((A) + 2), "i"((A) + 3), "i"(C), "i"((C) + 1), "i"((C) + 2), "i"((C) + 3))
#define VOPK4(OP, D, K) asm volatile(OP " v%c1, %0, v%c1\n\t" OP " v%c2, %0, v%c2\n\t" OP " v%c3, %0, v%c3\n\t" OP " v%c4, %0, v%c4" \
        :: "s"(K), "i"(D), "i"((D) + 1), "i"((D) + 2), "i"((D) + 3))
#define VUN2(OP, D) asm volatile(OP " v%c0, v%c0\n\t" OP " v%c1, v%c1" :: "i"(D), "i"((D) + 1))
#define CLAMP4(D, A) asm volatile("v_med3_f32 v%c2, v%c6, %0, %1\n\tv_med3_f32 v%c3, v%c7, %0, %1\n\tv_med3_f32 v%c4, v%c8, %0, %1\n\tv_med3_f32 v%c5, v%c9, %0, %1" \
        :: "s"(-60.0f), "v"(c60), "i"(D), "i"((D) + 1), "i"((D) + 2), "i"((D) + 3), "i"(A), "i"((A) + 1), "i"((A) + 2), "i"((A) + 3))
#define STORE4(A, RS, SO) asm volatile("buffer_store_dwordx4 v[%c3:%c4], %0, %1, %2 offen " OUT_BITS :: "v"(voff_out), "s"(RS), "s"(SO), "i"(A), "i"((A) + 3) : "memory")
#define PIN(X) asm volatile("" : "+v"(X))
#define NEXT_SA(TN, SA) ((TN) == 0 ? (int)srank * SHBYTES + wid * 16384 : (SA) + IBYTES)
    const unsigned red = (unsigned)(size_t)(LAS char*)lds + WRING_BYTES;
    unsigned put[4];
#pragma unroll
    for (int j = 0; j < 4; ++j) put[j] = red + wid * 4096 + j * 1024 + q * 256 + ((n ^ (4 * j + q)) * 16);
    const unsigned get = red + (n >> 2) * 1024 + (n & 3) * 256 + (((4 * wid + q) ^ n) * 16);
    float c60 = 60.0f;
    PIN(c60);
    constexpr float NLOG2E = -1.44269504088896340736f;
    int sa = (int)rank * SHBYTES + l0 * IBYTES + wid * 16384;
    int sa1 = sa + IBYTES;
    int so = (((int)rank * SH + l0 * IROWS) * NL + cg * 64) * 4;
    int so1 = 0, so2 = 0;
    int req = 0;
    unsigned orec = 0;
    const char* outp = (const char*)p.out;
    asm volatile("" : "+s"(outp));
#define DBG_TOP do {} while (0)
#define DBG_L0 do {} while (0)
#define DBG_L1 do {} while (0)
#define DBG_END do {} while (0)
#define DBG_TOP1 do {} while (0)
#define DBG_SEEN do {} while (0)
#define EDGE1(B, K) do { \
        if (H == 0) { X_LOAD(LOAD_FRAG(1 - (B), K, rs, sa1)); WAITV(14); } \
        if (H == 1) { LOAD_FRAG(B, (K) + 4, rs, sa); WAITV(6); } \
        if (H == 2) WAITV(12 - 2 * (K)); } while (0)
#define EDGE2(B, K) do { \
        if (H == 0) { X_LOAD(LOAD_FRAG(1 - (B), K, rs, sa1)); if ((K) < 7) WAITV(14 + ((K) == 6)); } \
        if (H == 1) { LOAD_FRAG(1 - (B), (K) - 4, rs, sa1); if ((K) < 7) WAITV(6 + ((K) == 6)); } \
        if (H == 2 && (K) < 7) WAITV(12 - 2 * (K) + ((K) == 6)); \
        if (H == 3) { LOAD_FRAG(1 - (B), 2 * (K) - 8, rs, sa1); LOAD_FRAG(1 - (B), 2 * (K) - 7, rs, sa1); if ((K) < 7) WAITV(2 * (K) + ((K) == 6)); } } while (0)
#define HALF1(B, H_) do { \
        constexpr int H = H_; \
        X_LDS(GET4(B)); if (H != 0) SAMPLE_ISSUE(); X_LDS(PUT(1 - (B), 0)); WAITV(H == 1 ? 6 : 14); \
        MFMA0(B, 0); X_LDS(PUT(1 - (B), 1)); MFMA0(B, 1); X_LDS(PUT(1 - (B), 2)); MFMA0(B, 2); X_LDS(PUT(1 - (B), 3)); MFMA0(B, 3); \
        EDGE1(B, 0); \
          \
        MFMA(B, 1, 0); MFMA(B, 1, 1); asm volatile("s_waitcnt lgkmcnt(4)" ::: "memory"); X_VALU(VOP4("v_add_f32", 78, 78, 82)); \
        MFMA(B, 1, 2); X_VALU(VOP4("v_add_f32", 86, 86, 90)); MFMA(B, 1, 3); \
        EDGE1(B, 1); \
        MFMA(B, 2, 0); MFMA(B, 2, 1); X_VALU(VOP4("v_add_f32", 78, 78, 86)); MFMA(B, 2, 2); X_VALU(VOPK4("v_mul_f32", 78, scale_s)); MFMA(B, 2, 3); \
        EDGE1(B, 2); \
        MFMA(B, 3, 0); MFMA(B, 3, 1); X_VALU(CLAMP4(82, 78)); MFMA(B, 3, 2); X_VALU(VOPK4("v_mul_f32", 82, NLOG2E)); MFMA(B, 3, 3); \
        EDGE1(B, 3); \
        MFMA(B, 4, 0); \
        if (H != 0) { asm volatile("s_waitcnt lgkmcnt(0)" ::: "memory"); SAMPLE_TAKE(); } \
        SAMPLE_ISSUE(); \
    } while (0)
#define HALF2(B, H_) do { \
        constexpr int H = H_; \
        const __amdgpu_buffer_rsrc_t ro = mk_rsrc(outp, orec); \
        MFMA(B, 4, 1); X_VALU(VUN2("v_exp_f32", 82)); MFMA(B, 4, 2); X_VALU(VUN2("v_exp_f32", 84)); MFMA(B, 4, 3); \
        EDGE2(B, 4); \
        MFMA(B, 5, 0); X_VALU(VOPK4("v_add_f32", 82, 1.0f)); MFMA(B, 5, 1); X_VALU(VUN2("v_rcp_f32", 82)); MFMA(B, 5, 2); X_VALU(VUN2("v_rcp_f32", 84)); MFMA(B, 5, 3); \
        EDGE2(B, 5); \
        MFMA(B, 6, 0); X_VALU(VOP4("v_mul_f32", 78, 78, 82)); MFMA(B, 6, 1); X_STORE(STORE4(78, ro, so2)); SB0; MFMA(B, 6, 2); SB0; \
        so2 = so1; so1 = so; so = tn == 0 ? (((int)srank * SH) * NL + cg * 64) * 4 : so + IROWS * NL * 4; \
        orec = tn + nloc >= NO_STORE_T ? (unsigned)NR * SH * NL * 4u : 0u; \
        asm volatile("" : "+s"(so), "+s"(orec)); SB0; MFMA(B, 6, 3); \
        EDGE2(B, 6); \
        MFMA(B, 7, 0); SB0; \
        int tq = tn + 1; asm volatile("" : "+s"(tq)); \
        int sax = NEXT_SA(tq, sa1); asm volatile("" : "+s"(sax)); SB0; MFMA(B, 7, 1); MFMA(B, 7, 2); MFMA(B, 7, 3); \
        EDGE2(B, 7); SB0; \
        DBG_END; \
        req = H == 1 ? 4 : H == 2 ? 0 : 8; sa = sa1; sa1 = sax; \
        if (tn >= 21 && wid == 0) STAMPF(2 + tn); \
        asm volatile("s_waitcnt lgkmcnt(0)\n\t" X_BAR ::: "memory", LOOP_REGS); \
        SAMPLE_TAKE(); \
    } while (0)
#define ITERATION(B) do { \
        if (req == 8 && tn < rdy) { HALF1(B, 0); HALF2(B, 0); } \
        else { \
            DBG_TOP1; \
            if (req == 0) { \
                DBG_TOP; \
                if (tn > rdy) { SPIN(({ ADVANCE(v_); rdy >= tn; }), SAMPLE()); SET_CINIT(); __builtin_amdgcn_s_waitcnt(0xC07F); } \
                DBG_SEEN; \
                LOAD_FRAG(B, 0, rs, sa); LOAD_FRAG(B, 1, rs, sa); LOAD_FRAG(B, 2, rs, sa); LOAD_FRAG(B, 3, rs, sa); \
            } \
            if (req == 8) HALF1(B, 2); else HALF1(B, 1); \
            if (tn >= rdy) HALF2(B, 2); else if (tn + UPG < rdy) HALF2(B, 3); else HALF2(B, 1); \
        } \
    } while (0)
    asm volatile("; item loop" ::: "memory", LOOP_REGS);
    SET_CINIT();
    for (int tn = 1 - nloc; tn <= NI; ++tn) {
        ITERATION(0);
        ++tn;
        ITERATION(1);
    }
    {
        const __amdgpu_buffer_rsrc_t ro = mk_rsrc(outp, (unsigned)NR * SH * NL * 4u);
        asm volatile("s_nop 7" ::: "memory");
        PUT(1, 0); PUT(1, 1); PUT(1, 2); PUT(1, 3);
        GET4(0);
        asm volatile("s_waitcnt lgkmcnt(4)\n\ts_barrier" ::: "memory", LOOP_REGS);
        asm volatile("ds_read_b128 v[94:97], %0 offset:%c1\n\tds_read_b128 v[98:101], %0 offset:%c1+4096\n\t"
                     "ds_read_b128 v[102:105], %0 offset:%c1+8192\n\tds_read_b128 v[106:109], %0 offset:%c1+12288\n\ts_waitcnt lgkmcnt(4)" :: "v"(get), "i"(RED_BYTES) : "memory");
        VOP4("v_add_f32", 78, 78, 82); VOP4("v_add_f32", 86, 86, 90); VOP4("v_add_f32", 78, 78, 86);
        asm volatile("s_waitcnt lgkmcnt(0)" ::: "memory");
        VOP4("v_add_f32", 94, 94, 98); VOP4("v_add_f32", 102, 102, 106); VOP4("v_add_f32", 94, 94, 102);
        VOPK4("v_mul_f32", 94, scale_s); CLAMP4(98, 94); VOPK4("v_mul_f32", 98, NLOG2E);
        VUN2("v_exp_f32", 98); VUN2("v_exp_f32", 100); VOPK4("v_mul_f32", 78, scale_s); VOPK4("v_add_f32", 98, 1.0f);
        VUN2("v_rcp_f32", 98); VUN2("v_rcp_f32", 100); CLAMP4(82, 78); VOP4("v_mul_f32", 94, 94, 98);
        STORE4(94, ro, so1);
        VOPK4("v_mul_f32", 82, NLOG2E); VUN2("v_exp_f32", 82); VUN2("v_exp_f32", 84); asm volatile("s_nop 0"); VOPK4("v_add_f32", 82, 1.0f);
        VUN2("v_rcp_f32", 82); VUN2("v_rcp_f32", 84); asm volatile("s_nop 0"); VOP4("v_mul_f32", 78, 78, 82);
        STORE4(78, ro, so2);
    }
    STAMP(63);
}

}

extern "C" void kernel_launch(int rank, int nranks, ncclComm_t comm,
                              hipStream_t stream,
                              void* const* d_in, const int* in_sizes, int n_in,
                              void* d_out, int out_size,
                              const int* in_dtypes, int out_dtype,
                              void* const* d_out_all,
                              void* const* d_ws_all, size_t ws_bytes) {
    (void)comm; (void)in_dtypes; (void)out_dtype; (void)d_out_all;
    if (nranks != NR || rank < 0 || rank >= NR || n_in < 4 || in_sizes[0] != SH * KD || in_sizes[1] != KD * NL ||
        out_size != NR * SH * NL || ws_bytes < WS_NEED) {
        fprintf(stderr, "ag_gemm: unexpected problem shape\n");
        return;
    }
    Params p;
    for (int i = 0; i < NR; ++i) p.ws[i] = (char*)d_ws_all[(rank + i) % NR];
    p.x = (const char*)d_in[0];
    p.w = (const char*)d_in[1];
    p.sx = (const float*)d_in[2];
    p.sw = (const float*)d_in[3];
    p.out = (float*)d_out;
    p.rank = rank;
    p.pad = 0;
    ag_gemm_kernel<<<dim3(NWG), dim3(256), 0, stream>>>(p);
    HIP_CHECK(hipGetLastError());
}
```

```cpp
#include <hip/hip_runtime.h>
#include <rccl/rccl.h>
#include <cstdio>

#define HIP_CHECK(expr) do { hipError_t _e = (expr); if (_e != hipSuccess) { \
    fprintf(stderr, "HIP error %s at %s:%d\n", hipGetErrorString(_e), __FILE__, __LINE__); \
    return; } } while (0)

namespace {

typedef int v8i __attribute__((ext_vector_type(8)));
typedef unsigned u32x4 __attribute__((ext_vector_type(4)));
typedef float v4f __attribute__((ext_vector_type(4)));
typedef short short2v __attribute__((ext_vector_type(2)));
#define LAS __attribute__((address_space(3)))
#define GAS __attribute__((address_space(1)))

constexpr int NR = 8;
constexpr int KD = 4096;
constexpr int NL = 2048;
constexpr int SH = 512;
constexpr int IROWS = 16;
constexpr int NI = SH / IROWS;
constexpr int IBYTES = IROWS * KD;
constexpr int SHBYTES = SH * KD;
constexpr int NP = 32;
#define NWATCH 2
constexpr int NWG = 256;
#define BC 8
#define A_AUX 17
#define PUSH_BITS "sc0 sc1"
#define OUT_BITS "sc1"
#define NO_STORE_T 2
#define W_TOUCH 1
#define PUSH_FIRST 1
#define UPG_STR "3"
#define ABL 0
#define X_VALU(x) do { if (!(ABL & 1)) { x; } } while (0)
#define X_LDS(x) do { if (!(ABL & 2)) { x; } } while (0)
#define X_STORE(x) do { if (!(ABL & 4)) { x; } } while (0)
#define X_SAMPLE(x) do { if (!(ABL & 8)) { x; } } while (0)
#define X_LOAD(x) do { if (!(ABL & 16)) { x; } } while (0)
#define X_BAR "s_barrier"
#define PACE 565
#define LEAD 2800
#define I1 28
constexpr int NPU = NP - NWATCH;
constexpr int NU = SHBYTES / 2048;
constexpr int G1 = 16 * I1;
constexpr int NGR = G1 + 32 * (NI - I1);

constexpr size_t WS_CNT = 0;
constexpr size_t WS_TMO = 32768;
constexpr size_t WS_TKT = 32768 + 256;
constexpr size_t WS_PROG = 131072;
constexpr size_t WS_FLAG = 65536;
constexpr size_t WS_LFLAG = 262144;
constexpr size_t WS_DUMMY = 49152;
constexpr size_t WS_STAGE = 2u << 20;
constexpr size_t WS_NEED = WS_STAGE + 2ull * NR * SHBYTES;
constexpr size_t WS_DBG = 64u << 20;
constexpr unsigned long long DEADLINE_TICKS = 40000000ull;
static_assert(WS_FLAG + (size_t)NR * 4096 <= WS_PROG && NGR <= 1024 && NPU <= G1 && WS_PROG + 224 * 128 <= WS_LFLAG && WS_LFLAG + 512 * 128 <= WS_STAGE, "flags");
constexpr int LU = 4;
constexpr int NLU = SHBYTES / 1024 / LU;
static_assert(NLU <= (NWG - NP) * 4, "one local unit per consumer wave");

struct Params {
    char* ws[NR];
    const char* x;
    const char* w;
    const float* sx;
    const float* sw;
    float* out;
    int rank;
    int pad;
};

#define STAMP(i) do {} while (0)
#define STAMPF(i) do {} while (0)

__device__ __forceinline__ __amdgpu_buffer_rsrc_t mk_rsrc(const void* base, unsigned bytes) {
    return __builtin_amdgcn_make_buffer_rsrc((void*)base, 0, bytes, 0x00020000);
}

__device__ __forceinline__ float silu_dq(float acc, float s) {
    const float y = acc * s;
    const float c = __builtin_amdgcn_fmed3f(y, -60.0f, 60.0f);
    const float e = __builtin_amdgcn_exp2f(c * -1.44269504088896340736f);
    return y * __builtin_amdgcn_rcpf(1.0f + e);
}

constexpr int WRING_BYTES = 4 * 4 * 8192;
constexpr int RED_BYTES = 4 * 4096;

__global__ __launch_bounds__(256, 1) void ag_gemm_kernel(const Params p) {
    __shared__ __attribute__((aligned(16))) char lds[WRING_BYTES + 2 * RED_BYTES];
    const int tid = threadIdx.x;
    const int lane = tid & 63;
    const int wid = __builtin_amdgcn_readfirstlane(tid >> 6);
    const int b = blockIdx.x;
    const unsigned rank = (unsigned)p.rank;
    char* const myws = p.ws[0];
    asm volatile("" :: "s"(p.ws[0]), "s"(p.ws[7]), "s"(p.x), "s"(p.w), "s"(p.out), "s"(p.rank));

    GAS unsigned* cnt = (GAS unsigned*)(myws + WS_CNT + (size_t)b * 128 + wid * 4);

    int rg = 0, cg = b;
    if (b >= NP) {
        const int i = (b & 7) * 28 + (b >> 3) - 4;
        rg = 1 + (i / BC) % 7;
        cg = (i / (7 * BC)) * BC + i % BC;
    }
    STAMP(0);
    if (__builtin_expect(rg == 0, PUSH_FIRST)) {
        if (wid != 0 || cg >= NPU) {
            const unsigned call = (unsigned)__builtin_amdgcn_readfirstlane(__hip_atomic_load(cnt, __ATOMIC_RELAXED, __HIP_MEMORY_SCOPE_AGENT)) + 1u;
            if (lane == 0) __hip_atomic_store(cnt, call, __ATOMIC_RELAXED, __HIP_MEMORY_SCOPE_AGENT);
            const int pw = (cg - NPU) * 4 + wid;
            if (pw >= 0 && pw < NR - 1) {
                const __amdgpu_buffer_rsrc_t rf = mk_rsrc(myws + WS_FLAG + (size_t)((rank + 1u + (unsigned)pw) & (NR - 1)) * 4096, (unsigned)NGR * 4u);
                GAS unsigned* const prog = (GAS unsigned*)(myws + WS_PROG + (size_t)pw * 4096 + (lane & 31) * 128);
                GAS unsigned* const tmo = (GAS unsigned*)(myws + WS_TMO);
                const unsigned long long t0 = __builtin_amdgcn_s_memrealtime();
                for (int rdy = 0, spins = 0; rdy < NI; ) {
                    const int g0 = rdy < I1 ? 16 * rdy : G1 + 32 * (rdy - I1);
                    const unsigned f = __builtin_amdgcn_raw_buffer_load_b32(rf, (lane & 31) * 4, g0 * 4, 17);
                    const unsigned m = ~(unsigned)__ballot((int)(f - call) >= 0);
                    const int k = m == 0u ? (rdy + 1 < I1 ? 2 : 1) : (rdy < I1 && (m & 0xffffu) == 0u ? 1 : 0);
                    if (k > 0) {
                        rdy += k;
                        if (lane < 32) __hip_atomic_store(prog, call * 64u + (unsigned)rdy, __ATOMIC_RELAXED, __HIP_MEMORY_SCOPE_AGENT);
                        for (int z = rdy - k + 1; z <= rdy; ++z) STAMPF(2 + z);
                    } else if ((++spins & 15) == 0 &&
                               ((unsigned)__builtin_amdgcn_readfirstlane(__hip_atomic_load(tmo, __ATOMIC_RELAXED, __HIP_MEMORY_SCOPE_AGENT)) == call ||
                                __builtin_amdgcn_s_memrealtime() - t0 > DEADLINE_TICKS)) {
                        if (lane == 0) __hip_atomic_store(tmo, call, __ATOMIC_RELAXED, __HIP_MEMORY_SCOPE_AGENT);
                        break;
                    }
                }
                STAMP(61);
            }
            return;
        }
        const unsigned vx = (lane & 15) * KD + (lane >> 4) * 16;
#define PUSH_REGS "v64", "v65", "v66", "v67", "v68", "v69", "v70", "v71", "v72", "v73", "v74", "v75", "v76", "v77", "v78", "v79", "v80", "v81"
        int prev = 0, cur = 2 * cg, nxt = 2 * cg + 1;
#define PUSH_LOAD(D, U, PP) do { const unsigned pc_ = (unsigned)min(U, NU - 1) * 2u + (PP); \
        asm volatile("global_load_dwordx4 v[%c2:%c2+3], %0, %1" :: "v"(vx + (pc_ >> 6) * IBYTES + (pc_ & 63u) * 64u), "s"(p.x), "i"(64 + 4 * (D)) : "memory"); } while (0)
        asm volatile("" ::: PUSH_REGS);
        PUSH_LOAD(0, cur, 0); PUSH_LOAD(1, cur, 1);
        asm volatile("global_load_dword v81, %0, off sc1" :: "v"(cnt) : "memory");
        char* fws = p.ws[1];
#pragma unroll
        for (int i = 2; i < NR; ++i) fws = lane == i - 1 ? p.ws[i] : fws;
        GAS unsigned* flag = (GAS unsigned*)(fws + WS_FLAG) + rank * 1024;
        GAS unsigned* sink = (GAS unsigned*)(myws + WS_DUMMY + cg * 128) + (lane & 7);
        GAS unsigned* const tkt = (GAS unsigned*)(myws + WS_TKT);
        unsigned vs = (unsigned)WS_STAGE + rank * SHBYTES + lane * 16;
        unsigned call;
        asm volatile("s_waitcnt vmcnt(0)\n\tv_readfirstlane_b32 %0, v81" : "=s"(call), "+v"(flag), "+v"(sink), "+v"(vs) :: "memory", PUSH_REGS);
        call += 1u;
        const unsigned par = (call & 1u) * (NR * SHBYTES);
        const unsigned tbase = (call - 1u) * (unsigned)NGR - (unsigned)NPU;
        const unsigned tp0 = (unsigned)__builtin_amdgcn_s_memrealtime();
        STAMP(1);
#define ENDS_GROUP(U) ((U) >= 2 * G1 || ((U) & 1))
#define PUSH_FLAG(U) do { GAS unsigned* f_ = ENDS_GROUP(U) ? flag + ((U) < 2 * G1 ? (U) >> 1 : (U) - G1) : sink; \
        asm volatile("s_mov_b64 exec, 0x7f\n\tglobal_store_dword %0, %1, off sc0 sc1\n\ts_mov_b64 exec, -1" :: "v"(f_), "v"(call) : "memory"); } while (0)
#define PUSH_STORE(D, PP) do { \
        if (PACE > 0) { const int w_ = cur < 2 * G1 ? cur & 1 : 0; \
            const int due_ = ((cur - w_) * (2 * PACE) / 32 + (2 * w_ + (PP)) * (PACE * NPU / 32) - LEAD) / 10; \
            while ((int)((unsigned)__builtin_amdgcn_s_memrealtime() - tp0) < due_) __builtin_amdgcn_s_sleep(1); } \
        const unsigned o_ = vs + par + (unsigned)(cur * 2 + (PP)) * 1024u; \
        asm volatile("global_store_dwordx4 %0, v[%c1:%c1+3], %2 sc0 sc1\n\tglobal_store_dwordx4 %0, v[%c1:%c1+3], %3 sc0 sc1\n\t" \
                     "global_store_dwordx4 %0, v[%c1:%c1+3], %4 sc0 sc1\n\tglobal_store_dwordx4 %0, v[%c1:%c1+3], %5 sc0 sc1\n\t" \
                     "global_store_dwordx4 %0, v[%c1:%c1+3], %6 sc0 sc1\n\tglobal_store_dwordx4 %0, v[%c1:%c1+3], %7 sc0 sc1\n\t" \
                     "global_store_dwordx4 %0, v[%c1:%c1+3], %8 sc0 sc1" \
                     :: "v"(o_), "i"(64 + 4 * (D)), "s"(p.ws[1]), "s"(p.ws[2]), "s"(p.ws[3]), "s"(p.ws[4]), "s"(p.ws[5]), "s"(p.ws[6]), "s"(p.ws[7]) : "memory"); } while (0)
#define PUSH_UNIT(DA, DB, DC, DD) \
            { const bool draw = nxt < NU && ENDS_GROUP(nxt); \
            asm volatile("s_mov_b64 exec, 1\n\tglobal_atomic_add v80, %0, %1, off sc0\n\ts_mov_b64 exec, -1" :: "v"(draw ? tkt : sink + 8), "v"(1u) : "memory"); \
            PUSH_LOAD(DC, nxt, 0); \
            PUSH_STORE(DA, 0); \
            asm volatile("s_waitcnt vmcnt(9)" ::: "memory", PUSH_REGS); \
            PUSH_FLAG(prev); \
            PUSH_LOAD(DD, nxt, 1); \
            PUSH_STORE(DB, 1); \
            unsigned tk; \
            asm volatile("s_waitcnt vmcnt(9)\n\tv_readfirstlane_b32 %0, v80" : "=s"(tk) :: "memory", PUSH_REGS); \
            const int g = (int)(tk - tbase); \
            prev = cur; cur = nxt; \
            nxt = draw ? (g < G1 ? 2 * g : G1 + g) : nxt + 1; \
            if (cur >= NU) break; }
        for (;;) {
            PUSH_UNIT(0, 1, 2, 3)
            PUSH_UNIT(2, 3, 0, 1)
        }
        asm volatile("s_waitcnt vmcnt(0)" ::: "memory", PUSH_REGS);
        PUSH_FLAG(prev);
        if (lane == 0) __hip_atomic_store(cnt, call, __ATOMIC_RELAXED, __HIP_MEMORY_SCOPE_AGENT);
        STAMP(63);
        return;
    }

    const int pi = rg - 1;
    const unsigned srank = (rank + 1u + (unsigned)pi) & (NR - 1);
    const int nloc = pi < 4 ? 5 : 4;
    const int l0 = pi < 4 ? 5 * pi : 4 * pi + 4;
    const int q = lane >> 4, n = lane & 15;
#define SB0 __builtin_amdgcn_sched_barrier(0)
#define WAITV(N) asm volatile("s_waitcnt vmcnt(%c0)" :: "i"(N) : "memory")
    typedef const __attribute__((address_space(4))) float* cfloat_p;
    const int scale_s = __builtin_amdgcn_readfirstlane(__builtin_bit_cast(int, *(cfloat_p)p.sx * *(cfloat_p)p.sw));

    const int lu = (pi * 32 + cg) * 4 + wid;
    unsigned call;
    const char* stage;
    {
        const __amdgpu_buffer_rsrc_t rx = mk_rsrc(p.x, lu < NLU ? (unsigned)SHBYTES : 0u);
        const __amdgpu_buffer_rsrc_t rw = mk_rsrc(p.w, (unsigned)KD * NL);
#define PRO_REGS "v128", "v129", "v130", "v131", "v132", "v133", "v134", "v135", "v136", "v137", "v138", "v139", "v140", "v141", "v142", "v143", "v144", "v145", "v146"
        asm volatile("global_load_dword v144, %0, off sc1\n\t"
                     "buffer_load_dwordx4 v[128:131], %1, %2, %3 offen\n\tbuffer_load_dwordx4 v[132:135], %1, %2, %3 offen offset:64\n\t"
                     "buffer_load_dwordx4 v[136:139], %1, %2, %3 offen offset:128\n\tbuffer_load_dwordx4 v[140:143], %1, %2, %3 offen offset:192"
                     :: "v"(cnt), "v"(n * KD + q * 16), "s"(rx), "s"(((lu * LU) >> 6) * IBYTES + ((lu * LU) & 63) * 64) : "memory", PRO_REGS);
        const int vw0 = (lane >> 2) * NL + (((lane & 3) ^ (q & 1)) * 16), vw1 = (lane >> 2) * NL + (((lane & 3) ^ (q & 1) ^ 2) * 16);
        const int sw = (wid * 1024) * NL + cg * 64;
        const unsigned lw = (unsigned)(size_t)(LAS char*)lds + wid * 32768;
        unsigned tr[4];
#pragma unroll
        for (int j = 0; j < 4; ++j) tr[j] = lw + (q * 16 + (n >> 1)) * 64 + ((j ^ ((n >> 3) | ((q & 1) << 1))) * 16) + (n & 1) * 8;
#define W_DMA(KS) do { _Pragma("unroll") for (int i_ = 0; i_ < 8; ++i_) \
            asm volatile("s_mov_b32 m0, %3\n\ts_nop 0\n\tbuffer_load_dwordx4 %0, %1, %2 offen lds" \
                         :: "v"((i_ & 1) ? vw1 : vw0), "s"(rw), "s"(sw + ((KS) * 128 + i_ * 16) * NL), "s"(lw + ((KS) & 3) * 8192 + i_ * 1024) : "memory"); } while (0)
#define W_TR(KS) do { _Pragma("unroll") for (int j_ = 0; j_ < 4; ++j_) \
            asm volatile("ds_read_b64_tr_b8 a[%c1:%c1+1], %0 offset:%c2\n\tds_read_b64_tr_b8 a[%c1+2:%c1+3], %0 offset:%c2+512\n\t" \
                         "ds_read_b64_tr_b8 a[%c1+4:%c1+5], %0 offset:%c2+4096\n\tds_read_b64_tr_b8 a[%c1+6:%c1+7], %0 offset:%c2+4608" \
                         :: "v"(tr[j_]), "i"((KS) * 32 + j_ * 8), "i"(((KS) & 3) * 8192) : "memory"); \
            asm volatile("s_waitcnt lgkmcnt(0)" ::: "memory"); } while (0)
        asm volatile("" ::: "a0", "a1", "a2", "a3", "a4", "a5", "a6", "a7", "a8", "a9", "a10", "a11", "a12", "a13", "a14", "a15", "a16", "a17", "a18", "a19", "a20", "a21", "a22", "a23", "a24", "a25", "a26", "a27", "a28", "a29", "a30", "a31", "a32", "a33", "a34", "a35", "a36", "a37", "a38", "a39", "a40", "a41", "a42", "a43", "a44", "a45", "a46", "a47", "a48", "a49", "a50", "a51", "a52", "a53", "a54", "a55", "a56", "a57", "a58", "a59", "a60", "a61", "a62", "a63", "a64", "a65", "a66", "a67", "a68", "a69", "a70", "a71", "a72", "a73", "a74", "a75", "a76", "a77", "a78", "a79", "a80", "a81", "a82", "a83", "a84", "a85", "a86", "a87", "a88", "a89", "a90", "a91", "a92", "a93", "a94", "a95", "a96", "a97", "a98", "a99", "a100", "a101", "a102", "a103", "a104", "a105", "a106", "a107", "a108", "a109", "a110", "a111", "a112", "a113", "a114", "a115", "a116", "a117", "a118", "a119", "a120", "a121", "a122", "a123", "a124", "a125", "a126", "a127", "a128", "a129", "a130", "a131", "a132", "a133", "a134", "a135", "a136", "a137", "a138", "a139", "a140", "a141", "a142", "a143", "a144", "a145", "a146", "a147", "a148", "a149", "a150", "a151", "a152", "a153", "a154", "a155", "a156", "a157", "a158", "a159", "a160", "a161", "a162", "a163", "a164", "a165", "a166", "a167", "a168", "a169", "a170", "a171", "a172", "a173", "a174", "a175", "a176", "a177", "a178", "a179", "a180", "a181", "a182", "a183", "a184", "a185", "a186", "a187", "a188", "a189", "a190", "a191", "a192", "a193", "a194", "a195", "a196", "a197", "a198", "a199", "a200", "a201", "a202", "a203", "a204", "a205", "a206", "a207", "a208", "a209", "a210", "a211", "a212", "a213", "a214", "a215", "a216", "a217", "a218", "a219", "a220", "a221", "a222", "a223", "a224", "a225", "a226", "a227", "a228", "a229", "a230", "a231", "a232", "a233", "a234", "a235", "a236", "a237", "a238", "a239", "a240", "a241", "a242", "a243", "a244", "a245", "a246", "a247", "a248", "a249", "a250", "a251", "a252", "a253", "a254", "a255");
        constexpr int LPR = BC / 2, NT = (KD * LPR + 112 * 64 - 1) / (112 * 64);
#pragma unroll
        for (int i = 0; i < NT; ++i) {
            const int li = ((((b >> 3) - 4) * 4 + wid) * NT + i) * 64 + lane;
            asm volatile("buffer_load_dword v145, %0, %1, 0 offen" :: "v"((li / LPR) * NL + (cg / BC) * (BC * 64) + (li % LPR) * 128), "s"(rw) : "memory", PRO_REGS);
        }
        W_DMA(0);
        asm volatile("s_waitcnt vmcnt(%c1)\n\tv_readfirstlane_b32 %0, v144" : "=s"(call) : "i"(8 + NT) : "memory", PRO_REGS);
        call += 1u;
        stage = myws + WS_STAGE + (size_t)(call & 1u) * (NR * SHBYTES);
        const __amdgpu_buffer_rsrc_t rl = mk_rsrc(stage + (size_t)rank * SHBYTES, lu < NLU ? (unsigned)SHBYTES : 0u);
        asm volatile("global_store_dword %0, %1, off sc1\n\t"
                     "buffer_store_dwordx4 v[128:131], %2, %3, %4 offen sc0 sc1\n\tbuffer_store_dwordx4 v[132:135], %2, %3, %4 offen offset:1024 sc0 sc1\n\t"
                     "buffer_store_dwordx4 v[136:139], %2, %3, %4 offen offset:2048 sc0 sc1\n\tbuffer_store_dwordx4 v[140:143], %2, %3, %4 offen offset:3072 sc0 sc1"
                     :: "v"(cnt), "v"(call), "v"(lane * 16), "s"(rl), "s"(lu * (LU * 1024)) : "memory", PRO_REGS);
        W_DMA(1); W_DMA(2); W_DMA(3);
        WAITV(24);
        if (lane == 0 && lu < NLU) __hip_atomic_store((GAS unsigned*)(myws + WS_LFLAG + lu * 128), call, __ATOMIC_RELAXED, __HIP_MEMORY_SCOPE_SYSTEM);
        STAMP(40); W_TR(0); W_DMA(4);
        WAITV(24); W_TR(1); W_DMA(5);
        WAITV(24); W_TR(2); W_DMA(6);
        WAITV(24); STAMP(41); W_TR(3); W_DMA(7);
        WAITV(24); W_TR(4);
        WAITV(16); W_TR(5);
        WAITV(8); W_TR(6);
        WAITV(0); W_TR(7);
    }
    STAMP(1);

    GAS unsigned* const tmo = (GAS unsigned*)(myws + WS_TMO);
    const __amdgpu_buffer_rsrc_t rs = mk_rsrc(stage, (unsigned)NR * SHBYTES);
    const char* const prog = myws + WS_PROG + (size_t)pi * 4096 + cg * 128;
    const int voff_a = lane * 16;
    const int voff_out = ((wid * 4 + q) * NL + n * 4) * 4;
    bool dead = false;
#define SPIN(READY_EXPR, LOADV) do { \
        const unsigned long long t0_ = __builtin_amdgcn_s_memrealtime(); \
        for (int spins_ = 1;; ++spins_) { \
            const unsigned v_ = LOADV; \
            if (READY_EXPR) break; \
            if ((spins_ & 15) == 0 && ((unsigned)__builtin_amdgcn_readfirstlane(__hip_atomic_load(tmo, __ATOMIC_RELAXED, __HIP_MEMORY_SCOPE_AGENT)) == call || \
                __builtin_amdgcn_s_memrealtime() - t0_ > DEADLINE_TICKS)) { \
                if (lane == 0) __hip_atomic_store(tmo, call, __ATOMIC_RELAXED, __HIP_MEMORY_SCOPE_AGENT); \
                dead = true; \
                rdy = NI; \
                break; \
            } \
            __builtin_amdgcn_s_sleep(1); \
        } } while (0)
#define SAMPLE() ({ unsigned sv_; asm volatile("s_load_dword %0, %1, 0x0 glc\n\ts_waitcnt lgkmcnt(0)" : "=&s"(sv_) : "s"(prog) : "memory"); sv_; })
#define ADVANCE(V) do { rdy = max(rdy, (int)((V) - call * 64u)); } while (0)
    int rdy = 0;
    {
        const __amdgpu_buffer_rsrc_t rfl = mk_rsrc(myws + WS_LFLAG, (unsigned)NLU * 128u);
        constexpr int LPI = 64 / LU;
        SPIN(__all(lane >= nloc * 4 || (int)(v_ - call) >= 0), __builtin_amdgcn_raw_buffer_load_b32(rfl, (((lane >> 2) + l0) * LPI + wid * 4 + (lane & 3)) * 128, 0, 17));
    }
    STAMP(42);

    const unsigned red = (unsigned)(size_t)(LAS char*)lds + WRING_BYTES;
    unsigned put[4];
#pragma unroll
    for (int j = 0; j < 4; ++j) put[j] = red + wid * 4096 + j * 1024 + q * 256 + ((n ^ (4 * j + q)) * 16);
    const unsigned get = red + (n >> 2) * 1024 + (n & 3) * 256 + (((4 * wid + q) ^ n) * 16);
    const __amdgpu_buffer_rsrc_t ro = mk_rsrc(p.out, (unsigned)NR * SH * NL * 4u);
    int tn = 1 - nloc, st = 1 + (nloc & 1);
    int sa = (int)rank * SHBYTES + l0 * IBYTES + wid * 16384;
    int sa1 = sa + IBYTES;
    int so2 = 0x7f000000;
#define DBG_A0 ""
#define DBG_A1 ""
#define DBG_A2 ""
#define DBG_AO
    unsigned long long tw = 0;
    int tws = NI + 1;
    for (;;) {
        unsigned smp;
        rdy = __builtin_amdgcn_readfirstlane(rdy);
        asm volatile(
        ".macro MF B, ks, j\n"
        " v_mfma_f32_16x16x128_f8f6f4 v[110+4*\\j:113+4*\\j], a[32*\\ks+8*\\j:32*\\ks+8*\\j+7], v[128+64*\\B+8*\\ks:135+64*\\B+8*\\ks], v[110+4*\\j:113+4*\\j]\n"
        ".endm\n"
        ".macro MF0 B, j\n"
        " v_mfma_f32_16x16x128_f8f6f4 v[110+4*\\j:113+4*\\j], a[8*\\j:8*\\j+7], v[128+64*\\B:135+64*\\B], v[74:77]\n"
        ".endm\n"
        ".macro LH B, ks, hf, sb\n"
        " buffer_load_dwordx4 v[128+64*(\\B)+8*(\\ks)+4*(\\hf):131+64*(\\B)+8*(\\ks)+4*(\\hf)], v[70+((\\ks)>>1)], %[rs], \\sb offen offset:((\\ks)&1)*2048+(\\hf)*1024 sc0 sc1\n"
        ".endm\n"
        ".macro V4 op, d, a, c\n"
        " \\op v[\\d], v[\\a], v[\\c]\n"
        " \\op v[\\d+1], v[\\a+1], v[\\c+1]\n"
        " \\op v[\\d+2], v[\\a+2], v[\\c+2]\n"
        " \\op v[\\d+3], v[\\a+3], v[\\c+3]\n"
        ".endm\n"
        ".macro K4 op, d, k\n"
        " \\op v[\\d], \\k, v[\\d]\n"
        " \\op v[\\d+1], \\k, v[\\d+1]\n"
        " \\op v[\\d+2], \\k, v[\\d+2]\n"
        " \\op v[\\d+3], \\k, v[\\d+3]\n"
        ".endm\n"
        ".macro U2 op, d\n"
        " \\op v[\\d], v[\\d]\n"
        " \\op v[\\d+1], v[\\d+1]\n"
        ".endm\n"
        ".macro CLAMP d, a\n"
        " v_med3_f32 v[\\d], v[\\a], %[m60], %[c60]\n"
        " v_med3_f32 v[\\d+1], v[\\a+1], %[m60], %[c60]\n"
        " v_med3_f32 v[\\d+2], v[\\a+2], %[m60], %[c60]\n"
        " v_med3_f32 v[\\d+3], v[\\a+3], %[m60], %[c60]\n"
        ".endm\n"
        ".macro GET4 d, B\n"
        " ds_read_b128 v[\\d:\\d+3], %[get] offset:(\\B)*16384\n"
        " ds_read_b128 v[\\d+4:\\d+7], %[get] offset:(\\B)*16384+4096\n"
        " ds_read_b128 v[\\d+8:\\d+11], %[get] offset:(\\B)*16384+8192\n"
        " ds_read_b128 v[\\d+12:\\d+15], %[get] offset:(\\B)*16384+12288\n"
        ".endm\n"
        ".macro SAMPLE\n"
        " s_load_dword %[smp], %[prog], 0x0 glc\n"
        ".endm\n"
        ".macro TAKE\n"
        " s_sub_i32 %[smp], %[smp], %[c64]\n"
        " s_max_i32 %[rdy], %[rdy], %[smp]\n"
        ".endm\n"
        ".macro REQ1 B, H, K, hf\n"
        " .if \\H == 0\n"
        "  LH 1-\\B, \\K, \\hf, %[sa1]\n"
        " .endif\n"
        " .if \\H == 1 && \\K < 2\n"
        "  LH \\B, \\K+6, \\hf, %[sa]\n"
        " .endif\n"
        " .if \\hf == 1\n"
        "  .if \\H == 0\n"
        "   s_waitcnt vmcnt(14)\n"
        "  .elseif \\H == 1 && \\K < 2\n"
        "   s_waitcnt vmcnt(10)\n"
        "  .else\n"
        "   s_waitcnt vmcnt(12-2*\\K)\n"
        "  .endif\n"
        " .endif\n"
        ".endm\n"
        ".macro REQ2 B, H, K, hf\n"
        " .if \\H == 0\n"
        "  LH 1-\\B, \\K, \\hf, %[sa1]\n"
        " .endif\n"
        " .if \\H == 1 && \\K >= 6\n"
        "  LH 1-\\B, \\K-2, \\hf, %[sa1]\n"
        " .endif\n"
        " .if \\H == 3 || (\\H == 1 && \\K < 6)\n"
        "  LH 1-\\B, 2*\\K-8+\\hf, 0, %[sa1]\n"
        "  LH 1-\\B, 2*\\K-8+\\hf, 1, %[sa1]\n"
        " .endif\n"
        " .if \\hf == 1 && \\K < 7\n"
        "  .if \\H == 0\n"
        "   s_waitcnt vmcnt(14+\\K/6)\n"
        "  .elseif \\H == 1 && \\K == 6\n"
        "   s_waitcnt vmcnt(11)\n"
        "  .elseif \\H == 2\n"
        "   s_waitcnt vmcnt(12-2*\\K+\\K/6)\n"
        "  .else\n"
        "   s_waitcnt vmcnt(2*\\K+\\K/6)\n"
        "  .endif\n"
        " .endif\n"
        ".endm\n"
        ".macro HALF1 B, H\n" DBG_A1
        " ds_write_b128 %[put0], v[110:113] offset:(1-\\B)*16384\n"
        " .if \\H == 1\n"
        "  s_waitcnt vmcnt(10)\n"
        " .else\n"
        "  s_waitcnt vmcnt(14)\n"
        " .endif\n"
        " MF0 \\B, 0\n"
        " ds_write_b128 %[put1], v[114:117] offset:(1-\\B)*16384\n"
        " REQ1 \\B, \\H, 0, 0\n"
        " MF0 \\B, 1\n"
        " ds_write_b128 %[put2], v[118:121] offset:(1-\\B)*16384\n"
        " MF0 \\B, 2\n"
        " ds_write_b128 %[put3], v[122:125] offset:(1-\\B)*16384\n"
        " MF0 \\B, 3\n"
        " REQ1 \\B, \\H, 0, 1\n"
        " MF \\B, 1, 0\n"
        " .if \\H != 0\n"
        "  SAMPLE\n"
        " .endif\n"
        " REQ1 \\B, \\H, 1, 0\n"
        " MF \\B, 1, 1\n"
        " V4 v_add_f32, 78, 78, 82\n"
        " MF \\B, 1, 2\n"
        " V4 v_add_f32, 86, 86, 90\n"
        " MF \\B, 1, 3\n"
        " REQ1 \\B, \\H, 1, 1\n"
        " MF \\B, 2, 0\n"
        " REQ1 \\B, \\H, 2, 0\n"
        " .if \\H == 0\n"
        "  s_waitcnt lgkmcnt(0)\n"
        "  SAMPLE\n"
        " .endif\n"
        " MF \\B, 2, 1\n"
        " V4 v_add_f32, 78, 78, 86\n"
        " MF \\B, 2, 2\n"
        " K4 v_mul_f32, 78, %[scale]\n"
        " MF \\B, 2, 3\n"
        " REQ1 \\B, \\H, 2, 1\n"
        " MF \\B, 3, 0\n"
        " REQ1 \\B, \\H, 3, 0\n"
        " MF \\B, 3, 1\n"
        " CLAMP 82, 78\n"
        " MF \\B, 3, 2\n"
        " K4 v_mul_f32, 82, %[nl2e]\n"
        " MF \\B, 3, 3\n"
        " REQ1 \\B, \\H, 3, 1\n"
        " .if \\H != 0\n"
        "  s_waitcnt lgkmcnt(0)\n"
        "  TAKE\n"
        " .endif\n"
        " MF \\B, 4, 0\n"
        " s_barrier\n"
        ".endm\n"
        ".macro HALF2 B, H\n"
        " .if \\H != 0\n"
        "  SAMPLE\n"
        " .endif\n"
        " REQ2 \\B, \\H, 4, 0\n"
        " MF \\B, 4, 1\n"
        " U2 v_exp_f32, 82\n"
        " MF \\B, 4, 2\n"
        " U2 v_exp_f32, 84\n"
        " MF \\B, 4, 3\n"
        " REQ2 \\B, \\H, 4, 1\n"
        " MF \\B, 5, 0\n"
        " REQ2 \\B, \\H, 5, 0\n"
        " K4 v_add_f32, 82, 1.0\n"
        " MF \\B, 5, 1\n"
        " U2 v_rcp_f32, 82\n"
        " MF \\B, 5, 2\n"
        " U2 v_rcp_f32, 84\n"
        " MF \\B, 5, 3\n"
        " REQ2 \\B, \\H, 5, 1\n"
        " MF \\B, 6, 0\n"
        " REQ2 \\B, \\H, 6, 0\n"
        " V4 v_mul_f32, 78, 78, 82\n"
        " MF \\B, 6, 1\n"
        " buffer_store_dwordx4 v[78:81], %[vout], %[ro], %[so2] offen sc1\n"
        " MF \\B, 6, 2\n"
        " s_add_i32 %[so2], %[so2], 0x20000\n"
        " s_cmp_eq_u32 %[tn], %[tA]\n"
        " s_cselect_b32 %[so2], %[ol], %[so2]\n"
        " s_cmp_eq_u32 %[tn], 2\n"
        " s_cselect_b32 %[so2], %[orm], %[so2]\n"
        " MF \\B, 6, 3\n"
        " REQ2 \\B, \\H, 6, 1\n"
        " MF \\B, 7, 0\n"
        " REQ2 \\B, \\H, 7, 0\n"
        " GET4 78, 1-\\B\n"
        " MF \\B, 7, 1\n"
        " REQ2 \\B, \\H, 7, 1\n"
        " s_mov_b32 %[sa], %[sa1]\n"
        " s_add_i32 %[sa1], %[sa1], 0x10000\n"
        " s_add_i32 %[tn], %[tn], 1\n"
        " MF \\B, 7, 2\n"
        " s_cmp_eq_u32 %[tn], 0\n"
        " s_cselect_b32 %[sa1], %[sr], %[sa1]\n"
        " s_waitcnt lgkmcnt(0)\n"
        " TAKE\n"
        " s_cmp_lt_i32 %[tn], %[rdy]\n"
        " MF \\B, 7, 3\n"
        ".endm\n"
        ".macro BSIDE B\n"
        "Lcold\\B\\()_%=:\n"
        " s_cmp_le_i32 %[tn], %[rdy]\n"
        " s_cbranch_scc1 Lgo\\B\\()_%=\n"
        " s_movk_i32 %[st], 64\n"
        "Lspin\\B\\()_%=:\n"
        " SAMPLE\n"
        " s_waitcnt lgkmcnt(0)\n"
        " TAKE\n"
        " s_cmp_le_i32 %[tn], %[rdy]\n"
        " s_cbranch_scc1 Lgo\\B\\()_%=\n"
        " s_sub_u32 %[st], %[st], 1\n"
        " s_cbranch_scc0 Lspin\\B\\()_%=\n"
        " s_movk_i32 %[st], 1+\\B\n"
        " s_branch Lexit_%=\n"
        "Lgo\\B\\()_%=:\n" DBG_A0
        " LH \\B, 0, 0, %[sa]\n"
        " LH \\B, 0, 1, %[sa]\n"
        " LH \\B, 1, 0, %[sa]\n"
        " LH \\B, 1, 1, %[sa]\n"
        " LH \\B, 2, 0, %[sa]\n"
        " LH \\B, 2, 1, %[sa]\n"
        " LH \\B, 3, 0, %[sa]\n"
        " LH \\B, 3, 1, %[sa]\n"
        " LH \\B, 4, 0, %[sa]\n"
        " LH \\B, 4, 1, %[sa]\n"
        " LH \\B, 5, 0, %[sa]\n"
        " LH \\B, 5, 1, %[sa]\n"
        "Lb\\B\\()_%=:\n"
        " HALF1 \\B, 1\n"
        "Lmid\\B\\()_%=:\n"
        " s_cmp_ge_i32 %[tn], %[rdy]\n"
        " s_cbranch_scc1 Lc2\\B\\()_%=\n"
        " s_add_i32 %[st], %[tn], " UPG_STR "\n"
        " s_cmp_lt_i32 %[st], %[rdy]\n"
        " s_cbranch_scc1 Ld2\\B\\()_%=\n"
        " HALF2 \\B, 1\n"
        ".endm\n"
        ".macro OTHER B, N\n"
        "Lc1\\B\\()_%=:\n"
        " HALF1 \\B, 2\n"
        " s_branch Lmid\\B\\()_%=\n"
        "Lc2\\B\\()_%=:\n"
        " .if \\B == 1\n"
        "  s_cmp_eq_u32 %[tn], 32\n"
        "  s_cbranch_scc1 Lfin_%=\n"
        " .endif\n"
        " HALF2 \\B, 2\n"
        " s_branch Lcold\\N\\()_%=\n"
        "Ld2\\B\\()_%=:\n"
        " HALF2 \\B, 3\n"
        " s_cbranch_scc1 La\\N\\()_%=\n"
        " s_branch Lc1\\N\\()_%=\n"
        ".endm\n"
        " s_nop 4\n"
        " v_mov_b32 v74, %[cin]\n"
        " v_mov_b32 v75, %[cin]\n"
        " v_mov_b32 v76, %[cin]\n"
        " v_mov_b32 v77, %[cin]\n"
        " v_mov_b32 v70, %[va]\n"
        " v_add_u32 v71, 0x1000, %[va]\n"
        " v_add_u32 v72, 0x2000, %[va]\n"
        " v_add_u32 v73, 0x3000, %[va]\n"
        " s_cmp_eq_u32 %[st], 2\n"
        " s_cbranch_scc1 Lcold1_%=\n"
        " BSIDE 0\n"
        " s_branch Lb1_%=\n"
        " BSIDE 1\n"
        " s_branch Lb0_%=\n"
        "La0_%=:\n"
        " HALF1 0, 0\n"
        " HALF2 0, 0\n"
        " s_cbranch_scc0 Lc11_%=\n"
        "La1_%=:\n"
        " HALF1 1, 0\n"
        " HALF2 1, 0\n"
        " s_cbranch_scc1 La0_%=\n"
        " OTHER 0, 1\n"
        " OTHER 1, 0\n"
        "Lfin_%=:\n" DBG_A2
        " GET4 94, 0\n"
        " MF 1, 4, 1\n"
        " U2 v_exp_f32, 82\n"
        " MF 1, 4, 2\n"
        " U2 v_exp_f32, 84\n"
        " MF 1, 4, 3\n"
        " s_waitcnt vmcnt(4)\n"
        " MF 1, 5, 0\n"
        " K4 v_add_f32, 82, 1.0\n"
        " MF 1, 5, 1\n"
        " U2 v_rcp_f32, 82\n"
        " MF 1, 5, 2\n"
        " U2 v_rcp_f32, 84\n"
        " MF 1, 5, 3\n"
        " s_waitcnt vmcnt(0) lgkmcnt(0)\n"
        " MF 1, 6, 0\n"
        " V4 v_mul_f32, 78, 78, 82\n"
        " MF 1, 7, 0\n"
        " buffer_store_dwordx4 v[78:81], %[vout], %[ro], %[so2] offen sc1\n"
        " V4 v_add_f32, 94, 94, 98\n"
        " MF 1, 6, 1\n"
        " V4 v_add_f32, 102, 102, 106\n"
        " MF 1, 7, 1\n"
        " V4 v_add_f32, 94, 94, 102\n"
        " MF 1, 6, 2\n"
        " ds_write_b128 %[put0], v[110:113] offset:16384\n"
        " K4 v_mul_f32, 94, %[scale]\n"
        " MF 1, 7, 2\n"
        " CLAMP 98, 94\n"
        " MF 1, 6, 3\n"
        " ds_write_b128 %[put1], v[114:117] offset:16384\n"
        " K4 v_mul_f32, 98, %[nl2e]\n"
        " MF 1, 7, 3\n"
        " U2 v_exp_f32, 98\n"
        " U2 v_exp_f32, 100\n"
        " ds_write_b128 %[put2], v[118:121] offset:16384\n"
        " K4 v_add_f32, 98, 1.0\n"
        " U2 v_rcp_f32, 98\n"
        " U2 v_rcp_f32, 100\n"
        " ds_write_b128 %[put3], v[122:125] offset:16384\n"
        " s_add_i32 %[so2], %[so2], 0x20000\n"
        " V4 v_mul_f32, 94, 94, 98\n"
        " buffer_store_dwordx4 v[94:97], %[vout], %[ro], %[so2] offen sc1\n"
        " s_waitcnt lgkmcnt(0)\n"
        " s_barrier\n"
        " GET4 78, 1\n"
        " s_add_i32 %[so2], %[so2], 0x20000\n"
        " s_waitcnt lgkmcnt(2)\n"
        " V4 v_add_f32, 78, 78, 82\n"
        " s_waitcnt lgkmcnt(0)\n"
        " V4 v_add_f32, 86, 86, 90\n"
        " V4 v_add_f32, 78, 78, 86\n"
        " K4 v_mul_f32, 78, %[scale]\n"
        " CLAMP 82, 78\n"
        " K4 v_mul_f32, 82, %[nl2e]\n"
        " U2 v_exp_f32, 82\n"
        " U2 v_exp_f32, 84\n"
        " s_mov_b32 %[st], 0\n"
        " K4 v_add_f32, 82, 1.0\n"
        " U2 v_rcp_f32, 82\n"
        " U2 v_rcp_f32, 84\n"
        " s_nop 0\n"
        " V4 v_mul_f32, 78, 78, 82\n"
        " buffer_store_dwordx4 v[78:81], %[vout], %[ro], %[so2] offen sc1\n"
        "Lexit_%=:\n"
        ".purgem MF\n"
        ".purgem MF0\n"
        ".purgem LH\n"
        ".purgem V4\n"
        ".purgem K4\n"
        ".purgem U2\n"
        ".purgem CLAMP\n"
        ".purgem GET4\n"
        ".purgem SAMPLE\n"
        ".purgem TAKE\n"
        ".purgem REQ1\n"
        ".purgem REQ2\n"
        ".purgem HALF1\n"
        ".purgem HALF2\n"
        ".purgem BSIDE\n"
        ".purgem OTHER\n"
            : [tn] "+s"(tn), [rdy] "+s"(rdy), [sa] "+s"(sa), [sa1] "+s"(sa1), [so2] "+s"(so2), [st] "+s"(st), [smp] "=&s"(smp) DBG_AO
            : [rs] "s"(rs), [ro] "s"(ro), [prog] "s"(prog), [c64] "s"(call * 64u), [scale] "s"(scale_s), [m60] "s"(-60.0f),
              [nl2e] "s"(-1.44269504088896340736f), [tA] "s"(2 - nloc), [ol] "s"((((int)rank * SH + l0 * IROWS) * NL + cg * 64) * 4),
              [orm] "s"((((int)srank * SH) * NL + cg * 64) * 4), [sr] "s"((int)srank * SHBYTES + wid * 16384), [cin] "s"(__builtin_amdgcn_readfirstlane(dead ? 0x7fc00000 : 0)),
              [va] "v"(voff_a), [vout] "v"(voff_out), [c60] "v"(60.0f), [get] "v"(get), [put0] "v"(put[0]), [put1] "v"(put[1]), [put2] "v"(put[2]), [put3] "v"(put[3])
            : "memory", "scc", "v70", "v71", "v72", "v73", "v74", "v75", "v76", "v77", "v78", "v79", "v80", "v81", "v82", "v83", "v84", "v85", "v86", "v87", "v88", "v89", "v90", "v91", "v92", "v93", "v94", "v95", "v96", "v97", "v98", "v99", "v100", "v101", "v102", "v103", "v104", "v105", "v106", "v107", "v108", "v109", "v110", "v111", "v112", "v113", "v114", "v115", "v116", "v117", "v118", "v119", "v120", "v121", "v122", "v123", "v124", "v125", "v126", "v127", "v128", "v129", "v130", "v131", "v132", "v133", "v134", "v135", "v136", "v137", "v138", "v139", "v140", "v141", "v142", "v143", "v144", "v145", "v146", "v147", "v148", "v149", "v150", "v151", "v152", "v153", "v154", "v155", "v156", "v157", "v158", "v159", "v160", "v161", "v162", "v163", "v164", "v165", "v166", "v167", "v168", "v169", "v170", "v171", "v172", "v173", "v174", "v175", "v176", "v177", "v178", "v179", "v180", "v181", "v182", "v183", "v184", "v185", "v186", "v187", "v188", "v189", "v190", "v191", "v192", "v193", "v194", "v195", "v196", "v197", "v198", "v199", "v200", "v201", "v202", "v203", "v204", "v205", "v206", "v207", "v208", "v209", "v210", "v211", "v212", "v213", "v214", "v215", "v216", "v217", "v218", "v219", "v220", "v221", "v222", "v223", "v224", "v225", "v226", "v227", "v228", "v229", "v230", "v231", "v232", "v233", "v234", "v235", "v236", "v237", "v238", "v239", "v240", "v241", "v242", "v243", "v244", "v245", "v246", "v247", "v248", "v249", "v250", "v251", "v252", "v253", "v254", "v255");
        if (st == 0) break;
        const unsigned long long now = __builtin_amdgcn_s_memrealtime();
        if (tws != tn) { tws = tn; tw = now; }
        if ((unsigned)__builtin_amdgcn_readfirstlane(__hip_atomic_load(tmo, __ATOMIC_RELAXED, __HIP_MEMORY_SCOPE_AGENT)) == call || now - tw > DEADLINE_TICKS) {
            if (lane == 0) __hip_atomic_store(tmo, call, __ATOMIC_RELAXED, __HIP_MEMORY_SCOPE_AGENT);
            dead = true;
            rdy = NI;
        }
    }
    STAMP(63);
}

}

extern "C" void kernel_launch(int rank, int nranks, ncclComm_t comm,
                              hipStream_t stream,
                              void* const* d_in, const int* in_sizes, int n_in,
                              void* d_out, int out_size,
                              const int* in_dtypes, int out_dtype,
                              void* const* d_out_all,
                              void* const* d_ws_all, size_t ws_bytes) {
    (void)comm; (void)in_dtypes; (void)out_dtype; (void)d_out_all;
    if (nranks != NR || rank < 0 || rank >= NR || n_in < 4 || in_sizes[0] != SH * KD || in_sizes[1] != KD * NL ||
        out_size != NR * SH * NL || ws_bytes < WS_NEED) {
        fprintf(stderr, "ag_gemm: unexpected problem shape\n");
        return;
    }
    Params p;
    for (int i = 0; i < NR; ++i) p.ws[i] = (char*)d_ws_all[(rank + i) % NR];
    p.x = (const char*)d_in[0];
    p.w = (const char*)d_in[1];
    p.sx = (const float*)d_in[2];
    p.sw = (const float*)d_in[3];
    p.out = (float*)d_out;
    p.rank = rank;
    p.pad = 0;
    ag_gemm_kernel<<<dim3(NWG), dim3(256), 0, stream>>>(p);
    HIP_CHECK(hipGetLastError());
}
```

```cpp
#include <hip/hip_runtime.h>
#include <rccl/rccl.h>
#include <cstdio>

#define HIP_CHECK(expr) do { hipError_t _e = (expr); if (_e != hipSuccess) { \
    fprintf(stderr, "HIP error %s at %s:%d\n", hipGetErrorString(_e), __FILE__, __LINE__); \
    return; } } while (0)

namespace {

typedef int v8i __attribute__((ext_vector_type(8)));
typedef unsigned u32x4 __attribute__((ext_vector_type(4)));
typedef unsigned u32x8 __attribute__((ext_vector_type(8)));
typedef float v4f __attribute__((ext_vector_type(4)));
typedef short short2v __attribute__((ext_vector_type(2)));
#define LAS __attribute__((address_space(3)))
#define GAS __attribute__((address_space(1)))

constexpr int NR = 8;
constexpr int KD = 4096;
constexpr int NL = 2048;
constexpr int SH = 512;
constexpr int IROWS = 16;
constexpr int NI = SH / IROWS;
constexpr int IBYTES = IROWS * KD;
constexpr int SHBYTES = SH * KD;
constexpr int NP = 32;
#define NWATCH 2
constexpr int NWG = 256;
#define BC 8
#define A_AUX 17
#define PUSH_BITS "sc0 sc1"
#define OUT_BITS "sc1"
#define NO_STORE_T 2
#define W_TOUCH 1
#define PUSH_FIRST 1
#define UPG_STR "3"
#define ABL 0
#define X_VALU(x) do { if (!(ABL & 1)) { x; } } while (0)
#define X_LDS(x) do { if (!(ABL & 2)) { x; } } while (0)
#define X_STORE(x) do { if (!(ABL & 4)) { x; } } while (0)
#define X_SAMPLE(x) do { if (!(ABL & 8)) { x; } } while (0)
#define X_LOAD(x) do { if (!(ABL & 16)) { x; } } while (0)
#define X_BAR "s_barrier"
#define I1 28
#define NPX 0
constexpr int NPU = NP - NWATCH;
constexpr int NPW = NPU + NPX;
constexpr int NU = SHBYTES / 2048;
constexpr int G1 = 16 * I1;
constexpr int NGR = G1 + 32 * (NI - I1);
#define RELAY 1
#define RDIV 54
#define GA_ 160
#define RCAP_ 24
#define ROFF 130
constexpr int GA = GA_;
constexpr int RCAP = RCAP_;
constexpr int NH = 84;

constexpr size_t WS_CNT = 0;
constexpr size_t WS_TMO = 32768;
constexpr size_t WS_TKT = 32768 + 256;
constexpr size_t WS_PLAN = 45056;
constexpr size_t WS_BEACON = 36864;
constexpr size_t WS_REQ = 40960;
constexpr size_t WS_PROG = 131072;
constexpr size_t WS_FLAG = 65536;
constexpr size_t WS_LFLAG = 262144;
constexpr size_t WS_DUMMY = 49152;
constexpr size_t WS_STAGE = 2u << 20;
constexpr size_t WS_NEED = WS_STAGE + 2ull * NR * SHBYTES;
constexpr size_t WS_DBG = 64u << 20;
constexpr unsigned long long DEADLINE_TICKS = 40000000ull;
static_assert(WS_FLAG + (size_t)NR * 4096 <= WS_PROG && NGR <= 1024 && NPW <= G1 && NPW <= 64 && WS_PROG + 224 * 128 <= WS_LFLAG && WS_LFLAG + 512 * 128 <= WS_STAGE, "flags");
constexpr int LU = 4;
constexpr int NLU = SHBYTES / 1024 / LU;
static_assert(NPW <= GA && GA + 7 * RCAP <= G1 && NH <= 3 * NPU && RCAP < 64 && NWATCH == 2 && NPX == 0, "relay");
static_assert(NLU <= (NWG - NP) * 4, "one local unit per consumer wave");

struct Params {
    char* ws[NR];
    const char* x;
    const char* w;
    const float* sx;
    const float* sw;
    float* out;
    int rank;
    int pad;
};

#define STAMP(i) do {} while (0)
#define STAMPF(i) do {} while (0)

__device__ __forceinline__ __amdgpu_buffer_rsrc_t mk_rsrc(const void* base, unsigned bytes) {
    return __builtin_amdgcn_make_buffer_rsrc((void*)base, 0, bytes, 0x00020000);
}

__device__ __forceinline__ float silu_dq(float acc, float s) {
    const float y = acc * s;
    const float c = __builtin_amdgcn_fmed3f(y, -60.0f, 60.0f);
    const float e = __builtin_amdgcn_exp2f(c * -1.44269504088896340736f);
    return y * __builtin_amdgcn_rcpf(1.0f + e);
}

constexpr int WRING_BYTES = 4 * 4 * 8192;
constexpr int RED_BYTES = 4 * 4096;

__global__ __launch_bounds__(256, 1) void ag_gemm_kernel(const Params p) {
    __shared__ __attribute__((aligned(16))) char lds[WRING_BYTES + 2 * RED_BYTES];
    const int tid = threadIdx.x;
    const int lane = tid & 63;
    const int wid = __builtin_amdgcn_readfirstlane(tid >> 6);
    const int b = blockIdx.x;
    const unsigned rank = (unsigned)p.rank;
    char* const myws = p.ws[0];
    asm volatile("" :: "s"(p.ws[0]), "s"(p.ws[7]), "s"(p.x), "s"(p.w), "s"(p.out), "s"(p.rank));

    GAS unsigned* cnt = (GAS unsigned*)(myws + WS_CNT + (size_t)b * 128 + wid * 4);

    int rg = 0, cg = b;
    if (b >= NP) {
        const int i = (b & 7) * 28 + (b >> 3) - 4;
        rg = 1 + (i / BC) % 7;
        cg = (i / (7 * BC)) * BC + i % BC;
    }
    STAMP(0);
    if (__builtin_expect(rg == 0, PUSH_FIRST)) {
        if (wid > (cg < NPX ? 1 : 0) || cg >= NPU) {
#define SLOAD(PTR) ({ unsigned sv_; asm volatile("s_load_dword %0, %1, 0x0 glc\n\ts_waitcnt lgkmcnt(0)" : "=&s"(sv_) : "s"(PTR) : "memory"); sv_; })
            const unsigned t0 = (unsigned)__builtin_amdgcn_s_memrealtime();
            const unsigned call = SLOAD(cnt) + 1u;
            if (lane == 0) __hip_atomic_store(cnt, call, __ATOMIC_RELAXED, __HIP_MEMORY_SCOPE_AGENT);
            const int pw = (cg - NPU) * 4 + wid;
            GAS unsigned* const tmo_ = (GAS unsigned*)(myws + WS_TMO);
            const unsigned tag = (call & 0xffffu) << 16, par1 = call & 1u;
            char* pws = p.ws[1];
#pragma unroll
            for (int i = 2; i < NR; ++i) pws = lane == i - 1 ? p.ws[i] : pws;
#define WAIT_FOR(COND) ({ bool ok_ = true; const unsigned long long w0_ = __builtin_amdgcn_s_memrealtime(); \
                for (int sp_ = 1; !(COND); ++sp_) { \
                    if ((sp_ & 15) == 0 && ((unsigned)__builtin_amdgcn_readfirstlane(__hip_atomic_load(tmo_, __ATOMIC_RELAXED, __HIP_MEMORY_SCOPE_AGENT)) == call || \
                                            __builtin_amdgcn_s_memrealtime() - w0_ > DEADLINE_TICKS)) { \
                        if (lane == 0) __hip_atomic_store(tmo_, call, __ATOMIC_RELAXED, __HIP_MEMORY_SCOPE_AGENT); \
                        ok_ = false; break; } \
                    __builtin_amdgcn_s_sleep(4); } ok_; })
            if (pw == NR - 1) {
                GAS unsigned* const bout = (GAS unsigned*)(pws + WS_BEACON + (par1 * NR + rank) * 128);
                const unsigned e0 = (unsigned)__builtin_amdgcn_s_memrealtime() - t0;
                unsigned bv = __hip_atomic_load((GAS unsigned*)(myws + WS_BEACON + (par1 * NR + ((rank + 1u + lane) & (NR - 1))) * 128), __ATOMIC_RELAXED, __HIP_MEMORY_SCOPE_SYSTEM);
                if (lane < NR - 1) __hip_atomic_store(bout, tag | (e0 & 0xffffu), __ATOMIC_RELAXED, __HIP_MEMORY_SCOPE_SYSTEM);
                asm volatile("" : "+v"(bv));
                const unsigned e1 = (unsigned)__builtin_amdgcn_s_memrealtime() - t0;
                if (lane < NR - 1) __hip_atomic_store(bout, tag | (e1 & 0xffffu), __ATOMIC_RELAXED, __HIP_MEMORY_SCOPE_SYSTEM);
                const int mine = (int)(e0 + e1) / 2;
                const int m = RELAY && lane < NR - 1 && (bv & 0xffff0000u) == tag ? min(max((int)(bv & 0xffffu) + ROFF - mine, 0) / RDIV, RCAP) : 0;
                int st = GA;
#pragma unroll
                for (int i = 0; i < NR - 1; ++i) st += lane > i ? __builtin_amdgcn_readlane(m, i) : 0;
                const unsigned rq = tag | (unsigned)(lane < NR - 1 ? st << 6 | m : st - GA);
                if (lane < NR - 1) __hip_atomic_store((GAS unsigned*)(pws + WS_REQ + (par1 * NR + rank) * 128), rq, __ATOMIC_RELAXED, __HIP_MEMORY_SCOPE_SYSTEM);
                const unsigned rq8 = (unsigned)__shfl((int)rq, lane & 7);
#pragma unroll
                for (int i = 0; i < 4; ++i) __hip_atomic_store((GAS unsigned*)(myws + WS_PLAN + (i * 8 + (lane >> 3)) * 128) + (lane & 7), rq8, __ATOMIC_RELAXED, __HIP_MEMORY_SCOPE_AGENT);
                if (lane == 0) __hip_atomic_store((GAS unsigned*)(myws + WS_TKT + (par1 ^ 1u) * 128), 0u, __ATOMIC_RELAXED, __HIP_MEMORY_SCOPE_AGENT);
                STAMP(14);
                for (unsigned e = 0; e < (unsigned)(RCAP * RDIV + 200); ) {
                    e = (unsigned)__builtin_amdgcn_s_memrealtime() - t0;
                    if (lane < NR - 1) __hip_atomic_store(bout, tag | min(e, 0xffffu), __ATOMIC_RELAXED, __HIP_MEMORY_SCOPE_SYSTEM);
                    __builtin_amdgcn_s_sleep(8);
                }
            } else if (pw < 0) {
                const int h = cg * 3 + wid - 1;
                if (h >= NH) return;
                const size_t soff = WS_STAGE + (size_t)par1 * (NR * SHBYTES) + lane * 16;
                u32x4 d[4];
                u32x8 pl;
                if (!WAIT_FOR(({ asm volatile("s_load_dwordx8 %0, %1, 0x0 glc\n\ts_waitcnt lgkmcnt(0)" : "=&s"(pl) : "s"(myws + WS_PLAN + cg * 128) : "memory");
                                (((pl[0] ^ tag) | (pl[1] ^ tag) | (pl[2] ^ tag) | (pl[3] ^ tag) | (pl[4] ^ tag) | (pl[5] ^ tag) | (pl[6] ^ tag) | (pl[7] ^ tag)) >> 16) == 0u; }))) return;
                const unsigned pm = pl[7];
                unsigned rq = pl[0];
#pragma unroll
                for (int i = 1; i < NR - 1; ++i) rq = lane == i ? pl[i] : rq;
                STAMP(10);
                for (int g = GA + h; g < GA + (int)(pm & 0xffffu); g += NH) {
                    const int to = __builtin_ctzll(__ballot(lane < NR - 1 && (unsigned)(g - (int)((rq >> 6) & 1023u)) < (rq & 63u)) | 128ull);
                    if (to >= NR - 1 || g >= G1) return;
                    char* const tw = (char*)(((size_t)(unsigned)__builtin_amdgcn_readlane((int)((size_t)pws >> 32), to) << 32) | (unsigned)__builtin_amdgcn_readlane((int)(size_t)pws, to));
#pragma unroll
                    for (int i = 0; i < 4; ++i) d[i] = *(const GAS u32x4*)(p.x + (size_t)((g >> 4) * IROWS + (lane & 15)) * KD + ((g & 15) * 4 + i) * 64 + (lane >> 4) * 16);
                    asm volatile("global_store_dwordx4 %4, %0, off sc0 sc1\n\tglobal_store_dwordx4 %4, %1, off offset:1024 sc0 sc1\n\t"
                                 "global_store_dwordx4 %4, %2, off offset:2048 sc0 sc1\n\tglobal_store_dwordx4 %4, %3, off offset:3072 sc0 sc1\n\ts_waitcnt vmcnt(0)"
                                 :: "v"(d[0]), "v"(d[1]), "v"(d[2]), "v"(d[3]), "v"(tw + soff + (size_t)rank * SHBYTES + (size_t)g * 4096) : "memory");
                    if (lane == 0) __hip_atomic_store((GAS unsigned*)(tw + WS_FLAG) + rank * 1024 + g, call, __ATOMIC_RELAXED, __HIP_MEMORY_SCOPE_SYSTEM);
                    STAMP(11);
                }
                const int si = h % (NR - 1);
                const unsigned src = (rank + 1u + (unsigned)si) & (NR - 1);
                if (!WAIT_FOR(((rq = SLOAD(myws + WS_REQ + (par1 * NR + src) * 128)) & 0xffff0000u) == tag)) return;
                for (int k = h / (NR - 1); k < (int)(rq & 63u); k += NH / (NR - 1)) {
                    const int g = (int)((rq >> 6) & 1023u) + k;
                    if (g >= G1) return;
                    if (!WAIT_FOR((int)((unsigned)__builtin_amdgcn_readfirstlane(__hip_atomic_load((GAS unsigned*)(myws + WS_FLAG) + src * 1024 + g, __ATOMIC_RELAXED, __HIP_MEMORY_SCOPE_SYSTEM)) - call) >= 0)) return;
                    STAMP(12);
                    const size_t go = soff + (size_t)src * SHBYTES + (size_t)g * 4096;
                    asm volatile("global_load_dwordx4 %0, %4, off sc0 sc1\n\tglobal_load_dwordx4 %1, %4, off offset:1024 sc0 sc1\n\t"
                                 "global_load_dwordx4 %2, %4, off offset:2048 sc0 sc1\n\tglobal_load_dwordx4 %3, %4, off offset:3072 sc0 sc1\n\ts_waitcnt vmcnt(0)"
                                 : "=&v"(d[0]), "=&v"(d[1]), "=&v"(d[2]), "=&v"(d[3]) : "v"(myws + go) : "memory");
#pragma unroll
                    for (int pr = 0; pr < NR - 1; ++pr)
                        if (pr != si)
                            asm volatile("global_store_dwordx4 %4, %0, off sc0 sc1\n\tglobal_store_dwordx4 %4, %1, off offset:1024 sc0 sc1\n\t"
                                         "global_store_dwordx4 %4, %2, off offset:2048 sc0 sc1\n\tglobal_store_dwordx4 %4, %3, off offset:3072 sc0 sc1\n\ts_nop 1"
                                         :: "v"(d[0]), "v"(d[1]), "v"(d[2]), "v"(d[3]), "v"(p.ws[1 + pr] + go) : "memory");
                    asm volatile("s_waitcnt vmcnt(0)" ::: "memory");
                    if (lane < NR - 1 && lane != si) __hip_atomic_store((GAS unsigned*)(pws + WS_FLAG) + src * 1024 + g, call, __ATOMIC_RELAXED, __HIP_MEMORY_SCOPE_SYSTEM);
                    STAMP(13);
                }
            } else {
                const __amdgpu_buffer_rsrc_t rf = mk_rsrc(myws + WS_FLAG + (size_t)((rank + 1u + (unsigned)pw) & (NR - 1)) * 4096, (unsigned)NGR * 4u);
                GAS unsigned* const prog = (GAS unsigned*)(myws + WS_PROG + (size_t)pw * 4096 + (lane & 31) * 128);
                GAS unsigned* const tmo = (GAS unsigned*)(myws + WS_TMO);
                const unsigned long long tw0 = __builtin_amdgcn_s_memrealtime();
                for (int rdy = 0, spins = 0; rdy < NI; ) {
                    const int g0 = rdy < I1 ? 16 * rdy : G1 + 32 * (rdy - I1);
                    const unsigned f = __builtin_amdgcn_raw_buffer_load_b32(rf, (lane & 31) * 4, g0 * 4, 17);
                    const unsigned m = ~(unsigned)__ballot((int)(f - call) >= 0);
                    const int k = m == 0u ? (rdy + 1 < I1 ? 2 : 1) : (rdy < I1 && (m & 0xffffu) == 0u ? 1 : 0);
                    if (k > 0) {
                        rdy += k;
                        if (lane < 32) __hip_atomic_store(prog, call * 64u + (unsigned)rdy, __ATOMIC_RELAXED, __HIP_MEMORY_SCOPE_AGENT);
                        for (int z = rdy - k + 1; z <= rdy; ++z) STAMPF(2 + z);
                    } else if ((++spins & 15) == 0 &&
                               ((unsigned)__builtin_amdgcn_readfirstlane(__hip_atomic_load(tmo, __ATOMIC_RELAXED, __HIP_MEMORY_SCOPE_AGENT)) == call ||
                                __builtin_amdgcn_s_memrealtime() - tw0 > DEADLINE_TICKS)) {
                        if (lane == 0) __hip_atomic_store(tmo, call, __ATOMIC_RELAXED, __HIP_MEMORY_SCOPE_AGENT);
                        break;
                    }
                }
                STAMP(61);
            }
            return;
        }
        const unsigned vx = (lane & 15) * KD + (lane >> 4) * 16;
#define PUSH_REGS "v64", "v65", "v66", "v67", "v68", "v69", "v70", "v71", "v72", "v73", "v74", "v75", "v76", "v77", "v78", "v79", "v80", "v81"
        const int pv = wid * NPU + cg;
        int prev = 0, cur = 2 * pv, nxt = 2 * pv + 1;
#define PUSH_LOAD(D, U, PP) do { const unsigned pc_ = (unsigned)min(U, NU - 1) * 2u + (PP); \
        asm volatile("global_load_dwordx4 v[%c2:%c2+3], %0, %1" :: "v"(vx + (pc_ >> 6) * IBYTES + (pc_ & 63u) * 64u), "s"(p.x), "i"(64 + 4 * (D)) : "memory"); } while (0)
        asm volatile("" ::: PUSH_REGS);
        PUSH_LOAD(0, cur, 0); PUSH_LOAD(1, cur, 1);
        asm volatile("global_load_dword v81, %0, off sc1" :: "v"(cnt) : "memory");
        char* fws = p.ws[1];
#pragma unroll
        for (int i = 2; i < NR; ++i) fws = lane == i - 1 ? p.ws[i] : fws;
        GAS unsigned* flag = (GAS unsigned*)(fws + WS_FLAG) + rank * 1024;
        GAS unsigned* sink = (GAS unsigned*)(myws + WS_DUMMY + pv * 128) + (lane & 7);
        unsigned vs = (unsigned)WS_STAGE + rank * SHBYTES + lane * 16;
        unsigned call;
        asm volatile("s_waitcnt vmcnt(0)\n\tv_readfirstlane_b32 %0, v81" : "=s"(call), "+v"(flag), "+v"(sink), "+v"(vs) :: "memory", PUSH_REGS);
        call += 1u;
        const unsigned par = (call & 1u) * (NR * SHBYTES);
        GAS unsigned* const tkt = (GAS unsigned*)(myws + WS_TKT + (call & 1u) * 128);
        int skip = -1;
        STAMP(1);
#define ENDS_GROUP(U) ((U) >= 2 * G1 || ((U) & 1))
#define PUSH_FLAG(U) do { GAS unsigned* f_ = ENDS_GROUP(U) ? flag + ((U) < 2 * G1 ? (U) >> 1 : (U) - G1) : sink; \
        asm volatile("s_mov_b64 exec, 0x7f\n\tglobal_store_dword %0, %1, off sc0 sc1\n\ts_mov_b64 exec, -1" :: "v"(f_), "v"(call) : "memory"); } while (0)
#define PUSH_STORE(D, PP) do { \
        const unsigned o_ = vs + par + (unsigned)(cur * 2 + (PP)) * 1024u; \
        asm volatile("global_store_dwordx4 %0, v[%c1:%c1+3], %2 sc0 sc1\n\tglobal_store_dwordx4 %0, v[%c1:%c1+3], %3 sc0 sc1\n\t" \
                     "global_store_dwordx4 %0, v[%c1:%c1+3], %4 sc0 sc1\n\tglobal_store_dwordx4 %0, v[%c1:%c1+3], %5 sc0 sc1\n\t" \
                     "global_store_dwordx4 %0, v[%c1:%c1+3], %6 sc0 sc1\n\tglobal_store_dwordx4 %0, v[%c1:%c1+3], %7 sc0 sc1\n\t" \
                     "global_store_dwordx4 %0, v[%c1:%c1+3], %8 sc0 sc1" \
                     :: "v"(o_), "i"(64 + 4 * (D)), "s"(p.ws[1]), "s"(p.ws[2]), "s"(p.ws[3]), "s"(p.ws[4]), "s"(p.ws[5]), "s"(p.ws[6]), "s"(p.ws[7]) : "memory"); } while (0)
#define PUSH_UNIT(DA, DB, DC, DD) \
            { const bool draw = nxt < NU && ENDS_GROUP(nxt); \
            asm volatile("s_mov_b64 exec, 1\n\tglobal_atomic_add v80, %0, %1, off sc0\n\ts_mov_b64 exec, -1" :: "v"(draw ? tkt : sink + 8), "v"(1u) : "memory"); \
            PUSH_LOAD(DC, nxt, 0); \
            PUSH_STORE(DA, 0); \
            asm volatile("s_waitcnt vmcnt(9)" ::: "memory", PUSH_REGS); \
            PUSH_FLAG(prev); \
            PUSH_LOAD(DD, nxt, 1); \
            PUSH_STORE(DB, 1); \
            unsigned tk; \
            asm volatile("s_waitcnt vmcnt(9)\n\tv_readfirstlane_b32 %0, v80" : "=s"(tk) :: "memory", PUSH_REGS); \
            int g = (int)tk + NPW; \
            if (draw && g >= GA) { \
                for (int sp = 0; skip < 0; ++sp) {       \
                    unsigned pm; asm volatile("s_load_dword %0, %1, 0x0 glc\n\ts_waitcnt lgkmcnt(0)" : "=&s"(pm) : "s"(myws + WS_PLAN + cg * 128 + 28) : "memory"); \
                    if ((pm >> 16) == (call & 0xffffu)) skip = (int)(pm & 0xffffu); \
                    else if (sp > 1000000) { if (lane == 0) __hip_atomic_store((GAS unsigned*)(myws + WS_TMO), call, __ATOMIC_RELAXED, __HIP_MEMORY_SCOPE_AGENT); skip = 0; } \
                } \
                g += skip; \
            } \
            prev = cur; cur = nxt; \
            nxt = draw ? (g < G1 ? 2 * g : G1 + g) : nxt + 1; \
            if (cur >= NU) break; }
        for (;;) {
            PUSH_UNIT(0, 1, 2, 3)
            PUSH_UNIT(2, 3, 0, 1)
        }
        asm volatile("s_waitcnt vmcnt(0)" ::: "memory", PUSH_REGS);
        PUSH_FLAG(prev);
        if (lane == 0) __hip_atomic_store(cnt, call, __ATOMIC_RELAXED, __HIP_MEMORY_SCOPE_AGENT);
        STAMP(63);
        return;
    }

    const int pi = rg - 1;
    const unsigned srank = (rank + 1u + (unsigned)pi) & (NR - 1);
    const int nloc = pi < 4 ? 5 : 4;
    const int l0 = pi < 4 ? 5 * pi : 4 * pi + 4;
    const int q = lane >> 4, n = lane & 15;
#define SB0 __builtin_amdgcn_sched_barrier(0)
#define WAITV(N) asm volatile("s_waitcnt vmcnt(%c0)" :: "i"(N) : "memory")
    typedef const __attribute__((address_space(4))) float* cfloat_p;
    const int scale_s = __builtin_amdgcn_readfirstlane(__builtin_bit_cast(int, *(cfloat_p)p.sx * *(cfloat_p)p.sw));

    const int lu = (pi * 32 + cg) * 4 + wid;
    unsigned call;
    const char* stage;
    {
        const __amdgpu_buffer_rsrc_t rx = mk_rsrc(p.x, lu < NLU ? (unsigned)SHBYTES : 0u);
        const __amdgpu_buffer_rsrc_t rw = mk_rsrc(p.w, (unsigned)KD * NL);
#define PRO_REGS "v128", "v129", "v130", "v131", "v132", "v133", "v134", "v135", "v136", "v137", "v138", "v139", "v140", "v141", "v142", "v143", "v144", "v145", "v146"
        asm volatile("global_load_dword v144, %0, off sc1\n\t"
                     "buffer_load_dwordx4 v[128:131], %1, %2, %3 offen\n\tbuffer_load_dwordx4 v[132:135], %1, %2, %3 offen offset:64\n\t"
                     "buffer_load_dwordx4 v[136:139], %1, %2, %3 offen offset:128\n\tbuffer_load_dwordx4 v[140:143], %1, %2, %3 offen offset:192"
                     :: "v"(cnt), "v"(n * KD + q * 16), "s"(rx), "s"(((lu * LU) >> 6) * IBYTES + ((lu * LU) & 63) * 64) : "memory", PRO_REGS);
        const int vw0 = (lane >> 2) * NL + (((lane & 3) ^ (q & 1)) * 16), vw1 = (lane >> 2) * NL + (((lane & 3) ^ (q & 1) ^ 2) * 16);
        const int sw = (wid * 1024) * NL + cg * 64;
        const unsigned lw = (unsigned)(size_t)(LAS char*)lds + wid * 32768;
        unsigned tr[4];
#pragma unroll
        for (int j = 0; j < 4; ++j) tr[j] = lw + (q * 16 + (n >> 1)) * 64 + ((j ^ ((n >> 3) | ((q & 1) << 1))) * 16) + (n & 1) * 8;
#define W_DMA(KS) do { _Pragma("unroll") for (int i_ = 0; i_ < 8; ++i_) \
            asm volatile("s_mov_b32 m0, %3\n\ts_nop 0\n\tbuffer_load_dwordx4 %0, %1, %2 offen lds" \
                         :: "v"((i_ & 1) ? vw1 : vw0), "s"(rw), "s"(sw + ((KS) * 128 + i_ * 16) * NL), "s"(lw + ((KS) & 3) * 8192 + i_ * 1024) : "memory"); } while (0)
#define W_TR(KS) do { _Pragma("unroll") for (int j_ = 0; j_ < 4; ++j_) \
            asm volatile("ds_read_b64_tr_b8 a[%c1:%c1+1], %0 offset:%c2\n\tds_read_b64_tr_b8 a[%c1+2:%c1+3], %0 offset:%c2+512\n\t" \
                         "ds_read_b64_tr_b8 a[%c1+4:%c1+5], %0 offset:%c2+4096\n\tds_read_b64_tr_b8 a[%c1+6:%c1+7], %0 offset:%c2+4608" \
                         :: "v"(tr[j_]), "i"((KS) * 32 + j_ * 8), "i"(((KS) & 3) * 8192) : "memory"); \
            asm volatile("s_waitcnt lgkmcnt(0)" ::: "memory"); } while (0)
        asm volatile("" ::: "a0", "a1", "a2", "a3", "a4", "a5", "a6", "a7", "a8", "a9", "a10", "a11", "a12", "a13", "a14", "a15", "a16", "a17", "a18", "a19", "a20", "a21", "a22", "a23", "a24", "a25", "a26", "a27", "a28", "a29", "a30", "a31", "a32", "a33", "a34", "a35", "a36", "a37", "a38", "a39", "a40", "a41", "a42", "a43", "a44", "a45", "a46", "a47", "a48", "a49", "a50", "a51", "a52", "a53", "a54", "a55", "a56", "a57", "a58", "a59", "a60", "a61", "a62", "a63", "a64", "a65", "a66", "a67", "a68", "a69", "a70", "a71", "a72", "a73", "a74", "a75", "a76", "a77", "a78", "a79", "a80", "a81", "a82", "a83", "a84", "a85", "a86", "a87", "a88", "a89", "a90", "a91", "a92", "a93", "a94", "a95", "a96", "a97", "a98", "a99", "a100", "a101", "a102", "a103", "a104", "a105", "a106", "a107", "a108", "a109", "a110", "a111", "a112", "a113", "a114", "a115", "a116", "a117", "a118", "a119", "a120", "a121", "a122", "a123", "a124", "a125", "a126", "a127", "a128", "a129", "a130", "a131", "a132", "a133", "a134", "a135", "a136", "a137", "a138", "a139", "a140", "a141", "a142", "a143", "a144", "a145", "a146", "a147", "a148", "a149", "a150", "a151", "a152", "a153", "a154", "a155", "a156", "a157", "a158", "a159", "a160", "a161", "a162", "a163", "a164", "a165", "a166", "a167", "a168", "a169", "a170", "a171", "a172", "a173", "a174", "a175", "a176", "a177", "a178", "a179", "a180", "a181", "a182", "a183", "a184", "a185", "a186", "a187", "a188", "a189", "a190", "a191", "a192", "a193", "a194", "a195", "a196", "a197", "a198", "a199", "a200", "a201", "a202", "a203", "a204", "a205", "a206", "a207", "a208", "a209", "a210", "a211", "a212", "a213", "a214", "a215", "a216", "a217", "a218", "a219", "a220", "a221", "a222", "a223", "a224", "a225", "a226", "a227", "a228", "a229", "a230", "a231", "a232", "a233", "a234", "a235", "a236", "a237", "a238", "a239", "a240", "a241", "a242", "a243", "a244", "a245", "a246", "a247", "a248", "a249", "a250", "a251", "a252", "a253", "a254", "a255");
        constexpr int LPR = BC / 2, NT = (KD * LPR + 112 * 64 - 1) / (112 * 64);
#pragma unroll
        for (int i = 0; i < NT; ++i) {
            const int li = ((((b >> 3) - 4) * 4 + wid) * NT + i) * 64 + lane;
            asm volatile("buffer_load_dword v145, %0, %1, 0 offen" :: "v"((li / LPR) * NL + (cg / BC) * (BC * 64) + (li % LPR) * 128), "s"(rw) : "memory", PRO_REGS);
        }
        W_DMA(0);
        asm volatile("s_waitcnt vmcnt(%c1)\n\tv_readfirstlane_b32 %0, v144" : "=s"(call) : "i"(8 + NT) : "memory", PRO_REGS);
        call += 1u;
        stage = myws + WS_STAGE + (size_t)(call & 1u) * (NR * SHBYTES);
        const __amdgpu_buffer_rsrc_t rl = mk_rsrc(stage + (size_t)rank * SHBYTES, lu < NLU ? (unsigned)SHBYTES : 0u);
        asm volatile("global_store_dword %0, %1, off sc1\n\t"
                     "buffer_store_dwordx4 v[128:131], %2, %3, %4 offen sc0 sc1\n\tbuffer_store_dwordx4 v[132:135], %2, %3, %4 offen offset:1024 sc0 sc1\n\t"
                     "buffer_store_dwordx4 v[136:139], %2, %3, %4 offen offset:2048 sc0 sc1\n\tbuffer_store_dwordx4 v[140:143], %2, %3, %4 offen offset:3072 sc0 sc1"
                     :: "v"(cnt), "v"(call), "v"(lane * 16), "s"(rl), "s"(lu * (LU * 1024)) : "memory", PRO_REGS);
        W_DMA(1); W_DMA(2); W_DMA(3);
        WAITV(24);
        if (lane == 0 && lu < NLU) __hip_atomic_store((GAS unsigned*)(myws + WS_LFLAG + lu * 128), call, __ATOMIC_RELAXED, __HIP_MEMORY_SCOPE_SYSTEM);
        STAMP(40); W_TR(0); W_DMA(4);
        WAITV(24); W_TR(1); W_DMA(5);
        WAITV(24); W_TR(2); W_DMA(6);
        WAITV(24); STAMP(41); W_TR(3); W_DMA(7);
        WAITV(24); W_TR(4);
        WAITV(16); W_TR(5);
        WAITV(8); W_TR(6);
        WAITV(0); W_TR(7);
    }
    STAMP(1);

    GAS unsigned* const tmo = (GAS unsigned*)(myws + WS_TMO);
    const __amdgpu_buffer_rsrc_t rs = mk_rsrc(stage, (unsigned)NR * SHBYTES);
    const char* const prog = myws + WS_PROG + (size_t)pi * 4096 + cg * 128;
    const int voff_a = lane * 16;
    const int voff_out = ((wid * 4 + q) * NL + n * 4) * 4;
    bool dead = false;
#define SPIN(READY_EXPR, LOADV) do { \
        const unsigned long long t0_ = __builtin_amdgcn_s_memrealtime(); \
        for (int spins_ = 1;; ++spins_) { \
            const unsigned v_ = LOADV; \
            if (READY_EXPR) break; \
            if ((spins_ & 15) == 0 && ((unsigned)__builtin_amdgcn_readfirstlane(__hip_atomic_load(tmo, __ATOMIC_RELAXED, __HIP_MEMORY_SCOPE_AGENT)) == call || \
                __builtin_amdgcn_s_memrealtime() - t0_ > DEADLINE_TICKS)) { \
                if (lane == 0) __hip_atomic_store(tmo, call, __ATOMIC_RELAXED, __HIP_MEMORY_SCOPE_AGENT); \
                dead = true; \
                rdy = NI; \
                break; \
            } \
            __builtin_amdgcn_s_sleep(1); \
        } } while (0)
#define SAMPLE() ({ unsigned sv_; asm volatile("s_load_dword %0, %1, 0x0 glc\n\ts_waitcnt lgkmcnt(0)" : "=&s"(sv_) : "s"(prog) : "memory"); sv_; })
#define ADVANCE(V) do { rdy = max(rdy, (int)((V) - call * 64u)); } while (0)
    int rdy = 0;
    {
        const __amdgpu_buffer_rsrc_t rfl = mk_rsrc(myws + WS_LFLAG, (unsigned)NLU * 128u);
        constexpr int LPI = 64 / LU;
        SPIN(__all(lane >= nloc * 4 || (int)(v_ - call) >= 0), __builtin_amdgcn_raw_buffer_load_b32(rfl, (((lane >> 2) + l0) * LPI + wid * 4 + (lane & 3)) * 128, 0, 17));
    }
    STAMP(42);

    const unsigned red = (unsigned)(size_t)(LAS char*)lds + WRING_BYTES;
    unsigned put[4];
#pragma unroll
    for (int j = 0; j < 4; ++j) put[j] = red + wid * 4096 + j * 1024 + q * 256 + ((n ^ (4 * j + q)) * 16);
    const unsigned get = red + (n >> 2) * 1024 + (n & 3) * 256 + (((4 * wid + q) ^ n) * 16);
    const __amdgpu_buffer_rsrc_t ro = mk_rsrc(p.out, (unsigned)NR * SH * NL * 4u);
    int tn = 1 - nloc, st = 1 + (nloc & 1);
    int sa = (int)rank * SHBYTES + l0 * IBYTES + wid * 16384;
    int sa1 = sa + IBYTES;
    int so2 = 0x7f000000;
#define DBG_A0 ""
#define DBG_A1 ""
#define DBG_A2 ""
#define DBG_AO
    unsigned long long tw = 0;
    int tws = NI + 1;
    for (;;) {
        unsigned smp;
        rdy = __builtin_amdgcn_readfirstlane(rdy);
        asm volatile(
        ".macro MF B, ks, j\n"
        " v_mfma_f32_16x16x128_f8f6f4 v[110+4*\\j:113+4*\\j], a[32*\\ks+8*\\j:32*\\ks+8*\\j+7], v[128+64*\\B+8*\\ks:135+64*\\B+8*\\ks], v[110+4*\\j:113+4*\\j]\n"
        ".endm\n"
        ".macro MF0 B, j\n"
        " v_mfma_f32_16x16x128_f8f6f4 v[110+4*\\j:113+4*\\j], a[8*\\j:8*\\j+7], v[128+64*\\B:135+64*\\B], v[74:77]\n"
        ".endm\n"
        ".macro LH B, ks, hf, sb\n"
        " buffer_load_dwordx4 v[128+64*(\\B)+8*(\\ks)+4*(\\hf):131+64*(\\B)+8*(\\ks)+4*(\\hf)], v[70+((\\ks)>>1)], %[rs], \\sb offen offset:((\\ks)&1)*2048+(\\hf)*1024 sc0 sc1\n"
        ".endm\n"
        ".macro V4 op, d, a, c\n"
        " \\op v[\\d], v[\\a], v[\\c]\n"
        " \\op v[\\d+1], v[\\a+1], v[\\c+1]\n"
        " \\op v[\\d+2], v[\\a+2], v[\\c+2]\n"
        " \\op v[\\d+3], v[\\a+3], v[\\c+3]\n"
        ".endm\n"
        ".macro K4 op, d, k\n"
        " \\op v[\\d], \\k, v[\\d]\n"
        " \\op v[\\d+1], \\k, v[\\d+1]\n"
        " \\op v[\\d+2], \\k, v[\\d+2]\n"
        " \\op v[\\d+3], \\k, v[\\d+3]\n"
        ".endm\n"
        ".macro U2 op, d\n"
        " \\op v[\\d], v[\\d]\n"
        " \\op v[\\d+1], v[\\d+1]\n"
        ".endm\n"
        ".macro CLAMP d, a\n"
        " v_med3_f32 v[\\d], v[\\a], %[m60], %[c60]\n"
        " v_med3_f32 v[\\d+1], v[\\a+1], %[m60], %[c60]\n"
        " v_med3_f32 v[\\d+2], v[\\a+2], %[m60], %[c60]\n"
        " v_med3_f32 v[\\d+3], v[\\a+3], %[m60], %[c60]\n"
        ".endm\n"
        ".macro GET4 d, B\n"
        " ds_read_b128 v[\\d:\\d+3], %[get] offset:(\\B)*16384\n"
        " ds_read_b128 v[\\d+4:\\d+7], %[get] offset:(\\B)*16384+4096\n"
        " ds_read_b128 v[\\d+8:\\d+11], %[get] offset:(\\B)*16384+8192\n"
        " ds_read_b128 v[\\d+12:\\d+15], %[get] offset:(\\B)*16384+12288\n"
        ".endm\n"
        ".macro SAMPLE\n"
        " s_load_dword %[smp], %[prog], 0x0 glc\n"
        ".endm\n"
        ".macro TAKE\n"
        " s_sub_i32 %[smp], %[smp], %[c64]\n"
        " s_max_i32 %[rdy], %[rdy], %[smp]\n"
        ".endm\n"
        ".macro REQ1 B, H, K, hf\n"
        " .if \\H == 0\n"
        "  LH 1-\\B, \\K, \\hf, %[sa1]\n"
        " .endif\n"
        " .if \\H == 1 && \\K < 2\n"
        "  LH \\B, \\K+6, \\hf, %[sa]\n"
        " .endif\n"
        " .if \\hf == 1\n"
        "  .if \\H == 0\n"
        "   s_waitcnt vmcnt(14)\n"
        "  .elseif \\H == 1 && \\K < 2\n"
        "   s_waitcnt vmcnt(10)\n"
        "  .else\n"
        "   s_waitcnt vmcnt(12-2*\\K)\n"
        "  .endif\n"
        " .endif\n"
        ".endm\n"
        ".macro REQ2 B, H, K, hf\n"
        " .if \\H == 0\n"
        "  LH 1-\\B, \\K, \\hf, %[sa1]\n"
        " .endif\n"
        " .if \\H == 1 && \\K >= 6\n"
        "  LH 1-\\B, \\K-2, \\hf, %[sa1]\n"
        " .endif\n"
        " .if \\H == 3 || (\\H == 1 && \\K < 6)\n"
        "  LH 1-\\B, 2*\\K-8+\\hf, 0, %[sa1]\n"
        "  LH 1-\\B, 2*\\K-8+\\hf, 1, %[sa1]\n"
        " .endif\n"
        " .if \\hf == 1 && \\K < 7\n"
        "  .if \\H == 0\n"
        "   s_waitcnt vmcnt(14+\\K/6)\n"
        "  .elseif \\H == 1 && \\K == 6\n"
        "   s_waitcnt vmcnt(11)\n"
        "  .elseif \\H == 2\n"
        "   s_waitcnt vmcnt(12-2*\\K+\\K/6)\n"
        "  .else\n"
        "   s_waitcnt vmcnt(2*\\K+\\K/6)\n"
        "  .endif\n"
        " .endif\n"
        ".endm\n"
        ".macro HALF1 B, H\n" DBG_A1
        " ds_write_b128 %[put0], v[110:113] offset:(1-\\B)*16384\n"
        " .if \\H == 1\n"
        "  s_waitcnt vmcnt(10)\n"
        " .else\n"
        "  s_waitcnt vmcnt(14)\n"
        " .endif\n"
        " MF0 \\B, 0\n"
        " ds_write_b128 %[put1], v[114:117] offset:(1-\\B)*16384\n"
        " REQ1 \\B, \\H, 0, 0\n"
        " MF0 \\B, 1\n"
        " ds_write_b128 %[put2], v[118:121] offset:(1-\\B)*16384\n"
        " MF0 \\B, 2\n"
        " ds_write_b128 %[put3], v[122:125] offset:(1-\\B)*16384\n"
        " MF0 \\B, 3\n"
        " REQ1 \\B, \\H, 0, 1\n"
        " MF \\B, 1, 0\n"
        " .if \\H != 0\n"
        "  SAMPLE\n"
        " .endif\n"
        " REQ1 \\B, \\H, 1, 0\n"
        " MF \\B, 1, 1\n"
        " V4 v_add_f32, 78, 78, 82\n"
        " MF \\B, 1, 2\n"
        " V4 v_add_f32, 86, 86, 90\n"
        " MF \\B, 1, 3\n"
        " REQ1 \\B, \\H, 1, 1\n"
        " MF \\B, 2, 0\n"
        " REQ1 \\B, \\H, 2, 0\n"
        " .if \\H == 0\n"
        "  s_waitcnt lgkmcnt(0)\n"
        "  SAMPLE\n"
        " .endif\n"
        " MF \\B, 2, 1\n"
        " V4 v_add_f32, 78, 78, 86\n"
        " MF \\B, 2, 2\n"
        " K4 v_mul_f32, 78, %[scale]\n"
        " MF \\B, 2, 3\n"
        " REQ1 \\B, \\H, 2, 1\n"
        " MF \\B, 3, 0\n"
        " REQ1 \\B, \\H, 3, 0\n"
        " MF \\B, 3, 1\n"
        " CLAMP 82, 78\n"
        " MF \\B, 3, 2\n"
        " K4 v_mul_f32, 82, %[nl2e]\n"
        " MF \\B, 3, 3\n"
        " REQ1 \\B, \\H, 3, 1\n"
        " .if \\H != 0\n"
        "  s_waitcnt lgkmcnt(0)\n"
        "  TAKE\n"
        " .endif\n"
        " MF \\B, 4, 0\n"
        " s_barrier\n"
        ".endm\n"
        ".macro HALF2 B, H\n"
        " .if \\H != 0\n"
        "  SAMPLE\n"
        " .endif\n"
        " REQ2 \\B, \\H, 4, 0\n"
        " MF \\B, 4, 1\n"
        " U2 v_exp_f32, 82\n"
        " MF \\B, 4, 2\n"
        " U2 v_exp_f32, 84\n"
        " MF \\B, 4, 3\n"
        " REQ2 \\B, \\H, 4, 1\n"
        " MF \\B, 5, 0\n"
        " REQ2 \\B, \\H, 5, 0\n"
        " K4 v_add_f32, 82, 1.0\n"
        " MF \\B, 5, 1\n"
        " U2 v_rcp_f32, 82\n"
        " MF \\B, 5, 2\n"
        " U2 v_rcp_f32, 84\n"
        " MF \\B, 5, 3\n"
        " REQ2 \\B, \\H, 5, 1\n"
        " MF \\B, 6, 0\n"
        " REQ2 \\B, \\H, 6, 0\n"
        " V4 v_mul_f32, 78, 78, 82\n"
        " MF \\B, 6, 1\n"
        " buffer_store_dwordx4 v[78:81], %[vout], %[ro], %[so2] offen sc1\n"
        " MF \\B, 6, 2\n"
        " s_add_i32 %[so2], %[so2], 0x20000\n"
        " s_cmp_eq_u32 %[tn], %[tA]\n"
        " s_cselect_b32 %[so2], %[ol], %[so2]\n"
        " s_cmp_eq_u32 %[tn], 2\n"
        " s_cselect_b32 %[so2], %[orm], %[so2]\n"
        " MF \\B, 6, 3\n"
        " REQ2 \\B, \\H, 6, 1\n"
        " MF \\B, 7, 0\n"
        " REQ2 \\B, \\H, 7, 0\n"
        " GET4 78, 1-\\B\n"
        " MF \\B, 7, 1\n"
        " REQ2 \\B, \\H, 7, 1\n"
        " s_mov_b32 %[sa], %[sa1]\n"
        " s_add_i32 %[sa1], %[sa1], 0x10000\n"
        " s_add_i32 %[tn], %[tn], 1\n"
        " MF \\B, 7, 2\n"
        " s_cmp_eq_u32 %[tn], 0\n"
        " s_cselect_b32 %[sa1], %[sr], %[sa1]\n"
        " s_waitcnt lgkmcnt(0)\n"
        " TAKE\n"
        " s_cmp_lt_i32 %[tn], %[rdy]\n"
        " MF \\B, 7, 3\n"
        ".endm\n"
        ".macro BSIDE B\n"
        "Lcold\\B\\()_%=:\n"
        " s_cmp_le_i32 %[tn], %[rdy]\n"
        " s_cbranch_scc1 Lgo\\B\\()_%=\n"
        " s_movk_i32 %[st], 64\n"
        "Lspin\\B\\()_%=:\n"
        " SAMPLE\n"
        " s_waitcnt lgkmcnt(0)\n"
        " TAKE\n"
        " s_cmp_le_i32 %[tn], %[rdy]\n"
        " s_cbranch_scc1 Lgo\\B\\()_%=\n"
        " s_sub_u32 %[st], %[st], 1\n"
        " s_cbranch_scc0 Lspin\\B\\()_%=\n"
        " s_movk_i32 %[st], 1+\\B\n"
        " s_branch Lexit_%=\n"
        "Lgo\\B\\()_%=:\n" DBG_A0
        " LH \\B, 0, 0, %[sa]\n"
        " LH \\B, 0, 1, %[sa]\n"
        " LH \\B, 1, 0, %[sa]\n"
        " LH \\B, 1, 1, %[sa]\n"
        " LH \\B, 2, 0, %[sa]\n"
        " LH \\B, 2, 1, %[sa]\n"
        " LH \\B, 3, 0, %[sa]\n"
        " LH \\B, 3, 1, %[sa]\n"
        " LH \\B, 4, 0, %[sa]\n"
        " LH \\B, 4, 1, %[sa]\n"
        " LH \\B, 5, 0, %[sa]\n"
        " LH \\B, 5, 1, %[sa]\n"
        "Lb\\B\\()_%=:\n"
        " HALF1 \\B, 1\n"
        "Lmid\\B\\()_%=:\n"
        " s_cmp_ge_i32 %[tn], %[rdy]\n"
        " s_cbranch_scc1 Lc2\\B\\()_%=\n"
        " s_add_i32 %[st], %[tn], " UPG_STR "\n"
        " s_cmp_lt_i32 %[st], %[rdy]\n"
        " s_cbranch_scc1 Ld2\\B\\()_%=\n"
        " HALF2 \\B, 1\n"
        ".endm\n"
        ".macro OTHER B, N\n"
        "Lc1\\B\\()_%=:\n"
        " HALF1 \\B, 2\n"
        " s_branch Lmid\\B\\()_%=\n"
        "Lc2\\B\\()_%=:\n"
        " .if \\B == 1\n"
        "  s_cmp_eq_u32 %[tn], 32\n"
        "  s_cbranch_scc1 Lfin_%=\n"
        " .endif\n"
        " HALF2 \\B, 2\n"
        " s_branch Lcold\\N\\()_%=\n"
        "Ld2\\B\\()_%=:\n"
        " HALF2 \\B, 3\n"
        " s_cbranch_scc1 La\\N\\()_%=\n"
        " s_branch Lc1\\N\\()_%=\n"
        ".endm\n"
        " s_nop 4\n"
        " v_mov_b32 v74, %[cin]\n"
        " v_mov_b32 v75, %[cin]\n"
        " v_mov_b32 v76, %[cin]\n"
        " v_mov_b32 v77, %[cin]\n"
        " v_mov_b32 v70, %[va]\n"
        " v_add_u32 v71, 0x1000, %[va]\n"
        " v_add_u32 v72, 0x2000, %[va]\n"
        " v_add_u32 v73, 0x3000, %[va]\n"
        " s_cmp_eq_u32 %[st], 2\n"
        " s_cbranch_scc1 Lcold1_%=\n"
        " BSIDE 0\n"
        " s_branch Lb1_%=\n"
        " BSIDE 1\n"
        " s_branch Lb0_%=\n"
        "La0_%=:\n"
        " HALF1 0, 0\n"
        " HALF2 0, 0\n"
        " s_cbranch_scc0 Lc11_%=\n"
        "La1_%=:\n"
        " HALF1 1, 0\n"
        " HALF2 1, 0\n"
        " s_cbranch_scc1 La0_%=\n"
        " OTHER 0, 1\n"
        " OTHER 1, 0\n"
        "Lfin_%=:\n" DBG_A2
        " GET4 94, 0\n"
        " MF 1, 4, 1\n"
        " U2 v_exp_f32, 82\n"
        " MF 1, 4, 2\n"
        " U2 v_exp_f32, 84\n"
        " MF 1, 4, 3\n"
        " s_waitcnt vmcnt(4)\n"
        " MF 1, 5, 0\n"
        " K4 v_add_f32, 82, 1.0\n"
        " MF 1, 5, 1\n"
        " U2 v_rcp_f32, 82\n"
        " MF 1, 5, 2\n"
        " U2 v_rcp_f32, 84\n"
        " MF 1, 5, 3\n"
        " s_waitcnt vmcnt(0) lgkmcnt(0)\n"
        " MF 1, 6, 0\n"
        " V4 v_mul_f32, 78, 78, 82\n"
        " MF 1, 7, 0\n"
        " buffer_store_dwordx4 v[78:81], %[vout], %[ro], %[so2] offen sc1\n"
        " V4 v_add_f32, 94, 94, 98\n"
        " MF 1, 6, 1\n"
        " V4 v_add_f32, 102, 102, 106\n"
        " MF 1, 7, 1\n"
        " V4 v_add_f32, 94, 94, 102\n"
        " MF 1, 6, 2\n"
        " ds_write_b128 %[put0], v[110:113] offset:16384\n"
        " K4 v_mul_f32, 94, %[scale]\n"
        " MF 1, 7, 2\n"
        " CLAMP 98, 94\n"
        " MF 1, 6, 3\n"
        " ds_write_b128 %[put1], v[114:117] offset:16384\n"
        " K4 v_mul_f32, 98, %[nl2e]\n"
        " MF 1, 7, 3\n"
        " U2 v_exp_f32, 98\n"
        " U2 v_exp_f32, 100\n"
        " ds_write_b128 %[put2], v[118:121] offset:16384\n"
        " K4 v_add_f32, 98, 1.0\n"
        " U2 v_rcp_f32, 98\n"
        " U2 v_rcp_f32, 100\n"
        " ds_write_b128 %[put3], v[122:125] offset:16384\n"
        " s_add_i32 %[so2], %[so2], 0x20000\n"
        " V4 v_mul_f32, 94, 94, 98\n"
        " buffer_store_dwordx4 v[94:97], %[vout], %[ro], %[so2] offen sc1\n"
        " s_waitcnt lgkmcnt(0)\n"
        " s_barrier\n"
        " GET4 78, 1\n"
        " s_add_i32 %[so2], %[so2], 0x20000\n"
        " s_waitcnt lgkmcnt(2)\n"
        " V4 v_add_f32, 78, 78, 82\n"
        " s_waitcnt lgkmcnt(0)\n"
        " V4 v_add_f32, 86, 86, 90\n"
        " V4 v_add_f32, 78, 78, 86\n"
        " K4 v_mul_f32, 78, %[scale]\n"
        " CLAMP 82, 78\n"
        " K4 v_mul_f32, 82, %[nl2e]\n"
        " U2 v_exp_f32, 82\n"
        " U2 v_exp_f32, 84\n"
        " s_mov_b32 %[st], 0\n"
        " K4 v_add_f32, 82, 1.0\n"
        " U2 v_rcp_f32, 82\n"
        " U2 v_rcp_f32, 84\n"
        " s_nop 0\n"
        " V4 v_mul_f32, 78, 78, 82\n"
        " buffer_store_dwordx4 v[78:81], %[vout], %[ro], %[so2] offen sc1\n"
        "Lexit_%=:\n"
        ".purgem MF\n"
        ".purgem MF0\n"
        ".purgem LH\n"
        ".purgem V4\n"
        ".purgem K4\n"
        ".purgem U2\n"
        ".purgem CLAMP\n"
        ".purgem GET4\n"
        ".purgem SAMPLE\n"
        ".purgem TAKE\n"
        ".purgem REQ1\n"
        ".purgem REQ2\n"
        ".purgem HALF1\n"
        ".purgem HALF2\n"
        ".purgem BSIDE\n"
        ".purgem OTHER\n"
            : [tn] "+s"(tn), [rdy] "+s"(rdy), [sa] "+s"(sa), [sa1] "+s"(sa1), [so2] "+s"(so2), [st] "+s"(st), [smp] "=&s"(smp) DBG_AO
            : [rs] "s"(rs), [ro] "s"(ro), [prog] "s"(prog), [c64] "s"(call * 64u), [scale] "s"(scale_s), [m60] "s"(-60.0f),
              [nl2e] "s"(-1.44269504088896340736f), [tA] "s"(2 - nloc), [ol] "s"((((int)rank * SH + l0 * IROWS) * NL + cg * 64) * 4),
              [orm] "s"((((int)srank * SH) * NL + cg * 64) * 4), [sr] "s"((int)srank * SHBYTES + wid * 16384), [cin] "s"(__builtin_amdgcn_readfirstlane(dead ? 0x7fc00000 : 0)),
              [va] "v"(voff_a), [vout] "v"(voff_out), [c60] "v"(60.0f), [get] "v"(get), [put0] "v"(put[0]), [put1] "v"(put[1]), [put2] "v"(put[2]), [put3] "v"(put[3])
            : "memory", "scc", "v70", "v71", "v72", "v73", "v74", "v75", "v76", "v77", "v78", "v79", "v80", "v81", "v82", "v83", "v84", "v85", "v86", "v87", "v88", "v89", "v90", "v91", "v92", "v93", "v94", "v95", "v96", "v97", "v98", "v99", "v100", "v101", "v102", "v103", "v104", "v105", "v106", "v107", "v108", "v109", "v110", "v111", "v112", "v113", "v114", "v115", "v116", "v117", "v118", "v119", "v120", "v121", "v122", "v123", "v124", "v125", "v126", "v127", "v128", "v129", "v130", "v131", "v132", "v133", "v134", "v135", "v136", "v137", "v138", "v139", "v140", "v141", "v142", "v143", "v144", "v145", "v146", "v147", "v148", "v149", "v150", "v151", "v152", "v153", "v154", "v155", "v156", "v157", "v158", "v159", "v160", "v161", "v162", "v163", "v164", "v165", "v166", "v167", "v168", "v169", "v170", "v171", "v172", "v173", "v174", "v175", "v176", "v177", "v178", "v179", "v180", "v181", "v182", "v183", "v184", "v185", "v186", "v187", "v188", "v189", "v190", "v191", "v192", "v193", "v194", "v195", "v196", "v197", "v198", "v199", "v200", "v201", "v202", "v203", "v204", "v205", "v206", "v207", "v208", "v209", "v210", "v211", "v212", "v213", "v214", "v215", "v216", "v217", "v218", "v219", "v220", "v221", "v222", "v223", "v224", "v225", "v226", "v227", "v228", "v229", "v230", "v231", "v232", "v233", "v234", "v235", "v236", "v237", "v238", "v239", "v240", "v241", "v242", "v243", "v244", "v245", "v246", "v247", "v248", "v249", "v250", "v251", "v252", "v253", "v254", "v255");
        if (st == 0) break;
        const unsigned long long now = __builtin_amdgcn_s_memrealtime();
        if (tws != tn) { tws = tn; tw = now; }
        if ((unsigned)__builtin_amdgcn_readfirstlane(__hip_atomic_load(tmo, __ATOMIC_RELAXED, __HIP_MEMORY_SCOPE_AGENT)) == call || now - tw > DEADLINE_TICKS) {
            if (lane == 0) __hip_atomic_store(tmo, call, __ATOMIC_RELAXED, __HIP_MEMORY_SCOPE_AGENT);
            dead = true;
            rdy = NI;
        }
    }
    STAMP(63);
}

}

extern "C" void kernel_launch(int rank, int nranks, ncclComm_t comm,
                              hipStream_t stream,
                              void* const* d_in, const int* in_sizes, int n_in,
                              void* d_out, int out_size,
                              const int* in_dtypes, int out_dtype,
                              void* const* d_out_all,
                              void* const* d_ws_all, size_t ws_bytes) {
    (void)comm; (void)in_dtypes; (void)out_dtype; (void)d_out_all;
    if (nranks != NR || rank < 0 || rank >= NR || n_in < 4 || in_sizes[0] != SH * KD || in_sizes[1] != KD * NL ||
        out_size != NR * SH * NL || ws_bytes < WS_NEED) {
        fprintf(stderr, "ag_gemm: unexpected problem shape\n");
        return;
    }
    Params p;
    for (int i = 0; i < NR; ++i) p.ws[i] = (char*)d_ws_all[(rank + i) % NR];
    p.x = (const char*)d_in[0];
    p.w = (const char*)d_in[1];
    p.sx = (const float*)d_in[2];
    p.sw = (const float*)d_in[3];
    p.out = (float*)d_out;
    p.rank = rank;
    p.pad = 0;
    ag_gemm_kernel<<<dim3(NWG), dim3(256), 0, stream>>>(p);
    HIP_CHECK(hipGetLastError());
}
```

```cpp
#include <hip/hip_runtime.h>
#include <rccl/rccl.h>
#include <cstdio>

#define HIP_CHECK(expr) do { hipError_t _e = (expr); if (_e != hipSuccess) { \
    fprintf(stderr, "HIP error %s at %s:%d\n", hipGetErrorString(_e), __FILE__, __LINE__); \
    return; } } while (0)

namespace {

typedef unsigned u32x4 __attribute__((ext_vector_type(4)));
typedef unsigned u32x8 __attribute__((ext_vector_type(8)));
#define LAS __attribute__((address_space(3)))
#define GAS __attribute__((address_space(1)))

constexpr int NR = 8;
constexpr int KD = 4096;
constexpr int NL = 2048;
constexpr int SH = 512;
constexpr int IROWS = 16;
constexpr int NI = SH / IROWS;
constexpr int IBYTES = IROWS * KD;
constexpr int SHBYTES = SH * KD;
constexpr int NWG = 256;
constexpr int NP = 32;
constexpr int NPU = 30;
constexpr int BC = 8;
constexpr int NU = SHBYTES / 2048;
constexpr int I1 = 28;
constexpr int G1 = 16 * I1;
constexpr int NGR = G1 + 32 * (NI - I1);
constexpr int LU = 4;
constexpr int NLU = SHBYTES / 1024 / LU;
constexpr int GA = 160;
constexpr int RCAP = 16;
constexpr int RDIV = 54;
constexpr int ROFF = 130;
constexpr int NH = 84;
#define UPG_STR "3"

constexpr size_t WS_CNT = 0;
constexpr size_t WS_TMO = 32768;
constexpr size_t WS_TKT = 32768 + 256;
constexpr size_t WS_BEACON = 36864;
constexpr size_t WS_PLAN = 45056;
constexpr size_t WS_DUMMY = 49152;
constexpr size_t WS_REQ = 57344;
constexpr size_t WS_FLAG = 65536;
constexpr size_t WS_PROG = 131072;
constexpr size_t WS_LFLAG = 262144;
constexpr size_t WS_STAGE = 2u << 20;
constexpr size_t WS_NEED = WS_STAGE + 2ull * NR * SHBYTES;
constexpr unsigned long long DEADLINE_TICKS = 40000000ull;
static_assert(WS_BEACON + 16 * 128 <= WS_PLAN && WS_PLAN + 32 * 128 <= WS_DUMMY && WS_DUMMY + 32 * 128 <= WS_REQ && WS_REQ + 64 * 128 <= WS_FLAG &&
              WS_FLAG + (size_t)NR * 4096 <= WS_PROG && WS_PROG + 224 * 128 <= WS_LFLAG && WS_LFLAG + NLU * 128 <= WS_STAGE && NGR <= 1024, "layout");
static_assert(NPU <= GA && GA + 7 * RCAP <= G1 && NH <= 3 * NPU && RCAP < 64 && 7 * RCAP <= 2 * NH, "relay");
static_assert(NLU <= (NWG - NP) * 4, "one local unit per consumer wave");

struct Params {
    char* ws[NR];
    const char* x;
    const char* w;
    const float* sx;
    const float* sw;
    float* out;
    int rank;
    int pad;
};

__device__ __forceinline__ __amdgpu_buffer_rsrc_t mk_rsrc(const void* base, unsigned bytes) {
    return __builtin_amdgcn_make_buffer_rsrc((void*)base, 0, bytes, 0x00020000);
}

constexpr int WRING_BYTES = 4 * 4 * 8192;
constexpr int RED_BYTES = 4 * 4096;

__global__ __launch_bounds__(256, 1) void ag_gemm_kernel(const Params p) {
    __shared__ __attribute__((aligned(16))) char lds[WRING_BYTES + 2 * RED_BYTES];
    const int tid = threadIdx.x;
    const int lane = tid & 63;
    const int wid = __builtin_amdgcn_readfirstlane(tid >> 6);
    const int b = blockIdx.x;
    const unsigned rank = (unsigned)p.rank;
    char* const myws = p.ws[0];
    asm volatile("" :: "s"(p.ws[0]), "s"(p.ws[7]), "s"(p.x), "s"(p.w), "s"(p.out), "s"(p.rank));

    GAS unsigned* cnt = (GAS unsigned*)(myws + WS_CNT + (size_t)b * 128 + wid * 4);

    int rg = 0, cg = b;
    if (b >= NP) {
        const int i = (b & 7) * 28 + (b >> 3) - 4;
        rg = 1 + (i / BC) % 7;
        cg = (i / (7 * BC)) * BC + i % BC;
    }
    if (__builtin_expect(rg == 0, 1)) {
        if (wid != 0 || cg >= NPU) {
#define SLOAD(PTR) ({ unsigned sv_; asm volatile("s_load_dword %0, %1, 0x0 glc\n\ts_waitcnt lgkmcnt(0)" : "=&s"(sv_) : "s"(PTR) : "memory"); sv_; })
            const unsigned t0 = (unsigned)__builtin_amdgcn_s_memrealtime();
            const unsigned call = SLOAD(cnt) + 1u;
            const int pw = (cg - NPU) * 4 + wid;
            if (lane == 0 && pw != NR - 1) __hip_atomic_store(cnt, call, __ATOMIC_RELAXED, __HIP_MEMORY_SCOPE_AGENT);
            GAS unsigned* const tmo_ = (GAS unsigned*)(myws + WS_TMO);
            const unsigned tag = (call & 0xffffu) << 16, par1 = call & 1u;
            char* pws = p.ws[1];
#pragma unroll
            for (int i = 2; i < NR; ++i) pws = lane == i - 1 ? p.ws[i] : pws;
#define WAIT_FOR(COND) ({ bool ok_ = true; const unsigned long long w0_ = __builtin_amdgcn_s_memrealtime(); \
                for (int sp_ = 1; !(COND); ++sp_) { \
                    if ((sp_ & 15) == 0 && ((unsigned)__builtin_amdgcn_readfirstlane(__hip_atomic_load(tmo_, __ATOMIC_RELAXED, __HIP_MEMORY_SCOPE_AGENT)) == call || \
                                            __builtin_amdgcn_s_memrealtime() - w0_ > DEADLINE_TICKS)) { \
                        if (lane == 0) __hip_atomic_store(tmo_, call, __ATOMIC_RELAXED, __HIP_MEMORY_SCOPE_AGENT); \
                        ok_ = false; break; } \
                    __builtin_amdgcn_s_sleep(4); } ok_; })
            if (pw == NR - 1) {
                GAS unsigned* const bout = (GAS unsigned*)(pws + WS_BEACON + (par1 * NR + rank) * 128);
                const unsigned e0 = (unsigned)__builtin_amdgcn_s_memrealtime() - t0;
                unsigned bv = __hip_atomic_load((GAS unsigned*)(myws + WS_BEACON + (par1 * NR + ((rank + 1u + lane) & (NR - 1))) * 128), __ATOMIC_RELAXED, __HIP_MEMORY_SCOPE_SYSTEM);
                if (lane < NR - 1) __hip_atomic_store(bout, tag | (e0 & 0xffffu), __ATOMIC_RELAXED, __HIP_MEMORY_SCOPE_SYSTEM);
                asm volatile("" : "+v"(bv));
                const unsigned e1 = (unsigned)__builtin_amdgcn_s_memrealtime() - t0;
                if (lane < NR - 1) __hip_atomic_store(bout, tag | (e1 & 0xffffu), __ATOMIC_RELAXED, __HIP_MEMORY_SCOPE_SYSTEM);
                const int mine = (int)(e0 + e1) / 2;
                const int m = lane < NR - 1 && (bv & 0xffff0000u) == tag ? min(max((int)(bv & 0xffffu) + ROFF - mine, 0) / RDIV, RCAP) : 0;
                int st = GA;
#pragma unroll
                for (int i = 0; i < NR - 1; ++i) st += lane > i ? __builtin_amdgcn_readlane(m, i) : 0;
                const unsigned rq = tag | (unsigned)(lane < NR - 1 ? st << 6 | m : st - GA);
                const unsigned rq8 = (unsigned)__shfl((int)rq, lane & 7);
                char* const pws8 = (char*)(((size_t)(unsigned)__shfl((int)((size_t)pws >> 32), lane & 7) << 32) | (unsigned)__shfl((int)(size_t)pws, lane & 7));
#pragma unroll
                for (int i = 0; i < 4; ++i) {
                    __hip_atomic_store((GAS unsigned*)(myws + WS_PLAN + (i * 8 + (lane >> 3)) * 128) + (lane & 7), rq8, __ATOMIC_RELAXED, __HIP_MEMORY_SCOPE_AGENT);
                    if ((lane & 7) < NR - 1) __hip_atomic_store((GAS unsigned*)(pws8 + WS_REQ + (par1 * 32 + i * 8 + (lane >> 3)) * 128) + rank, rq8, __ATOMIC_RELAXED, __HIP_MEMORY_SCOPE_SYSTEM);
                }
                if (lane == 0) {
                    __hip_atomic_store((GAS unsigned*)(myws + WS_TKT + (par1 ^ 1u) * 128), 0u, __ATOMIC_RELAXED, __HIP_MEMORY_SCOPE_AGENT);
                    __hip_atomic_store(cnt, call, __ATOMIC_RELAXED, __HIP_MEMORY_SCOPE_AGENT);
                }
                for (unsigned e = 0; e < (unsigned)(RCAP * RDIV + 200); ) {
                    e = (unsigned)__builtin_amdgcn_s_memrealtime() - t0;
                    if (lane < NR - 1) __hip_atomic_store(bout, tag | min(e, 0xffffu), __ATOMIC_RELAXED, __HIP_MEMORY_SCOPE_SYSTEM);
                    __builtin_amdgcn_s_sleep(8);
                }
            } else if (pw < 0) {
                const int h = (wid - 1) * (NH / 3) + cg;
                if (cg >= NH / 3) return;
                const size_t soff = WS_STAGE + (size_t)par1 * (NR * SHBYTES) + lane * 16;
                u32x4 d[4];
                u32x8 pl;
                if (!WAIT_FOR(({ asm volatile("s_load_dwordx8 %0, %1, 0x0 glc\n\ts_waitcnt lgkmcnt(0)" : "=&s"(pl) : "s"(myws + WS_PLAN + cg * 128) : "memory");
                                (((pl[0] ^ tag) | (pl[1] ^ tag) | (pl[2] ^ tag) | (pl[3] ^ tag) | (pl[4] ^ tag) | (pl[5] ^ tag) | (pl[6] ^ tag) | (pl[7] ^ tag)) >> 16) == 0u; }))) return;
                unsigned rq = pl[0];
#pragma unroll
                for (int i = 1; i < NR - 1; ++i) rq = lane == i ? pl[i] : rq;
                {
                    u32x4 e[4];
                    char* tw[2];
#pragma unroll
                    for (int r = 0; r < 2; ++r) {
                        const int g = GA + h + r * NH;
                        const int to = __builtin_ctzll(__ballot(lane < NR - 1 && (unsigned)(g - (int)((rq >> 6) & 1023u)) < (rq & 63u)) | 128ull);
                        tw[r] = to >= NR - 1 || g >= G1 ? nullptr :
                            (char*)(((size_t)(unsigned)__builtin_amdgcn_readlane((int)((size_t)pws >> 32), to & 7) << 32) | (unsigned)__builtin_amdgcn_readlane((int)(size_t)pws, to & 7));
                        if (tw[r])
#pragma unroll
                            for (int i = 0; i < 4; ++i) (r ? e : d)[i] = *(const GAS u32x4*)(p.x + (size_t)((g >> 4) * IROWS + (lane & 15)) * KD + ((g & 15) * 4 + i) * 64 + (lane >> 4) * 16);
                    }
#define STORE4(D, A) asm volatile("global_store_dwordx4 %4, %0, off sc0 sc1\n\tglobal_store_dwordx4 %4, %1, off offset:1024 sc0 sc1\n\t" \
                                  "global_store_dwordx4 %4, %2, off offset:2048 sc0 sc1\n\tglobal_store_dwordx4 %4, %3, off offset:3072 sc0 sc1\n\ts_nop 1" \
                                  :: "v"(D[0]), "v"(D[1]), "v"(D[2]), "v"(D[3]), "v"(A) : "memory")
                    if (tw[1]) asm volatile("" : "+v"(e[0]), "+v"(e[1]), "+v"(e[2]), "+v"(e[3]));
                    if (tw[0]) STORE4(d, tw[0] + soff + (size_t)rank * SHBYTES + (size_t)(GA + h) * 4096);
                    if (tw[1]) STORE4(e, tw[1] + soff + (size_t)rank * SHBYTES + (size_t)(GA + h + NH) * 4096);
                    asm volatile("s_waitcnt vmcnt(0)" ::: "memory");
#pragma unroll
                    for (int r = 0; r < 2; ++r)
                        if (tw[r] && lane == 0) __hip_atomic_store((GAS unsigned*)(tw[r] + WS_FLAG) + rank * 1024 + GA + h + r * NH, call, __ATOMIC_RELAXED, __HIP_MEMORY_SCOPE_SYSTEM);
                }
                for (unsigned todo = 0xffu & ~(1u << rank); todo; ) {
                    unsigned ok = 0;
                    if (!WAIT_FOR(({ asm volatile("s_load_dwordx8 %0, %1, 0x0 glc\n\ts_waitcnt lgkmcnt(0)" : "=&s"(pl) : "s"(myws + WS_REQ + (par1 * 32 + cg) * 128) : "memory");
                                    ok = 0; _Pragma("unroll") for (int i = 0; i < NR; ++i) ok |= ((pl[i] ^ tag) >> 16) == 0u ? 1u << i : 0u; (ok &= todo) != 0u; }))) return;
                    const unsigned src = (unsigned)__builtin_ctz(ok);
                    todo &= ~(1u << src);
                    unsigned rs_ = pl[0];
#pragma unroll
                    for (int i = 1; i < NR; ++i) rs_ = src == (unsigned)i ? pl[i] : rs_;
                    const int si = (int)((src - rank - 1u) & (NR - 1));
                    const int k = (h + NH - 12 * si) % NH, g = (int)((rs_ >> 6) & 1023u) + k;
                    if (k >= (int)(rs_ & 63u) || g >= G1) continue;
                    if (!WAIT_FOR((int)((unsigned)__builtin_amdgcn_readfirstlane(__hip_atomic_load((GAS unsigned*)(myws + WS_FLAG) + src * 1024 + g, __ATOMIC_RELAXED, __HIP_MEMORY_SCOPE_SYSTEM)) - call) >= 0)) return;
                    const size_t go = soff + (size_t)src * SHBYTES + (size_t)g * 4096;
                    asm volatile("global_load_dwordx4 %0, %4, off sc0 sc1\n\tglobal_load_dwordx4 %1, %4, off offset:1024 sc0 sc1\n\t"
                                 "global_load_dwordx4 %2, %4, off offset:2048 sc0 sc1\n\tglobal_load_dwordx4 %3, %4, off offset:3072 sc0 sc1\n\ts_waitcnt vmcnt(0)"
                                 : "=&v"(d[0]), "=&v"(d[1]), "=&v"(d[2]), "=&v"(d[3]) : "v"(myws + go) : "memory");
#pragma unroll
                    for (int pr = 0; pr < NR - 1; ++pr)
                        if (pr != si) STORE4(d, p.ws[1 + pr] + go);
                    asm volatile("s_waitcnt vmcnt(0)" ::: "memory");
                    if (lane < NR - 1 && lane != si) __hip_atomic_store((GAS unsigned*)(pws + WS_FLAG) + src * 1024 + g, call, __ATOMIC_RELAXED, __HIP_MEMORY_SCOPE_SYSTEM);
                }
            } else {
                const __amdgpu_buffer_rsrc_t rf = mk_rsrc(myws + WS_FLAG + (size_t)((rank + 1u + (unsigned)pw) & (NR - 1)) * 4096, (unsigned)NGR * 4u);
                GAS unsigned* const prog = (GAS unsigned*)(myws + WS_PROG + (size_t)pw * 4096 + (lane & 31) * 128);
                GAS unsigned* const tmo = (GAS unsigned*)(myws + WS_TMO);
                const unsigned long long tw0 = __builtin_amdgcn_s_memrealtime();
                for (int rdy = 0, spins = 0; rdy < NI; ) {
                    const int g0 = rdy < I1 ? 16 * rdy : G1 + 32 * (rdy - I1);
                    const unsigned f = __builtin_amdgcn_raw_buffer_load_b32(rf, (lane & 31) * 4, g0 * 4, 17);
                    const unsigned m = ~(unsigned)__ballot((int)(f - call) >= 0);
                    const int k = m == 0u ? (rdy + 1 < I1 ? 2 : 1) : (rdy < I1 && (m & 0xffffu) == 0u ? 1 : 0);
                    if (k > 0) {
                        rdy += k;
                        if (lane < 32) __hip_atomic_store(prog, call * 64u + (unsigned)rdy, __ATOMIC_RELAXED, __HIP_MEMORY_SCOPE_AGENT);
                    } else if ((++spins & 15) == 0 &&
                               ((unsigned)__builtin_amdgcn_readfirstlane(__hip_atomic_load(tmo, __ATOMIC_RELAXED, __HIP_MEMORY_SCOPE_AGENT)) == call ||
                                __builtin_amdgcn_s_memrealtime() - tw0 > DEADLINE_TICKS)) {
                        if (lane == 0) __hip_atomic_store(tmo, call, __ATOMIC_RELAXED, __HIP_MEMORY_SCOPE_AGENT);
                        break;
                    }
                }
            }
            return;
        }
        const unsigned vx = (lane & 15) * KD + (lane >> 4) * 16;
#define PUSH_REGS "s94", "v64", "v65", "v66", "v67", "v68", "v69", "v70", "v71", "v72", "v73", "v74", "v75", "v76", "v77", "v78", "v79", "v80", "v81"
        int prev = 0, cur = 2 * cg, nxt = 2 * cg + 1;
#define PUSH_LOAD(D, U, PP) do { const unsigned pc_ = min((unsigned)(U), (unsigned)NU - 1u) * 2u + (PP); \
        asm volatile("global_load_dwordx4 v[%c2:%c2+3], %0, %1" :: "v"(vx + (pc_ >> 6) * IBYTES + (pc_ & 63u) * 64u), "s"(p.x), "i"(64 + 4 * (D)) : "memory"); } while (0)
        asm volatile("" ::: PUSH_REGS);
        PUSH_LOAD(0, cur, 0); PUSH_LOAD(1, cur, 1);
        asm volatile("global_load_dword v81, %0, off sc1" :: "v"(cnt) : "memory");
        char* fws = p.ws[1];
#pragma unroll
        for (int i = 2; i < NR; ++i) fws = lane == i - 1 ? p.ws[i] : fws;
        GAS unsigned* flag = (GAS unsigned*)(fws + WS_FLAG) + rank * 1024;
        GAS unsigned* sink = (GAS unsigned*)(myws + WS_DUMMY + cg * 128) + (lane & 7);
        unsigned vs = (unsigned)WS_STAGE + rank * SHBYTES + lane * 16;
        unsigned call;
        asm volatile("s_waitcnt vmcnt(0)\n\tv_readfirstlane_b32 %0, v81" : "=s"(call), "+v"(flag), "+v"(sink), "+v"(vs) :: "memory", PUSH_REGS);
        call += 1u;
        const unsigned par = (call & 1u) * (NR * SHBYTES);
        GAS unsigned* const tkt = (GAS unsigned*)(myws + WS_TKT + (call & 1u) * 128);
        int skip = -1;
#define ENDS_GROUP(U) ((U) >= 2 * G1 || ((U) & 1))
#define FLAG_ADDR(U) (ENDS_GROUP(U) ? flag + ((U) < 2 * G1 ? (U) >> 1 : (U) - G1) : sink)
#define PUSH_FLAG(U) asm volatile("s_mov_b64 exec, 0x7f\n\tglobal_store_dword %0, %1, off sc0 sc1\n\ts_mov_b64 exec, -1" :: "v"(FLAG_ADDR(U)), "v"(call) : "memory")
#define STORE7(R, O) "global_store_dwordx4 " O ", v[" R ":" R "+3], %[p1] sc0 sc1\n\tglobal_store_dwordx4 " O ", v[" R ":" R "+3], %[p2] sc0 sc1\n\t" \
                     "global_store_dwordx4 " O ", v[" R ":" R "+3], %[p3] sc0 sc1\n\tglobal_store_dwordx4 " O ", v[" R ":" R "+3], %[p4] sc0 sc1\n\t" \
                     "global_store_dwordx4 " O ", v[" R ":" R "+3], %[p5] sc0 sc1\n\tglobal_store_dwordx4 " O ", v[" R ":" R "+3], %[p6] sc0 sc1\n\t" \
                     "global_store_dwordx4 " O ", v[" R ":" R "+3], %[p7] sc0 sc1"
#define PEERS [p1] "s"(p.ws[1]), [p2] "s"(p.ws[2]), [p3] "s"(p.ws[3]), [p4] "s"(p.ws[4]), [p5] "s"(p.ws[5]), [p6] "s"(p.ws[6]), [p7] "s"(p.ws[7])
#define XOFF(U, PP) (vx + ((min((unsigned)(U), (unsigned)NU - 1u) * 2u + (PP)) >> 6) * IBYTES + ((min((unsigned)(U), (unsigned)NU - 1u) * 2u + (PP)) & 63u) * 64u)
#define PUSH_UNIT(DA, DB, DC, DD) \
            { const bool draw = (unsigned)nxt < (unsigned)NU && ENDS_GROUP(nxt); \
            if (skip < 0) asm volatile("s_load_dword s94, %0, 0x0 glc" :: "s"(myws + WS_PLAN + cg * 128 + 28) : "memory", "s94");     \
            asm volatile("s_mov_b64 exec, 1\n\tglobal_atomic_add v80, %0, %1, off sc0\n\ts_mov_b64 exec, -1" :: "v"(draw ? tkt : sink + 8), "v"(1u) : "memory"); \
            PUSH_LOAD(DC, nxt, 0); \
            asm volatile("s_waitcnt vmcnt(9)\n\ts_mov_b64 exec, 0x7f\n\tglobal_store_dword %[fa], %[cv], off sc0 sc1\n\ts_mov_b64 exec, -1\n\t" \
                         "global_load_dwordx4 v[%c[dd]:%c[dd]+3], %[xo], %[x]\n\t" STORE7("%c[db]", "%[o]") \
                         :: [fa] "v"(FLAG_ADDR(prev)), [cv] "v"(call), [dd] "i"(64 + 4 * (DD)), [xo] "v"(XOFF(nxt, 1)), [x] "s"(p.x), \
                            [db] "i"(64 + 4 * (DB)), [o] "v"(vs + (unsigned)cur * 2048u + 1024u), PEERS : "memory", PUSH_REGS); \
            prev = cur; cur = nxt; \
            if ((unsigned)cur >= (unsigned)NU) break; \
            unsigned tk, pv_; \
            asm volatile("s_waitcnt vmcnt(9) lgkmcnt(0)\n\t" STORE7("%c[dc]", "%[o]") "\n\tv_readfirstlane_b32 %[tk], v80\n\ts_mov_b32 %[pv], s94" \
                         : [tk] "=&s"(tk), [pv] "=&s"(pv_) : [dc] "i"(64 + 4 * (DC)), [o] "v"(vs + (unsigned)cur * 2048u), PEERS : "memory", PUSH_REGS); \
            if (skip < 0 && (pv_ >> 16) == (call & 0xffffu)) skip = (int)(pv_ & 0xffffu); \
            int g = (int)tk + NPU; \
            if (draw && g >= GA) { \
                for (int sp = 0; skip < 0; ++sp) {       \
                    unsigned pm; asm volatile("s_load_dword %0, %1, 0x0 glc\n\ts_waitcnt lgkmcnt(0)" : "=&s"(pm) : "s"(myws + WS_PLAN + cg * 128 + 28) : "memory"); \
                    if ((pm >> 16) == (call & 0xffffu)) skip = (int)(pm & 0xffffu); \
                    else if (sp > 1000000) { __hip_atomic_store((GAS unsigned*)(myws + WS_TMO), call, __ATOMIC_RELAXED, __HIP_MEMORY_SCOPE_AGENT); skip = 0; } \
                } \
                g += skip; \
            } \
            nxt = draw ? (g < G1 ? 2 * g : G1 + g) : nxt + 1; }
        vs += par;
        asm volatile(STORE7("64", "%[o]") :: [o] "v"(vs + (unsigned)cur * 2048u), PEERS : "memory", PUSH_REGS);
        for (;;) {
            PUSH_UNIT(0, 1, 2, 3)
            PUSH_UNIT(2, 3, 0, 1)
        }
        asm volatile("s_waitcnt vmcnt(0)" ::: "memory", PUSH_REGS);
        PUSH_FLAG(prev);
        if (lane == 0) __hip_atomic_store(cnt, call, __ATOMIC_RELAXED, __HIP_MEMORY_SCOPE_AGENT);
        return;
    }

    const int pi = rg - 1;
    const unsigned srank = (rank + 1u + (unsigned)pi) & (NR - 1);
    const int nloc = pi < 4 ? 5 : 4;
    const int l0 = pi < 4 ? 5 * pi : 4 * pi + 4;
    const int q = lane >> 4, n = lane & 15;
#define WAITV(N) asm volatile("s_waitcnt vmcnt(%c0)" :: "i"(N) : "memory")
    typedef const __attribute__((address_space(4))) float* cfloat_p;
    const int scale_s = __builtin_amdgcn_readfirstlane(__builtin_bit_cast(int, *(cfloat_p)p.sx * *(cfloat_p)p.sw));

    const int lu = (pi * 32 + cg) * 4 + wid;
    unsigned call;
    const char* stage;
    {
        const __amdgpu_buffer_rsrc_t rx = mk_rsrc(p.x, lu < NLU ? (unsigned)SHBYTES : 0u);
        const __amdgpu_buffer_rsrc_t rw = mk_rsrc(p.w, (unsigned)KD * NL);
#define PRO_REGS "v128", "v129", "v130", "v131", "v132", "v133", "v134", "v135", "v136", "v137", "v138", "v139", "v140", "v141", "v142", "v143", "v144", "v145"
        asm volatile("global_load_dword v144, %0, off sc1\n\t"
                     "buffer_load_dwordx4 v[128:131], %1, %2, %3 offen\n\tbuffer_load_dwordx4 v[132:135], %1, %2, %3 offen offset:64\n\t"
                     "buffer_load_dwordx4 v[136:139], %1, %2, %3 offen offset:128\n\tbuffer_load_dwordx4 v[140:143], %1, %2, %3 offen offset:192"
                     :: "v"(cnt), "v"(n * KD + q * 16), "s"(rx), "s"(((lu * LU) >> 6) * IBYTES + ((lu * LU) & 63) * 64) : "memory", PRO_REGS);
        const int vw0 = (lane >> 2) * NL + (((lane & 3) ^ (q & 1)) * 16), vw1 = (lane >> 2) * NL + (((lane & 3) ^ (q & 1) ^ 2) * 16);
        const int sw = (wid * 1024) * NL + cg * 64;
        const unsigned lw = (unsigned)(size_t)(LAS char*)lds + wid * 32768;
        unsigned tr[4];
#pragma unroll
        for (int j = 0; j < 4; ++j) tr[j] = lw + (q * 16 + (n >> 1)) * 64 + ((j ^ ((n >> 3) | ((q & 1) << 1))) * 16) + (n & 1) * 8;
#define W_DMA(KS) do { _Pragma("unroll") for (int i_ = 0; i_ < 8; ++i_) \
            asm volatile("s_mov_b32 m0, %3\n\ts_nop 0\n\tbuffer_load_dwordx4 %0, %1, %2 offen lds" \
                         :: "v"((i_ & 1) ? vw1 : vw0), "s"(rw), "s"(sw + ((KS) * 128 + i_ * 16) * NL), "s"(lw + ((KS) & 3) * 8192 + i_ * 1024) : "memory"); } while (0)
#define W_TR(KS) do { _Pragma("unroll") for (int j_ = 0; j_ < 4; ++j_) \
            asm volatile("ds_read_b64_tr_b8 a[%c1:%c1+1], %0 offset:%c2\n\tds_read_b64_tr_b8 a[%c1+2:%c1+3], %0 offset:%c2+512\n\t" \
                         "ds_read_b64_tr_b8 a[%c1+4:%c1+5], %0 offset:%c2+4096\n\tds_read_b64_tr_b8 a[%c1+6:%c1+7], %0 offset:%c2+4608" \
                         :: "v"(tr[j_]), "i"((KS) * 32 + j_ * 8), "i"(((KS) & 3) * 8192) : "memory"); \
            asm volatile("s_waitcnt lgkmcnt(0)" ::: "memory"); } while (0)
        asm volatile("" ::: "a0", "a1", "a2", "a3", "a4", "a5", "a6", "a7", "a8", "a9", "a10", "a11", "a12", "a13", "a14", "a15", "a16", "a17", "a18", "a19", "a20", "a21", "a22", "a23", "a24", "a25", "a26", "a27", "a28", "a29", "a30", "a31", "a32", "a33", "a34", "a35", "a36", "a37", "a38", "a39", "a40", "a41", "a42", "a43", "a44", "a45", "a46", "a47", "a48", "a49", "a50", "a51", "a52", "a53", "a54", "a55", "a56", "a57", "a58", "a59", "a60", "a61", "a62", "a63", "a64", "a65", "a66", "a67", "a68", "a69", "a70", "a71", "a72", "a73", "a74", "a75", "a76", "a77", "a78", "a79", "a80", "a81", "a82", "a83", "a84", "a85", "a86", "a87", "a88", "a89", "a90", "a91", "a92", "a93", "a94", "a95", "a96", "a97", "a98", "a99", "a100", "a101", "a102", "a103", "a104", "a105", "a106", "a107", "a108", "a109", "a110", "a111", "a112", "a113", "a114", "a115", "a116", "a117", "a118", "a119", "a120", "a121", "a122", "a123", "a124", "a125", "a126", "a127", "a128", "a129", "a130", "a131", "a132", "a133", "a134", "a135", "a136", "a137", "a138", "a139", "a140", "a141", "a142", "a143", "a144", "a145", "a146", "a147", "a148", "a149", "a150", "a151", "a152", "a153", "a154", "a155", "a156", "a157", "a158", "a159", "a160", "a161", "a162", "a163", "a164", "a165", "a166", "a167", "a168", "a169", "a170", "a171", "a172", "a173", "a174", "a175", "a176", "a177", "a178", "a179", "a180", "a181", "a182", "a183", "a184", "a185", "a186", "a187", "a188", "a189", "a190", "a191", "a192", "a193", "a194", "a195", "a196", "a197", "a198", "a199", "a200", "a201", "a202", "a203", "a204", "a205", "a206", "a207", "a208", "a209", "a210", "a211", "a212", "a213", "a214", "a215", "a216", "a217", "a218", "a219", "a220", "a221", "a222", "a223", "a224", "a225", "a226", "a227", "a228", "a229", "a230", "a231", "a232", "a233", "a234", "a235", "a236", "a237", "a238", "a239", "a240", "a241", "a242", "a243", "a244", "a245", "a246", "a247", "a248", "a249", "a250", "a251", "a252", "a253", "a254", "a255");
        constexpr int LPR = BC / 2, NT = (KD * LPR + 112 * 64 - 1) / (112 * 64);
#pragma unroll
        for (int i = 0; i < NT; ++i) {
            const int li = ((((b >> 3) - 4) * 4 + wid) * NT + i) * 64 + lane;
            asm volatile("buffer_load_dword v145, %0, %1, 0 offen" :: "v"((li / LPR) * NL + (cg / BC) * (BC * 64) + (li % LPR) * 128), "s"(rw) : "memory", PRO_REGS);
        }
        W_DMA(0);
        asm volatile("s_waitcnt vmcnt(%c1)\n\tv_readfirstlane_b32 %0, v144" : "=s"(call) : "i"(8 + NT) : "memory", PRO_REGS);
        call += 1u;
        stage = myws + WS_STAGE + (size_t)(call & 1u) * (NR * SHBYTES);
        const __amdgpu_buffer_rsrc_t rl = mk_rsrc(stage + (size_t)rank * SHBYTES, lu < NLU ? (unsigned)SHBYTES : 0u);
        asm volatile("global_store_dword %0, %1, off sc1\n\t"
                     "buffer_store_dwordx4 v[128:131], %2, %3, %4 offen sc0 sc1\n\tbuffer_store_dwordx4 v[132:135], %2, %3, %4 offen offset:1024 sc0 sc1\n\t"
                     "buffer_store_dwordx4 v[136:139], %2, %3, %4 offen offset:2048 sc0 sc1\n\tbuffer_store_dwordx4 v[140:143], %2, %3, %4 offen offset:3072 sc0 sc1"
                     :: "v"(cnt), "v"(call), "v"(lane * 16), "s"(rl), "s"(lu * (LU * 1024)) : "memory", PRO_REGS);
        W_DMA(1); W_DMA(2); W_DMA(3);
        WAITV(24);
        if (lane == 0 && lu < NLU) __hip_atomic_store((GAS unsigned*)(myws + WS_LFLAG + lu * 128), call, __ATOMIC_RELAXED, __HIP_MEMORY_SCOPE_SYSTEM);
        W_TR(0); W_DMA(4);
        WAITV(24); W_TR(1); W_DMA(5);
        WAITV(24); W_TR(2); W_DMA(6);
        WAITV(24); W_TR(3); W_DMA(7);
        WAITV(24); W_TR(4);
        WAITV(16); W_TR(5);
        WAITV(8); W_TR(6);
        WAITV(0); W_TR(7);
    }

    GAS unsigned* const tmo = (GAS unsigned*)(myws + WS_TMO);
    const __amdgpu_buffer_rsrc_t rs = mk_rsrc(stage, (unsigned)NR * SHBYTES);
    const char* const prog = myws + WS_PROG + (size_t)pi * 4096 + cg * 128;
    const int voff_a = lane * 16;
    const int voff_out = ((wid * 4 + q) * NL + n * 4) * 4;
    bool dead = false;
#define SPIN(READY_EXPR, LOADV) do { \
        const unsigned long long t0_ = __builtin_amdgcn_s_memrealtime(); \
        for (int spins_ = 1;; ++spins_) { \
            const unsigned v_ = LOADV; \
            if (READY_EXPR) break; \
            if ((spins_ & 15) == 0 && ((unsigned)__builtin_amdgcn_readfirstlane(__hip_atomic_load(tmo, __ATOMIC_RELAXED, __HIP_MEMORY_SCOPE_AGENT)) == call || \
                __builtin_amdgcn_s_memrealtime() - t0_ > DEADLINE_TICKS)) { \
                if (lane == 0) __hip_atomic_store(tmo, call, __ATOMIC_RELAXED, __HIP_MEMORY_SCOPE_AGENT); \
                dead = true; \
                rdy = NI; \
                break; \
            } \
            __builtin_amdgcn_s_sleep(1); \
        } } while (0)
    int rdy = 0;
    {
        const __amdgpu_buffer_rsrc_t rfl = mk_rsrc(myws + WS_LFLAG, (unsigned)NLU * 128u);
        constexpr int LPI = 64 / LU;
        SPIN(__all(lane >= nloc * 4 || (int)(v_ - call) >= 0), __builtin_amdgcn_raw_buffer_load_b32(rfl, (((lane >> 2) + l0) * LPI + wid * 4 + (lane & 3)) * 128, 0, 17));
    }

    const unsigned red = (unsigned)(size_t)(LAS char*)lds + WRING_BYTES;
    unsigned put[4];
#pragma unroll
    for (int j = 0; j < 4; ++j) put[j] = red + wid * 4096 + j * 1024 + q * 256 + ((n ^ (4 * j + q)) * 16);
    const unsigned get = red + (n >> 2) * 1024 + (n & 3) * 256 + (((4 * wid + q) ^ n) * 16);
    const __amdgpu_buffer_rsrc_t ro = mk_rsrc(p.out, (unsigned)NR * SH * NL * 4u);
    int tn = 1 - nloc, st = 1 + (nloc & 1);
    int sa = (int)rank * SHBYTES + l0 * IBYTES + wid * 16384;
    int sa1 = sa + IBYTES;
    int so2 = 0x7f000000;
    unsigned long long tw = 0;
    int tws = NI + 1;
    for (;;) {
        unsigned smp;
        rdy = __builtin_amdgcn_readfirstlane(rdy);
        asm volatile(
        ".macro MF B, ks, j\n"
        " v_mfma_f32_16x16x128_f8f6f4 v[110+4*\\j:113+4*\\j], a[32*\\ks+8*\\j:32*\\ks+8*\\j+7], v[128+64*\\B+8*\\ks:135+64*\\B+8*\\ks], v[110+4*\\j:113+4*\\j]\n"
        ".endm\n"
        ".macro MF0 B, j\n"
        " v_mfma_f32_16x16x128_f8f6f4 v[110+4*\\j:113+4*\\j], a[8*\\j:8*\\j+7], v[128+64*\\B:135+64*\\B], v[74:77]\n"
        ".endm\n"
        ".macro LH B, ks, hf, sb\n"
        " buffer_load_dwordx4 v[128+64*(\\B)+8*(\\ks)+4*(\\hf):131+64*(\\B)+8*(\\ks)+4*(\\hf)], v[70+((\\ks)>>1)], %[rs], \\sb offen offset:((\\ks)&1)*2048+(\\hf)*1024 sc0 sc1\n"
        ".endm\n"
        ".macro V4 op, d, a, c\n"
        " \\op v[\\d], v[\\a], v[\\c]\n"
        " \\op v[\\d+1], v[\\a+1], v[\\c+1]\n"
        " \\op v[\\d+2], v[\\a+2], v[\\c+2]\n"
        " \\op v[\\d+3], v[\\a+3], v[\\c+3]\n"
        ".endm\n"
        ".macro K4 op, d, k\n"
        " \\op v[\\d], \\k, v[\\d]\n"
        " \\op v[\\d+1], \\k, v[\\d+1]\n"
        " \\op v[\\d+2], \\k, v[\\d+2]\n"
        " \\op v[\\d+3], \\k, v[\\d+3]\n"
        ".endm\n"
        ".macro U2 op, d\n"
        " \\op v[\\d], v[\\d]\n"
        " \\op v[\\d+1], v[\\d+1]\n"
        ".endm\n"
        ".macro CLAMP d, a\n"
        " v_med3_f32 v[\\d], v[\\a], %[m60], %[c60]\n"
        " v_med3_f32 v[\\d+1], v[\\a+1], %[m60], %[c60]\n"
        " v_med3_f32 v[\\d+2], v[\\a+2], %[m60], %[c60]\n"
        " v_med3_f32 v[\\d+3], v[\\a+3], %[m60], %[c60]\n"
        ".endm\n"
        ".macro GET4 d, B\n"
        " ds_read_b128 v[\\d:\\d+3], %[get] offset:(\\B)*16384\n"
        " ds_read_b128 v[\\d+4:\\d+7], %[get] offset:(\\B)*16384+4096\n"
        " ds_read_b128 v[\\d+8:\\d+11], %[get] offset:(\\B)*16384+8192\n"
        " ds_read_b128 v[\\d+12:\\d+15], %[get] offset:(\\B)*16384+12288\n"
        ".endm\n"
        ".macro SAMPLE\n"
        " s_load_dword %[smp], %[prog], 0x0 glc\n"
        ".endm\n"
        ".macro TAKE\n"
        " s_sub_i32 %[smp], %[smp], %[c64]\n"
        " s_max_i32 %[rdy], %[rdy], %[smp]\n"
        ".endm\n"
        ".macro REQ1 B, H, K, hf\n"
        " .if \\H == 0\n"
        "  LH 1-\\B, \\K, \\hf, %[sa1]\n"
        " .endif\n"
        " .if \\H == 1 && \\K < 2\n"
        "  LH \\B, \\K+6, \\hf, %[sa]\n"
        " .endif\n"
        " .if \\hf == 1\n"
        "  .if \\H == 0\n"
        "   s_waitcnt vmcnt(14)\n"
        "  .elseif \\H == 1 && \\K < 2\n"
        "   s_waitcnt vmcnt(10)\n"
        "  .else\n"
        "   s_waitcnt vmcnt(12-2*\\K)\n"
        "  .endif\n"
        " .endif\n"
        ".endm\n"
        ".macro REQ2 B, H, K, hf\n"
        " .if \\H == 0\n"
        "  LH 1-\\B, \\K, \\hf, %[sa1]\n"
        " .endif\n"
        " .if \\H == 1 && \\K >= 6\n"
        "  LH 1-\\B, \\K-2, \\hf, %[sa1]\n"
        " .endif\n"
        " .if \\H == 3 || (\\H == 1 && \\K < 6)\n"
        "  LH 1-\\B, 2*\\K-8+\\hf, 0, %[sa1]\n"
        "  LH 1-\\B, 2*\\K-8+\\hf, 1, %[sa1]\n"
        " .endif\n"
        " .if \\hf == 1 && \\K < 7\n"
        "  .if \\H == 0\n"
        "   s_waitcnt vmcnt(14+\\K/6)\n"
        "  .elseif \\H == 1 && \\K == 6\n"
        "   s_waitcnt vmcnt(11)\n"
        "  .elseif \\H == 2\n"
        "   s_waitcnt vmcnt(12-2*\\K+\\K/6)\n"
        "  .else\n"
        "   s_waitcnt vmcnt(2*\\K+\\K/6)\n"
        "  .endif\n"
        " .endif\n"
        ".endm\n"
        ".macro HALF1 B, H\n"
        " ds_write_b128 %[put0], v[110:113] offset:(1-\\B)*16384\n"
        " .if \\H == 1\n"
        "  s_waitcnt vmcnt(10)\n"
        " .else\n"
        "  s_waitcnt vmcnt(14)\n"
        " .endif\n"
        " MF0 \\B, 0\n"
        " ds_write_b128 %[put1], v[114:117] offset:(1-\\B)*16384\n"
        " REQ1 \\B, \\H, 0, 0\n"
        " MF0 \\B, 1\n"
        " ds_write_b128 %[put2], v[118:121] offset:(1-\\B)*16384\n"
        " MF0 \\B, 2\n"
        " ds_write_b128 %[put3], v[122:125] offset:(1-\\B)*16384\n"
        " MF0 \\B, 3\n"
        " REQ1 \\B, \\H, 0, 1\n"
        " MF \\B, 1, 0\n"
        " .if \\H != 0\n"
        "  SAMPLE\n"
        " .endif\n"
        " REQ1 \\B, \\H, 1, 0\n"
        " MF \\B, 1, 1\n"
        " V4 v_add_f32, 78, 78, 82\n"
        " MF \\B, 1, 2\n"
        " V4 v_add_f32, 86, 86, 90\n"
        " MF \\B, 1, 3\n"
        " REQ1 \\B, \\H, 1, 1\n"
        " MF \\B, 2, 0\n"
        " REQ1 \\B, \\H, 2, 0\n"
        " .if \\H == 0\n"
        "  s_waitcnt lgkmcnt(0)\n"
        "  SAMPLE\n"
        " .endif\n"
        " MF \\B, 2, 1\n"
        " V4 v_add_f32, 78, 78, 86\n"
        " MF \\B, 2, 2\n"
        " K4 v_mul_f32, 78, %[scale]\n"
        " MF \\B, 2, 3\n"
        " REQ1 \\B, \\H, 2, 1\n"
        " MF \\B, 3, 0\n"
        " REQ1 \\B, \\H, 3, 0\n"
        " MF \\B, 3, 1\n"
        " CLAMP 82, 78\n"
        " MF \\B, 3, 2\n"
        " K4 v_mul_f32, 82, %[nl2e]\n"
        " MF \\B, 3, 3\n"
        " REQ1 \\B, \\H, 3, 1\n"
        " .if \\H != 0\n"
        "  s_waitcnt lgkmcnt(0)\n"
        "  TAKE\n"
        " .endif\n"
        " MF \\B, 4, 0\n"
        " s_barrier\n"
        ".endm\n"
        ".macro HALF2 B, H\n"
        " .if \\H != 0\n"
        "  SAMPLE\n"
        " .endif\n"
        " REQ2 \\B, \\H, 4, 0\n"
        " MF \\B, 4, 1\n"
        " U2 v_exp_f32, 82\n"
        " MF \\B, 4, 2\n"
        " U2 v_exp_f32, 84\n"
        " MF \\B, 4, 3\n"
        " REQ2 \\B, \\H, 4, 1\n"
        " MF \\B, 5, 0\n"
        " REQ2 \\B, \\H, 5, 0\n"
        " K4 v_add_f32, 82, 1.0\n"
        " MF \\B, 5, 1\n"
        " U2 v_rcp_f32, 82\n"
        " MF \\B, 5, 2\n"
        " U2 v_rcp_f32, 84\n"
        " MF \\B, 5, 3\n"
        " REQ2 \\B, \\H, 5, 1\n"
        " MF \\B, 6, 0\n"
        " REQ2 \\B, \\H, 6, 0\n"
        " V4 v_mul_f32, 78, 78, 82\n"
        " MF \\B, 6, 1\n"
        " buffer_store_dwordx4 v[78:81], %[vout], %[ro], %[so2] offen sc1\n"
        " MF \\B, 6, 2\n"
        " s_add_i32 %[so2], %[so2], 0x20000\n"
        " s_cmp_eq_u32 %[tn], %[tA]\n"
        " s_cselect_b32 %[so2], %[ol], %[so2]\n"
        " s_cmp_eq_u32 %[tn], 2\n"
        " s_cselect_b32 %[so2], %[orm], %[so2]\n"
        " MF \\B, 6, 3\n"
        " REQ2 \\B, \\H, 6, 1\n"
        " MF \\B, 7, 0\n"
        " REQ2 \\B, \\H, 7, 0\n"
        " GET4 78, 1-\\B\n"
        " MF \\B, 7, 1\n"
        " REQ2 \\B, \\H, 7, 1\n"
        " s_mov_b32 %[sa], %[sa1]\n"
        " s_add_i32 %[sa1], %[sa1], 0x10000\n"
        " s_add_i32 %[tn], %[tn], 1\n"
        " MF \\B, 7, 2\n"
        " s_cmp_eq_u32 %[tn], 0\n"
        " s_cselect_b32 %[sa1], %[sr], %[sa1]\n"
        " s_waitcnt lgkmcnt(0)\n"
        " TAKE\n"
        " s_cmp_lt_i32 %[tn], %[rdy]\n"
        " MF \\B, 7, 3\n"
        ".endm\n"
        ".macro BSIDE B\n"
        "Lcold\\B\\()_%=:\n"
        " s_cmp_le_i32 %[tn], %[rdy]\n"
        " s_cbranch_scc1 Lgo\\B\\()_%=\n"
        " s_movk_i32 %[st], 64\n"
        "Lspin\\B\\()_%=:\n"
        " SAMPLE\n"
        " s_waitcnt lgkmcnt(0)\n"
        " TAKE\n"
        " s_cmp_le_i32 %[tn], %[rdy]\n"
        " s_cbranch_scc1 Lgo\\B\\()_%=\n"
        " s_sub_u32 %[st], %[st], 1\n"
        " s_cbranch_scc0 Lspin\\B\\()_%=\n"
        " s_movk_i32 %[st], 1+\\B\n"
        " s_branch Lexit_%=\n"
        "Lgo\\B\\()_%=:\n"
        " LH \\B, 0, 0, %[sa]\n"
        " LH \\B, 0, 1, %[sa]\n"
        " LH \\B, 1, 0, %[sa]\n"
        " LH \\B, 1, 1, %[sa]\n"
        " LH \\B, 2, 0, %[sa]\n"
        " LH \\B, 2, 1, %[sa]\n"
        " LH \\B, 3, 0, %[sa]\n"
        " LH \\B, 3, 1, %[sa]\n"
        " LH \\B, 4, 0, %[sa]\n"
        " LH \\B, 4, 1, %[sa]\n"
        " LH \\B, 5, 0, %[sa]\n"
        " LH \\B, 5, 1, %[sa]\n"
        "Lb\\B\\()_%=:\n"
        " HALF1 \\B, 1\n"
        "Lmid\\B\\()_%=:\n"
        " s_cmp_ge_i32 %[tn], %[rdy]\n"
        " s_cbranch_scc1 Lc2\\B\\()_%=\n"
        " s_add_i32 %[st], %[tn], " UPG_STR "\n"
        " s_cmp_lt_i32 %[st], %[rdy]\n"
        " s_cbranch_scc1 Ld2\\B\\()_%=\n"
        " HALF2 \\B, 1\n"
        ".endm\n"
        ".macro OTHER B, N\n"
        "Lc1\\B\\()_%=:\n"
        " HALF1 \\B, 2\n"
        " s_branch Lmid\\B\\()_%=\n"
        "Lc2\\B\\()_%=:\n"
        " .if \\B == 1\n"
        "  s_cmp_eq_u32 %[tn], 32\n"
        "  s_cbranch_scc1 Lfin_%=\n"
        " .endif\n"
        " HALF2 \\B, 2\n"
        " s_branch Lcold\\N\\()_%=\n"
        "Ld2\\B\\()_%=:\n"
        " HALF2 \\B, 3\n"
        " s_cbranch_scc1 La\\N\\()_%=\n"
        " s_branch Lc1\\N\\()_%=\n"
        ".endm\n"
        " s_nop 4\n"
        " v_mov_b32 v74, %[cin]\n"
        " v_mov_b32 v75, %[cin]\n"
        " v_mov_b32 v76, %[cin]\n"
        " v_mov_b32 v77, %[cin]\n"
        " v_mov_b32 v70, %[va]\n"
        " v_add_u32 v71, 0x1000, %[va]\n"
        " v_add_u32 v72, 0x2000, %[va]\n"
        " v_add_u32 v73, 0x3000, %[va]\n"
        " s_cmp_eq_u32 %[st], 2\n"
        " s_cbranch_scc1 Lcold1_%=\n"
        " BSIDE 0\n"
        " s_branch Lb1_%=\n"
        " BSIDE 1\n"
        " s_branch Lb0_%=\n"
        "La0_%=:\n"
        " HALF1 0, 0\n"
        " HALF2 0, 0\n"
        " s_cbranch_scc0 Lc11_%=\n"
        "La1_%=:\n"
        " HALF1 1, 0\n"
        " HALF2 1, 0\n"
        " s_cbranch_scc1 La0_%=\n"
        " OTHER 0, 1\n"
        " OTHER 1, 0\n"
        "Lfin_%=:\n"
        " GET4 94, 0\n"
        " MF 1, 4, 1\n"
        " U2 v_exp_f32, 82\n"
        " MF 1, 4, 2\n"
        " U2 v_exp_f32, 84\n"
        " MF 1, 4, 3\n"
        " s_waitcnt vmcnt(4)\n"
        " MF 1, 5, 0\n"
        " K4 v_add_f32, 82, 1.0\n"
        " MF 1, 5, 1\n"
        " U2 v_rcp_f32, 82\n"
        " MF 1, 5, 2\n"
        " U2 v_rcp_f32, 84\n"
        " MF 1, 5, 3\n"
        " s_waitcnt vmcnt(0) lgkmcnt(0)\n"
        " MF 1, 6, 0\n"
        " V4 v_mul_f32, 78, 78, 82\n"
        " MF 1, 7, 0\n"
        " buffer_store_dwordx4 v[78:81], %[vout], %[ro], %[so2] offen sc1\n"
        " V4 v_add_f32, 94, 94, 98\n"
        " MF 1, 6, 1\n"
        " V4 v_add_f32, 102, 102, 106\n"
        " MF 1, 7, 1\n"
        " V4 v_add_f32, 94, 94, 102\n"
        " MF 1, 6, 2\n"
        " ds_write_b128 %[put0], v[110:113] offset:16384\n"
        " K4 v_mul_f32, 94, %[scale]\n"
        " MF 1, 7, 2\n"
        " CLAMP 98, 94\n"
        " MF 1, 6, 3\n"
        " ds_write_b128 %[put1], v[114:117] offset:16384\n"
        " K4 v_mul_f32, 98, %[nl2e]\n"
        " MF 1, 7, 3\n"
        " U2 v_exp_f32, 98\n"
        " U2 v_exp_f32, 100\n"
        " ds_write_b128 %[put2], v[118:121] offset:16384\n"
        " K4 v_add_f32, 98, 1.0\n"
        " U2 v_rcp_f32, 98\n"
        " U2 v_rcp_f32, 100\n"
        " ds_write_b128 %[put3], v[122:125] offset:16384\n"
        " s_add_i32 %[so2], %[so2], 0x20000\n"
        " V4 v_mul_f32, 94, 94, 98\n"
        " buffer_store_dwordx4 v[94:97], %[vout], %[ro], %[so2] offen sc1\n"
        " s_waitcnt lgkmcnt(0)\n"
        " s_barrier\n"
        " GET4 78, 1\n"
        " s_add_i32 %[so2], %[so2], 0x20000\n"
        " s_waitcnt lgkmcnt(2)\n"
        " V4 v_add_f32, 78, 78, 82\n"
        " s_waitcnt lgkmcnt(0)\n"
        " V4 v_add_f32, 86, 86, 90\n"
        " V4 v_add_f32, 78, 78, 86\n"
        " K4 v_mul_f32, 78, %[scale]\n"
        " CLAMP 82, 78\n"
        " K4 v_mul_f32, 82, %[nl2e]\n"
        " U2 v_exp_f32, 82\n"
        " U2 v_exp_f32, 84\n"
        " s_mov_b32 %[st], 0\n"
        " K4 v_add_f32, 82, 1.0\n"
        " U2 v_rcp_f32, 82\n"
        " U2 v_rcp_f32, 84\n"
        " s_nop 0\n"
        " V4 v_mul_f32, 78, 78, 82\n"
        " buffer_store_dwordx4 v[78:81], %[vout], %[ro], %[so2] offen sc1\n"
        "Lexit_%=:\n"
        ".purgem MF\n"
        ".purgem MF0\n"
        ".purgem LH\n"
        ".purgem V4\n"
        ".purgem K4\n"
        ".purgem U2\n"
        ".purgem CLAMP\n"
        ".purgem GET4\n"
        ".purgem SAMPLE\n"
        ".purgem TAKE\n"
        ".purgem REQ1\n"
        ".purgem REQ2\n"
        ".purgem HALF1\n"
        ".purgem HALF2\n"
        ".purgem BSIDE\n"
        ".purgem OTHER\n"
            : [tn] "+&s"(tn), [rdy] "+&s"(rdy), [sa] "+&s"(sa), [sa1] "+&s"(sa1), [so2] "+&s"(so2), [st] "+&s"(st), [smp] "=&s"(smp)
            : [rs] "s"(rs), [ro] "s"(ro), [prog] "s"(prog), [c64] "s"(call * 64u), [scale] "s"(scale_s), [m60] "s"(-60.0f),
              [nl2e] "s"(-1.44269504088896340736f), [tA] "s"(2 - nloc), [ol] "s"((((int)rank * SH + l0 * IROWS) * NL + cg * 64) * 4),
              [orm] "s"((((int)srank * SH) * NL + cg * 64) * 4), [sr] "s"((int)srank * SHBYTES + wid * 16384), [cin] "s"(__builtin_amdgcn_readfirstlane(dead ? 0x7fc00000 : 0)),
              [va] "v"(voff_a), [vout] "v"(voff_out), [c60] "v"(60.0f), [get] "v"(get), [put0] "v"(put[0]), [put1] "v"(put[1]), [put2] "v"(put[2]), [put3] "v"(put[3])
            : "memory", "scc", "v70", "v71", "v72", "v73", "v74", "v75", "v76", "v77", "v78", "v79", "v80", "v81", "v82", "v83", "v84", "v85", "v86", "v87", "v88", "v89", "v90", "v91", "v92", "v93", "v94", "v95", "v96", "v97", "v98", "v99", "v100", "v101", "v102", "v103", "v104", "v105", "v106", "v107", "v108", "v109", "v110", "v111", "v112", "v113", "v114", "v115", "v116", "v117", "v118", "v119", "v120", "v121", "v122", "v123", "v124", "v125", "v126", "v127", "v128", "v129", "v130", "v131", "v132", "v133", "v134", "v135", "v136", "v137", "v138", "v139", "v140", "v141", "v142", "v143", "v144", "v145", "v147", "v148", "v149", "v150", "v151", "v152", "v153", "v154", "v155", "v156", "v157", "v158", "v159", "v160", "v161", "v162", "v163", "v164", "v165", "v166", "v167", "v168", "v169", "v170", "v171", "v172", "v173", "v174", "v175", "v176", "v177", "v178", "v179", "v180", "v181", "v182", "v183", "v184", "v185", "v186", "v187", "v188", "v189", "v190", "v191", "v192", "v193", "v194", "v195", "v196", "v197", "v198", "v199", "v200", "v201", "v202", "v203", "v204", "v205", "v206", "v207", "v208", "v209", "v210", "v211", "v212", "v213", "v214", "v215", "v216", "v217", "v218", "v219", "v220", "v221", "v222", "v223", "v224", "v225", "v226", "v227", "v228", "v229", "v230", "v231", "v232", "v233", "v234", "v235", "v236", "v237", "v238", "v239", "v240", "v241", "v242", "v243", "v244", "v245", "v246", "v247", "v248", "v249", "v250", "v251", "v252", "v253", "v254", "v255");
        if (st == 0) break;
        const unsigned long long now = __builtin_amdgcn_s_memrealtime();
        if (tws != tn) { tws = tn; tw = now; }
        if ((unsigned)__builtin_amdgcn_readfirstlane(__hip_atomic_load(tmo, __ATOMIC_RELAXED, __HIP_MEMORY_SCOPE_AGENT)) == call || now - tw > DEADLINE_TICKS) {
            if (lane == 0) __hip_atomic_store(tmo, call, __ATOMIC_RELAXED, __HIP_MEMORY_SCOPE_AGENT);
            dead = true;
            rdy = NI;
        }
    }
}

}

extern "C" void kernel_launch(int rank, int nranks, ncclComm_t comm,
                              hipStream_t stream,
                              void* const* d_in, const int* in_sizes, int n_in,
                              void* d_out, int out_size,
                              const int* in_dtypes, int out_dtype,
                              void* const* d_out_all,
                              void* const* d_ws_all, size_t ws_bytes) {
    (void)comm; (void)in_dtypes; (void)out_dtype; (void)d_out_all;
    if (nranks != NR || rank < 0 || rank >= NR || n_in < 4 || in_sizes[0] != SH * KD || in_sizes[1] != KD * NL ||
        out_size != NR * SH * NL || ws_bytes < WS_NEED) {
        fprintf(stderr, "ag_gemm: unexpected problem shape\n");
        return;
    }
    Params p;
    for (int i = 0; i < NR; ++i) p.ws[i] = (char*)d_ws_all[(rank + i) % NR];
    p.x = (const char*)d_in[0];
    p.w = (const char*)d_in[1];
    p.sx = (const float*)d_in[2];
    p.sw = (const float*)d_in[3];
    p.out = (float*)d_out;
    p.rank = rank;
    p.pad = 0;
    ag_gemm_kernel<<<dim3(NWG), dim3(256), 0, stream>>>(p);
    HIP_CHECK(hipGetLastError());
}
```
